```python
import numpy as np

kernel_name = "gpu_ag_gemm_m8192_k8192_n8192_bf16_gelu"


ABSMAX_RTOL = 0.008

COMPUTE_DTYPE = "bfloat16"
EPILOGUE = "gelu"
IN_QUANT = None
OUT_QUANT = None
IN_DTYPES = {"x": "bfloat16", "w_mat": "bfloat16"}
WORLD_SIZE = 8

def _to_bf16(x):
    u = np.ascontiguousarray(x, dtype=np.float32).view(np.uint32)
    return ((u + 0x7FFF + ((u >> 16) & 1)) & 0xFFFF0000).view(np.float32)

def setup_inputs():

    x = _to_bf16(np.random.randn(8192, 8192).astype(np.float32))
    w_mat = _to_bf16((np.random.randn(8192, 8192) / np.sqrt(8192)).astype(np.float32))
    return {"x": x, "w_mat": w_mat}

def _epilogue(y):
    c = 0.7978845608028654
    return (0.5 * y * (1.0 + np.tanh(c * (y + 0.044715 * y ** 3)))).astype(np.float32)

def reference(x, w_mat):
    y = (x @ w_mat).astype(np.float32)
    return _epilogue(y)
```

```cpp
#include <hip/hip_runtime.h>
#include <rccl/rccl.h>
#include <cstdio>

#define HIP_CHECK(expr) do { hipError_t _e = (expr); if (_e != hipSuccess) { \
    fprintf(stderr, "HIP error %s at %s:%d\n", hipGetErrorString(_e), __FILE__, __LINE__); \
    return; } } while (0)

namespace agk {

#define LAS __attribute__((address_space(3)))
typedef unsigned short u16;
typedef short s16x4 __attribute__((ext_vector_type(4)));
typedef short s16x8 __attribute__((ext_vector_type(8)));
typedef __bf16 bf16x8 __attribute__((ext_vector_type(8)));
typedef float f32x4 __attribute__((ext_vector_type(4)));
typedef unsigned u32x4 __attribute__((ext_vector_type(4)));
typedef unsigned u32x2 __attribute__((ext_vector_type(2)));

constexpr int NR = 8;
constexpr int KDIM = 8192;
constexpr unsigned XROW = KDIM * 2;
constexpr int NT = KDIM / 64;
constexpr int NCH = KDIM / 128;
constexpr int NTHREADS = 512;
constexpr int NWG = 256;
constexpr int NCOPY = 28;
constexpr int GEMM0 = 32;
constexpr int NGEMM = NWG - GEMM0;
constexpr int NG2 = 128;
constexpr int STAGES = 3;
constexpr int LDS_BYTES = STAGES * (256 * 128 + 64 * 256);
constexpr unsigned long long DEADLINE = 50000000ull;

constexpr size_t WS_CTR = 0;
constexpr size_t WS_TMO = 0x4000;
constexpr size_t WS_Q = 0x4100;
constexpr size_t WS_FA = 0x8000;
constexpr size_t WS_FB = 0x9000;
constexpr size_t WS_FO = 0xA000;
constexpr size_t WS_DBG = 0x10000;
constexpr size_t WS_DATA = 1u << 20;
constexpr size_t XR_BYTES = (size_t)3072 * XROW;
constexpr size_t WR_BYTES = (size_t)2 * KDIM * 1024;
constexpr size_t PAR_BYTES = XR_BYTES + WR_BYTES;
constexpr size_t WS_NEED = WS_DATA + 2 * PAR_BYTES;

struct Params {
    const char* x; const char* w; char* out;
    char* ws[NR]; char* outs[NR];
    int rank;
};

__device__ __forceinline__ unsigned ld_sys(const unsigned* p) { return __hip_atomic_load(p, __ATOMIC_RELAXED, __HIP_MEMORY_SCOPE_SYSTEM); }
__device__ __forceinline__ void st_sys(unsigned* p, unsigned v) { __hip_atomic_store(p, v, __ATOMIC_RELAXED, __HIP_MEMORY_SCOPE_SYSTEM); }
__device__ __forceinline__ unsigned ld_agent(const unsigned* p) { return __hip_atomic_load(p, __ATOMIC_RELAXED, __HIP_MEMORY_SCOPE_AGENT); }
__device__ __forceinline__ void st_agent(unsigned* p, unsigned v) { __hip_atomic_store(p, v, __ATOMIC_RELAXED, __HIP_MEMORY_SCOPE_AGENT); }
__device__ __forceinline__ bool geq(unsigned v, unsigned call) { return (int)(v - call) >= 0; }
__device__ __forceinline__ bool thread0() { int t = threadIdx.x; asm volatile("" : "+v"(t)); return t == 0; }
#define WAIT_V(n) asm volatile("s_waitcnt vmcnt(" #n ")" ::: "memory")
#define WAIT_L(n) asm volatile("s_waitcnt lgkmcnt(" #n ")" ::: "memory")
#define BAR __builtin_amdgcn_s_barrier()
#define SCHED __builtin_amdgcn_sched_barrier(0)

__device__ __forceinline__ void st16_sys(void* p, u32x4 v) {
    asm volatile("global_store_dwordx4 %0, %1, off sc0 sc1\n\ts_nop 1" :: "v"(p), "v"(v) : "memory");
}
__device__ __forceinline__ unsigned long long poll_once(const unsigned* p, bool need, unsigned call) {
    unsigned v;
    asm volatile("global_load_dword %0, %1, off sc0 sc1\n\ts_waitcnt vmcnt(0)" : "=v"(v) : "v"(p) : "memory");
    return __ballot(!need || geq(v, call));
}
__device__ __forceinline__ bool spin_check(unsigned* tmo, unsigned call, unsigned long long t0) {
    unsigned tv;
    asm volatile("global_load_dword %0, %1, off sc1\n\ts_waitcnt vmcnt(0)" : "=v"(tv) : "v"(tmo) : "memory");
    if (__builtin_amdgcn_readfirstlane(tv) == call) return true;
    if (__builtin_amdgcn_s_memrealtime() - t0 > DEADLINE) { st_agent(tmo, call); return true; }
    __builtin_amdgcn_s_sleep(2);
    return false;
}
__device__ __forceinline__ void wait_all(const unsigned* p, bool need, unsigned call, unsigned* tmo) {
    if (poll_once(p, need, call) == ~0ull) return;
    const unsigned long long t0 = __builtin_amdgcn_s_memrealtime();
    for (;;) {
        if (spin_check(tmo, call, t0)) return;
        if (poll_once(p, need, call) == ~0ull) return;
    }
}

__host__ __device__ __forceinline__ int lds_byte(int r, int c) { const int st = (r >> 4) * 2 + (c >> 5), rr = r & 15, cc = c & 31, ob = rr * 64 + cc * 2; return st * 1024 + (ob ^ (((ob >> 9) & 1) << 5)); }
__host__ __device__ __forceinline__ void stage_rc(int b, int& R, int& C) { const int st = b / 1024, sb = b % 1024, swz = sb ^ (((sb >> 9) & 1) << 5); R = (st >> 1) * 16 + swz / 64; C = (st & 1) * 32 + (swz % 64) / 2; }
__device__ __forceinline__ unsigned cvt_pk_bf16(float lo, float hi) { unsigned r; asm volatile("v_cvt_pk_bf16_f32 %0, %1, %2" : "=v"(r) : "v"(lo), "v"(hi)); return r; }
__device__ __forceinline__ float gelu_tanh(float y) {
    const float k1 = -2.0f * 0.7978845608028654f * 1.4426950408889634f;
    const float k2 = k1 * 0.044715f;
    const float t = y * (k2 * (y * y) + k1);
    return y * __builtin_amdgcn_rcpf(1.0f + __builtin_amdgcn_exp2f(t));
}

struct Tile {
    const char* A; const char* B; char* O;
    unsigned ldb;
    const unsigned* fA; const unsigned* fB;
    unsigned* fO;
};

template <int BM>
__device__ __forceinline__ void gemm_tile(LAS unsigned char* lds, const Tile& tl, const unsigned call, unsigned* tmo) {
    constexpr int WAVES_M = BM == 256 ? 4 : 2, WAVES_N = 8 / WAVES_M;
    constexpr int MF = BM / WAVES_M / 16, NF = 128 / WAVES_N / 16;
    constexpr int ABYTES = BM * 128, SBYTES = ABYTES + 16384;
    constexpr int RA = ABYTES / 8192;
    const int tid = threadIdx.x, wid = __builtin_amdgcn_readfirstlane(tid >> 6), lane = tid & 63;
    const int wr = wid / WAVES_N, wc = wid % WAVES_N, fr = lane & 15, fq = lane >> 4;
    unsigned voffA[RA], voffB[2];
#pragma unroll
    for (int i = 0; i < RA; ++i) { int R, C; stage_rc(tid * 16 + i * 8192, R, C); voffA[i] = (unsigned)R * XROW + (unsigned)C * 2u; }
#pragma unroll
    for (int i = 0; i < 2; ++i) { const int b = tid * 16 + i * 8192, row = b >> 8, chpos = (b >> 4) & 15, ch = chpos ^ (((row & 3) << 2) | ((row >> 2) & 3));
        voffB[i] = (unsigned)row * tl.ldb + (unsigned)ch * 16u; }
    const size_t kstepB = (size_t)64 * tl.ldb;
    const unsigned ldsbase = (unsigned)(size_t)lds;
    const unsigned ldsw = (unsigned)wid * 1024u;
    const unsigned aoff = ldsbase + (unsigned)lds_byte(wr * (MF * 16) + fr, fq * 8);
    unsigned boff0;
    { const int g = lane >> 4, q = (lane & 15) >> 2, p = lane & 3; const int row = 8 * g + q, ch = 2 * (wc * NF) + (p >> 1);
      boff0 = (unsigned)(ABYTES + 256 * row + 16 * (ch ^ (((row & 3) << 2) | ((row >> 2) & 3))) + 8 * (p & 1)); }
    unsigned boff[NF][2];
#pragma unroll
    for (int n = 0; n < NF; ++n)
#pragma unroll
        for (int t = 0; t < 2; ++t) boff[n][t] = ldsbase + (boff0 ^ (unsigned)(n * 32 + t * 16)) + (unsigned)(t * 1024);

    f32x4 acc[MF][NF];
#pragma unroll
    for (int m = 0; m < MF; ++m)
#pragma unroll
        for (int n = 0; n < NF; ++n) acc[m][n] = (f32x4){0.f, 0.f, 0.f, 0.f};

    int ready = (tl.fA == nullptr && tl.fB == nullptr) ? NCH : 0;
    auto ensure = [&](int c) {
        if (c >= ready) {
            const int idx = c + (lane & 31); const bool isA = lane < 32;
            const unsigned* base = isA ? tl.fA : tl.fB;
            const bool need = base != nullptr && idx < NCH;
            const unsigned* p = need ? base + idx : tmo;
            unsigned long long mk = poll_once(p, need, call);
            unsigned both = (unsigned)mk & (unsigned)(mk >> 32);
            if (!(both & 1u)) {
                const unsigned long long t0 = __builtin_amdgcn_s_memrealtime();
                for (;;) {
                    if (spin_check(tmo, call, t0)) { both = ~0u; break; }
                    mk = poll_once(p, need, call); both = (unsigned)mk & (unsigned)(mk >> 32);
                    if (both & 1u) break;
                }
            }
            ready = c + (both == ~0u ? 32 : __builtin_ctz(~both));
        }
    };
#define STAGE_TILE(kt, sb) do { const char* _a = tl.A + (size_t)(kt) * 128; const char* _b = tl.B + (size_t)(kt) * kstepB; \
        _Pragma("unroll") for (int _i = 0; _i < RA; ++_i) __builtin_amdgcn_global_load_lds((const unsigned*)(_a + voffA[_i]), (LAS unsigned*)(lds + (sb) + ldsw + _i * 8192), 16, 0, 0); \
        _Pragma("unroll") for (int _i = 0; _i < 2; ++_i) __builtin_amdgcn_global_load_lds((const unsigned*)(_b + voffB[_i]), (LAS unsigned*)(lds + (sb) + ABYTES + ldsw + _i * 8192), 16, 0, 0); } while (0)
#define RD128(dst, addr, off) asm volatile("ds_read_b128 %0, %1 offset:%2" : "=v"(dst) : "v"(addr), "i"(off) : "memory")
#define TRRD(dst, addr, off) asm volatile("ds_read_b64_tr_b16 %0, %1 offset:%2" : "=v"(dst) : "v"(addr), "i"(off) : "memory")

    if (wid == 0) ensure(0);
    BAR;
    STAGE_TILE(0, 0);
    STAGE_TILE(1, SBYTES);
    unsigned sb_rd = 0, sb_wr = 2 * SBYTES;
    for (int t = 0; t < NT; ++t) {
        const int tn = t + 2 < NT ? t + 2 : NT - 1;
        if (wid == 0 && (t & 1) == 0) ensure(tn >> 1);
        if constexpr (RA == 4) WAIT_V(6); else WAIT_V(3);
        BAR;
        STAGE_TILE(tn, sb_wr);
        s16x8 Af[2][MF]; s16x4 Bl[2][NF], Bh[2][NF];
        const unsigned a = aoff + sb_rd;
#pragma unroll
        for (int k = 0; k < 2; ++k) {
#pragma unroll
            for (int m = 0; m < MF; ++m) RD128(Af[k][m], a, m * 2048 + k * 1024);
#pragma unroll
            for (int n = 0; n < NF; ++n) { TRRD(Bl[k][n], boff[n][0] + sb_rd, k * 8192); TRRD(Bh[k][n], boff[n][1] + sb_rd, k * 8192); }
        }
#pragma unroll
        for (int k = 0; k < 2; ++k) {
            if (k == 0) { if constexpr (BM == 256) WAIT_L(12); else WAIT_L(6); } else WAIT_L(0);
            SCHED;
            __builtin_amdgcn_s_setprio(1);
#pragma unroll
            for (int m = 0; m < MF; ++m)
#pragma unroll
                for (int n = 0; n < NF; ++n) {
                    const s16x8 bf = __builtin_shufflevector(Bl[k][n], Bh[k][n], 0, 1, 2, 3, 4, 5, 6, 7);
                    acc[m][n] = __builtin_amdgcn_mfma_f32_16x16x32_bf16(__builtin_bit_cast(bf16x8, bf), __builtin_bit_cast(bf16x8, Af[k][m]), acc[m][n], 0, 0, 0);
                }
            __builtin_amdgcn_s_setprio(0);
            SCHED;
        }
        sb_rd = sb_rd == 2 * SBYTES ? 0 : sb_rd + SBYTES;
        sb_wr = sb_wr == 2 * SBYTES ? 0 : sb_wr + SBYTES;
    }
    WAIT_V(0);
    BAR;
    unsigned bad = 0;
    if (tl.fA != nullptr || tl.fB != nullptr) {
        unsigned tv;
        asm volatile("global_load_dword %0, %1, off sc1\n\ts_waitcnt vmcnt(0)" : "=v"(tv) : "v"(tmo) : "memory");
        bad = __builtin_amdgcn_readfirstlane(tv) == call;
    }
    int tz = threadIdx.x; asm volatile("" : "+v"(tz));
    const int lane2 = tz & 63, fr2 = lane2 & 15, fq2 = lane2 >> 4;
#pragma unroll
    for (int m = 0; m < MF; ++m) { const int row = wr * (MF * 16) + m * 16 + fr2;
#pragma unroll
        for (int n = 0; n < NF; ++n) { const f32x4 v = acc[m][n];
            u32x2 w; w.x = cvt_pk_bf16(gelu_tanh(v[0]), gelu_tanh(v[1])); w.y = cvt_pk_bf16(gelu_tanh(v[2]), gelu_tanh(v[3]));
            if (bad) { w.x = 0x7fc07fc0u; w.y = 0x7fc07fc0u; }
            const int col = wc * (NF * 16) + n * 16 + fq2 * 4; const int ch = col >> 3, hf = (col >> 2) & 1;
            *(LAS u32x2*)(lds + row * 256 + ((ch ^ fr2) << 4) + hf * 8) = w; } }
    WAIT_L(0); BAR;
    constexpr int RPW = BM / 8;
    if (tl.fO != nullptr) {
#pragma unroll 4
        for (int i = 0; i < RPW / 4; ++i) { const int row = wid * RPW + i * 4 + (lane2 >> 4), chp = lane2 & 15;
            const u32x4 v = *(const LAS u32x4*)(lds + row * 256 + chp * 16);
            st16_sys(tl.O + (size_t)row * 2048 + ((chp ^ (row & 15)) << 4), v); }
        WAIT_V(0);
        BAR;
        if (thread0()) st_sys(tl.fO, call);
    } else {
#pragma unroll 4
        for (int i = 0; i < RPW / 4; ++i) { const int row = wid * RPW + i * 4 + (lane2 >> 4), chp = lane2 & 15;
            const u32x4 v = *(const LAS u32x4*)(lds + row * 256 + chp * 16);
            *(u32x4*)(tl.O + (size_t)row * 2048 + ((chp ^ (row & 15)) << 4)) = v; }
        WAIT_L(0);
        BAR;
    }
#undef STAGE_TILE
#undef RD128
#undef TRRD
}

__device__ __forceinline__ Tile make_tile(const Params& p, char* wsl, char* par, int rb, int ro, int cb, int oi) {
    const int r = p.rank; Tile t;
    int grow;
    if (rb < 4) { t.A = p.x + (size_t)(rb * 256 + ro) * XROW; t.fA = nullptr; grow = r * 1024 + rb * 256; }
    else {
        t.A = par + (size_t)((rb - 4) * 256 + ro) * XROW; t.fA = (const unsigned*)(wsl + WS_FA) + rb * NCH;
        if (rb < 8) grow = (r ^ 1) * 1024 + (rb - 4) * 256;
        else { const int j = (rb - 8) >> 1, xv = (j & 1) + 2 + (j >> 1) * 4; grow = (r ^ xv) * 1024 + (r >> 2) * 512 + (rb & 1) * 256; }
    }
    grow += ro;
    if (cb < 8) { t.B = p.w + cb * 256; t.ldb = 2048; t.fB = nullptr; t.O = p.out + (size_t)grow * 2048 + cb * 256; t.fO = nullptr; }
    else {
        const int j = (cb - 8) >> 2, cl = cb & 3;
        t.B = par + XR_BYTES + (size_t)j * (WR_BYTES / 2) + cl * 256; t.ldb = 1024;
        t.fB = (const unsigned*)(wsl + WS_FB) + (j * 2 + (cl >> 1)) * NCH;
        const int J = r ^ (4 + j);
        t.O = p.outs[J] + (size_t)grow * 2048 + j * 1024 + cl * 256;
        t.fO = (unsigned*)(p.ws[J] + WS_FO) + j * 128 + oi * 4 + cl;
    }
    return t;
}

struct Unit { const char* src; char* dst; unsigned sstride, dstride; int wide; unsigned* flag; const unsigned* dep; };

__device__ __forceinline__ bool make_unit(const Params& p, char* wsl, size_t paroff, int L, int j, int n, Unit& u) {
    const int r = p.rank;
    const int xv = L == 0 ? 1 : (L < 5 ? ((L - 1) & 1) + 2 + ((L - 1) >> 1) * 4 : L - 1);
    const int peer = r ^ xv;
    char* pw = p.ws[peer]; char* ppar = pw + paroff;
    u.dep = nullptr;
    if (n < 32) {
        const int un = j + 4 * n, c = un >> 1, b = un & 1;
        if (L < 5) {
            const int srow = L == 0 ? b * 512 : (peer >> 2) * 512 + b * 256;
            const int rb = L == 0 ? 4 + 2 * b : 8 + 2 * (L - 1) + b;
            u.src = p.x + (size_t)srow * XROW + c * 256; u.sstride = XROW;
            u.dst = ppar + (size_t)(rb - 4) * 256 * XROW + c * 256; u.dstride = XROW; u.wide = 0;
            u.flag = (unsigned*)(pw + WS_FA) + rb * NCH + c;
        } else {
            const int h = L - 5;
            u.src = p.w + (size_t)c * 128 * 2048 + h * 1024 + b * 512; u.sstride = 2048;
            u.dst = ppar + XR_BYTES + (size_t)h * (WR_BYTES / 2) + (size_t)c * 128 * 1024 + b * 512; u.dstride = 1024; u.wide = 1;
            u.flag = (unsigned*)(pw + WS_FB) + (h * 2 + b) * NCH + c;
        }
        return true;
    }
    if (L >= 5) return false;
    const int m = j + 4 * (n - 32);
    if (L == 0) {
        if (m >= 26) return false;
        const int c = (m >> 1) * 5, blk = 1 + 2 * (m & 1), rb = 4 + blk;
        u.src = p.x + (size_t)blk * 256 * XROW + c * 256;
        u.dst = ppar + (size_t)blk * 256 * XROW + c * 256;
        u.flag = (unsigned*)(pw + WS_FA) + rb * NCH + c;
    } else {
        const int c = (m >> 1) * 5 + 1 + (r & 1) + 2 * (m & 1);
        if (c >= NCH) return false;
        const int I = peer ^ 1, xi = r ^ I;
        const int jI = (xi & 1) + ((xi >> 2) & 1) * 2;
        const int rbl = 8 + 2 * jI + 1;
        const int blk = 2 * (r >> 2) + 1, rb = 4 + blk;
        u.src = wsl + paroff + (size_t)(rbl - 4) * 256 * XROW + c * 256;
        u.dep = (const unsigned*)(wsl + WS_FA) + rbl * NCH + c;
        u.dst = ppar + (size_t)blk * 256 * XROW + c * 256;
        u.flag = (unsigned*)(pw + WS_FA) + rb * NCH + c;
    }
    u.sstride = XROW; u.dstride = XROW; u.wide = 0;
    return true;
}

__device__ __forceinline__ void unit_load(const Unit& u, u32x4 (&v)[8], int tid) {
    const int sh = u.wide ? 5 : 4; const unsigned rpr = 512u >> sh;
    const char* s = u.src + (size_t)(tid >> sh) * u.sstride + (tid & ((1 << sh) - 1)) * 16;
    const size_t st = (size_t)rpr * u.sstride;
#pragma unroll
    for (int i = 0; i < 8; ++i) asm volatile("global_load_dwordx4 %0, %1, off" : "=v"(v[i]) : "v"(s + i * st) : "memory");
}
__device__ __forceinline__ void unit_store(const Unit& u, u32x4 (&v)[8], int tid) {
    const int sh = u.wide ? 5 : 4; const unsigned rpr = 512u >> sh;
    char* d = u.dst + (size_t)(tid >> sh) * u.dstride + (tid & ((1 << sh) - 1)) * 16;
    const size_t st = (size_t)rpr * u.dstride;
#pragma unroll
    for (int i = 0; i < 8; ++i) st16_sys(d + i * st, v[i]);
}
#define PIN8(v) asm volatile("" : "+v"(v[0]), "+v"(v[1]), "+v"(v[2]), "+v"(v[3]), "+v"(v[4]), "+v"(v[5]), "+v"(v[6]), "+v"(v[7]))

__device__ __forceinline__ void dep_wait(const Unit& u, unsigned call, unsigned* tmo, int wid, int lane) {
    if (u.dep != nullptr) wait_all(u.dep, lane == 0, call, tmo);
}

__device__ void copy_engine(const Params& p, char* wsl, size_t paroff, int cw, unsigned call, unsigned* tmo) {
    const int tid = threadIdx.x, wid = __builtin_amdgcn_readfirstlane(tid >> 6), lane = tid & 63;
    const int L = cw % 7, j = cw / 7;
    u32x4 va[8], vb[8];
    Unit ua, ub;
    unsigned* f1 = nullptr;
    unsigned* f2 = nullptr;
    make_unit(p, wsl, paroff, L, j, 0, ua);
    dep_wait(ua, call, tmo, wid, lane);
    unit_load(ua, va, tid);
    bool hb = make_unit(p, wsl, paroff, L, j, 1, ub);
    dep_wait(ub, call, tmo, wid, lane);
    unit_load(ub, vb, tid);
    WAIT_V(8);
    int n = 2;
    for (;;) {
        PIN8(va);
        if (f2 != nullptr) { BAR; if (thread0()) st_sys(f2, call); }
        unit_store(ua, va, tid);
        f2 = f1; f1 = ua.flag;
        if (!hb) break;
        const bool ha = make_unit(p, wsl, paroff, L, j, n++, ua);
        if (ha) { dep_wait(ua, call, tmo, wid, lane); unit_load(ua, va, tid); WAIT_V(16); }
        else WAIT_V(8);
        PIN8(vb);
        if (f2 != nullptr) { BAR; if (thread0()) st_sys(f2, call); }
        unit_store(ub, vb, tid);
        f2 = f1; f1 = ub.flag;
        if (!ha) break;
        hb = make_unit(p, wsl, paroff, L, j, n++, ub);
        if (hb) { dep_wait(ub, call, tmo, wid, lane); unit_load(ub, vb, tid); WAIT_V(16); }
        else WAIT_V(8);
    }
    WAIT_V(8);
    BAR;
    if (f2 != nullptr && thread0()) st_sys(f2, call);
    WAIT_V(0);
    BAR;
    if (thread0()) st_sys(f1, call);
}

__global__ __launch_bounds__(NTHREADS, 2) void ag_gemm_kernel(const Params p) {
    extern __shared__ __attribute__((aligned(16))) unsigned char shm[];
    LAS unsigned char* lds = (LAS unsigned char*)shm;
    char* wsl = p.ws[p.rank];
    unsigned* tmo = (unsigned*)(wsl + WS_TMO);
    const int bid = blockIdx.x, tid = threadIdx.x;
    unsigned call;
    {
        unsigned* ctr = (unsigned*)(wsl + WS_CTR + (size_t)bid * 64);
        unsigned v;
        asm volatile("global_load_dword %0, %1, off sc1\n\ts_waitcnt vmcnt(0)" : "=v"(v) : "v"(ctr) : "memory");
        call = __builtin_amdgcn_readfirstlane(v) + 1u;
        if (call == 0u) call = 1u;
        __syncthreads();
        if (thread0()) st_agent(ctr, call);
    }
    const size_t paroff = WS_DATA + (size_t)(call & 1u) * PAR_BYTES;
    char* par = wsl + paroff;
    if (bid < GEMM0) {
        if (bid < NCOPY) copy_engine(p, wsl, paroff, bid, call, tmo);
        else if (bid == NCOPY && tid < 64) {
            const unsigned* fo = (const unsigned*)(wsl + WS_FO);
            wait_all(fo + tid, true, call, tmo);
            wait_all(fo + 64 + tid, tid < 24, call, tmo);
            wait_all(fo + 128 + tid, true, call, tmo);
            wait_all(fo + 192 + tid, tid < 24, call, tmo);
            unsigned tv = ld_agent(tmo);
            if (tv == call && tid == 0) *(u16*)(p.out + (size_t)p.rank * 1024 * 2048) = 0x7fc0;
        }
        return;
    }
    {
        const int g = bid - GEMM0, xcd = g & 7, slot = g >> 3;
        const int g1 = (xcd >> 2) * 7 + slot % 7, cb = (xcd & 3) * 4 + slot / 7;
        const int rb = g1 < 5 ? g1 : (g1 == 5 ? 6 : g1 + 2);
        const Tile t = make_tile(p, wsl, par, rb, 0, cb, g1);
        gemm_tile<256>(lds, t, call, tmo);
    }
    unsigned* q = (unsigned*)(wsl + WS_Q);
    const unsigned qbase = (call - 1u) * (unsigned)(NG2 + NGEMM);
    for (;;) {
        LAS unsigned* sw = (LAS unsigned*)(lds + LDS_BYTES - 16);
        if (thread0()) *sw = __hip_atomic_fetch_add(q, 1u, __ATOMIC_RELAXED, __HIP_MEMORY_SCOPE_AGENT) - qbase;
        __syncthreads();
        const unsigned tk = __builtin_amdgcn_readfirstlane(*sw);
        __syncthreads();
        if (tk >= (unsigned)NG2) break;
        const int cb = (int)((tk >> 3) + 8) & 15, g2 = tk & 7;
        const Tile t = make_tile(p, wsl, par, 5 + 2 * (g2 >> 2), (g2 & 3) * 64, cb, 14 + g2);
        gemm_tile<64>(lds, t, call, tmo);
    }
}

}

extern "C" void kernel_launch(int rank, int nranks, ncclComm_t comm,
                              hipStream_t stream,
                              void* const* d_in, const int* in_sizes, int n_in,
                              void* d_out, int out_size,
                              const int* in_dtypes, int out_dtype,
                              void* const* d_out_all,
                              void* const* d_ws_all, size_t ws_bytes) {
    using namespace agk;
    if (nranks != NR || (unsigned)rank >= (unsigned)NR || ws_bytes < WS_NEED) { fprintf(stderr, "unsupported configuration\n"); return; }
    static bool attr_set[64] = {};
    int dev = 0; HIP_CHECK(hipGetDevice(&dev));
    if (dev >= 0 && dev < 64 && !attr_set[dev]) {
        HIP_CHECK(hipFuncSetAttribute((const void*)ag_gemm_kernel, hipFuncAttributeMaxDynamicSharedMemorySize, LDS_BYTES));
        attr_set[dev] = true;
    }
    Params p;
    p.x = (const char*)d_in[0]; p.w = (const char*)d_in[1]; p.out = (char*)d_out; p.rank = rank;
    for (int i = 0; i < NR; ++i) { p.ws[i] = (char*)d_ws_all[i]; p.outs[i] = (char*)d_out_all[i]; }
    hipLaunchKernelGGL(ag_gemm_kernel, dim3(NWG), dim3(NTHREADS), LDS_BYTES, stream, p);
    HIP_CHECK(hipGetLastError());
}
```

```cpp
#include <hip/hip_runtime.h>
#include <rccl/rccl.h>
#include <cstdio>

#define HIP_CHECK(expr) do { hipError_t _e = (expr); if (_e != hipSuccess) { \
    fprintf(stderr, "HIP error %s at %s:%d\n", hipGetErrorString(_e), __FILE__, __LINE__); \
    return; } } while (0)

namespace agk {

#define LAS __attribute__((address_space(3)))
typedef unsigned short u16;
typedef short s16x4 __attribute__((ext_vector_type(4)));
typedef short s16x8 __attribute__((ext_vector_type(8)));
typedef __bf16 bf16x8 __attribute__((ext_vector_type(8)));
typedef float f32x4 __attribute__((ext_vector_type(4)));
typedef unsigned u32x4 __attribute__((ext_vector_type(4)));
typedef unsigned u32x2 __attribute__((ext_vector_type(2)));

constexpr int NR = 8;
constexpr int KDIM = 8192;
constexpr unsigned XROW = KDIM * 2;
constexpr int NT = KDIM / 64;
constexpr int NCH = KDIM / 128;
constexpr int NTHREADS = 512;
constexpr int NWG = 256;
constexpr int NCOPY = 87;
#define AW 2
#define RPF 7
constexpr int NDIR1 = 56 - 4 * RPF;
constexpr int NCW = NCOPY * AW;
constexpr int FOR0 = 88;
constexpr int OWN0 = 200;
constexpr int NGEMM = NWG - FOR0;
constexpr int NG2 = 128;
constexpr int LDS_BYTES = 3 * (256 * 128 + 64 * 256);
constexpr unsigned long long DEADLINE = 50000000ull;

constexpr size_t WS_CTR = 0;
constexpr size_t WS_TMO = 0x4000;
constexpr size_t WS_Q = 0x4100;
constexpr size_t WS_FA = 0x10000;
constexpr size_t WS_FB = 0x18000;
constexpr size_t WS_FO = 0x1A000;
constexpr size_t WS_FS = 0x1B000;
constexpr size_t WS_FR = 0x1C000;
constexpr size_t WS_DBG = 0x20000;
constexpr size_t WS_DATA = 1u << 20;
constexpr size_t UNIT = 65536;
constexpr size_t XR_BYTES = (size_t)12 * NCH * UNIT;
constexpr size_t WR_BYTES = (size_t)4 * NCH * UNIT;
constexpr size_t PAR_BYTES = XR_BYTES + WR_BYTES;
constexpr size_t WS_STG = WS_DATA + 2 * PAR_BYTES;
constexpr size_t STG_BYTES = (size_t)128 * UNIT;
constexpr size_t WS_RB = WS_STG + STG_BYTES;
constexpr size_t RB_BYTES = (size_t)8 * 14 * UNIT;
constexpr size_t WS_NEED = WS_RB + RB_BYTES;
static_assert(RPF >= 0 && RPF <= 14, "relay count");

struct Params {
    const char* x; const char* w; char* out;
    char* ws[NR]; char* outs[NR];
    int rank;
};

__device__ __forceinline__ void st_sys(unsigned* p, unsigned v) { __hip_atomic_store(p, v, __ATOMIC_RELAXED, __HIP_MEMORY_SCOPE_SYSTEM); }
__device__ __forceinline__ unsigned ld_agent(const unsigned* p) { return __hip_atomic_load(p, __ATOMIC_RELAXED, __HIP_MEMORY_SCOPE_AGENT); }
__device__ __forceinline__ void st_agent(unsigned* p, unsigned v) { __hip_atomic_store(p, v, __ATOMIC_RELAXED, __HIP_MEMORY_SCOPE_AGENT); }
__device__ __forceinline__ bool geq(unsigned v, unsigned call) { return (int)(v - call) >= 0; }
__device__ __forceinline__ int opaque_tid() { int t = threadIdx.x; asm volatile("" : "+v"(t)); return t; }
__device__ __forceinline__ bool thread0() { return opaque_tid() == 0; }
__device__ __forceinline__ bool lane0() { return (opaque_tid() & 63) == 0; }
#define WAIT_V(n) asm volatile("s_waitcnt vmcnt(" #n ")" ::: "memory")
#define WAIT_L(n) asm volatile("s_waitcnt lgkmcnt(" #n ")" ::: "memory")
#define BAR __builtin_amdgcn_s_barrier()
#define SCHED __builtin_amdgcn_sched_barrier(0)
#define STAMP(wsl, i) do { } while (0)

__device__ __forceinline__ void st16_sys(void* p, u32x4 v) {
    asm volatile("global_store_dwordx4 %0, %1, off sc0 sc1\n\ts_nop 1" :: "v"(p), "v"(v) : "memory");
}
__device__ __forceinline__ unsigned long long poll_once(const unsigned* p, bool need, unsigned call) {
    unsigned v;
    asm volatile("global_load_dword %0, %1, off sc0 sc1\n\ts_waitcnt vmcnt(0)" : "=v"(v) : "v"(p) : "memory");
    return __ballot(!need || geq(v, call));
}
__device__ __forceinline__ bool timed_out(const unsigned* tmo, unsigned call) {
    unsigned tv;
    asm volatile("global_load_dword %0, %1, off sc1\n\ts_waitcnt vmcnt(0)" : "=v"(tv) : "v"(tmo) : "memory");
    return __builtin_amdgcn_readfirstlane(tv) == call;
}
__device__ __forceinline__ bool spin_check(unsigned* tmo, unsigned call, unsigned long long t0) {
    if (timed_out(tmo, call)) return true;
    if (__builtin_amdgcn_s_memrealtime() - t0 > DEADLINE) { st_agent(tmo, call); return true; }
    __builtin_amdgcn_s_sleep(2);
    return false;
}
__device__ __forceinline__ void wait_all(const unsigned* p, bool need, unsigned call, unsigned* tmo) {
    if (poll_once(p, need, call) == ~0ull) return;
    const unsigned long long t0 = __builtin_amdgcn_s_memrealtime();
    for (;;) {
        if (spin_check(tmo, call, t0)) return;
        if (poll_once(p, need, call) == ~0ull) return;
    }
}

__host__ __device__ __forceinline__ int lds_byte(int r, int c) { const int st = (r >> 4) * 2 + (c >> 5), rr = r & 15, cc = c & 31, ob = rr * 64 + cc * 2; return st * 1024 + (ob ^ (((ob >> 9) & 1) << 5)); }
__host__ __device__ __forceinline__ void stage_rc(int b, int& R, int& C) { const int st = b / 1024, sb = b % 1024, swz = sb ^ (((sb >> 9) & 1) << 5); R = (st >> 1) * 16 + swz / 64; C = (st & 1) * 32 + (swz % 64) / 2; }
__device__ __forceinline__ unsigned cvt_pk_bf16(float lo, float hi) { unsigned r; asm volatile("v_cvt_pk_bf16_f32 %0, %1, %2" : "=v"(r) : "v"(lo), "v"(hi)); return r; }
__device__ __forceinline__ float gelu_tanh(float y) {
    const float k1 = -2.0f * 0.7978845608028654f * 1.4426950408889634f;
    const float k2 = k1 * 0.044715f;
    const float t = y * (k2 * (y * y) + k1);
    return y * __builtin_amdgcn_rcpf(1.0f + __builtin_amdgcn_exp2f(t));
}
__device__ __forceinline__ u32x2 gelu_pack(f32x4 v, unsigned bad) {
    u32x2 w; w.x = cvt_pk_bf16(gelu_tanh(v[0]), gelu_tanh(v[1])); w.y = cvt_pk_bf16(gelu_tanh(v[2]), gelu_tanh(v[3]));
    if (bad) { w.x = 0x7fc07fc0u; w.y = 0x7fc07fc0u; }
    return w;
}

struct Tile {
    const char* A; const char* B; char* O;
    unsigned lda, csA;
    unsigned ldb;
    const unsigned* fA; const unsigned* fB;
    unsigned* fO; int nfO;
};

__device__ __forceinline__ unsigned poll_chunks(const unsigned* p, bool need, unsigned call) {
    u32x4 v;
    asm volatile("global_load_dwordx4 %0, %1, off sc0 sc1\n\ts_waitcnt vmcnt(0)" : "=v"(v) : "v"(p) : "memory");
    const unsigned long long mk = __ballot(!need || (geq(v.x, call) && geq(v.y, call) && geq(v.z, call) && geq(v.w, call)));
    const unsigned both = (unsigned)mk & (unsigned)(mk >> 32);
    return both & (both >> 1) & 0x55555555u;
}
__device__ __forceinline__ void ensure(const Tile& tl, int c, int& ready, unsigned call, unsigned* tmo, int lane) {
    if (c >= ready) {
        const int cc = c + ((lane >> 1) & 15);
        const unsigned* base = lane < 32 ? tl.fA : tl.fB;
        const bool need = base != nullptr && cc < NCH;
        const unsigned* p = need ? base + cc * 8 + (lane & 1) * 4 : (const unsigned*)((const char*)tmo - WS_TMO + WS_FA);
        unsigned m = poll_chunks(p, need, call);
        if (!(m & 1u)) {
            const unsigned long long t0 = __builtin_amdgcn_s_memrealtime();
            for (;;) {
                if (spin_check(tmo, call, t0)) { m = 0x55555555u; break; }
                m = poll_chunks(p, need, call);
                if (m & 1u) break;
            }
        }
        const unsigned miss = ~m & 0x55555555u;
        ready = c + (miss ? (__builtin_ctz(miss) >> 1) : 16);
    }
}
#define RD128(dst, addr, off) asm volatile("ds_read_b128 %0, %1 offset:%2" : "=v"(dst) : "v"(addr), "i"(off) : "memory")
#define TRRD(dst, addr, off) asm volatile("ds_read_b64_tr_b16 %0, %1 offset:%2" : "=v"(dst) : "v"(addr), "i"(off) : "memory")
#define GLDS(g, l) __builtin_amdgcn_global_load_lds((const unsigned*)(g), (LAS unsigned*)(l), 16, 0, 0)

template <int BM, int KT>
__device__ __forceinline__ void gemm_tile(LAS unsigned char* lds, const Tile& tl, const unsigned call, unsigned* tmo, int sbase) {
    char* const wsl_dbg = (char*)tmo - WS_TMO; (void)wsl_dbg; (void)sbase;
    constexpr int WAVES_M = BM == 256 ? 4 : 2, WAVES_N = 8 / WAVES_M;
    constexpr int MF = BM / WAVES_M / 16, NF = 128 / WAVES_N / 16;
    constexpr int ABYTES = BM * 128, TBYTES = ABYTES + 16384, SBYTES = KT * TBYTES;
    constexpr int RA = ABYTES / 8192;
    constexpr int NS = NT / KT;
    static_assert(3 * SBYTES <= LDS_BYTES, "LDS");
    const int tid = threadIdx.x, wid = __builtin_amdgcn_readfirstlane(tid >> 6), lane = tid & 63;
    const int wr = wid / WAVES_N, wc = wid % WAVES_N, fr = lane & 15, fq = lane >> 4;
    unsigned voffA[RA], voffB[2];
#pragma unroll
    for (int i = 0; i < RA; ++i) { int R, C; stage_rc(tid * 16 + i * 8192, R, C); voffA[i] = (unsigned)R * tl.lda + (unsigned)C * 2u; }
#pragma unroll
    for (int i = 0; i < 2; ++i) { const int b = tid * 16 + i * 8192, row = b >> 8, chpos = (b >> 4) & 15, ch = chpos ^ (((row & 3) << 2) | ((row >> 2) & 3));
        voffB[i] = (unsigned)row * tl.ldb + (unsigned)ch * 16u; }
    const size_t kstepB = (size_t)64 * tl.ldb;
    const unsigned ldsbase = (unsigned)(size_t)lds;
    const unsigned ldsw = (unsigned)wid * 1024u;
    const unsigned aoff = ldsbase + (unsigned)lds_byte(wr * (MF * 16) + fr, fq * 8);
    unsigned boff0;
    { const int g = lane >> 4, q = (lane & 15) >> 2, p = lane & 3; const int row = 8 * g + q, ch = 2 * (wc * NF) + (p >> 1);
      boff0 = (unsigned)(ABYTES + 256 * row + 16 * (ch ^ (((row & 3) << 2) | ((row >> 2) & 3))) + 8 * (p & 1)); }
    unsigned boff[NF][2];
#pragma unroll
    for (int n = 0; n < NF; ++n)
#pragma unroll
        for (int t = 0; t < 2; ++t) boff[n][t] = ldsbase + (boff0 ^ (unsigned)(n * 32 + t * 16)) + (unsigned)(t * 1024);

    f32x4 acc[MF][NF];
#pragma unroll
    for (int m = 0; m < MF; ++m)
#pragma unroll
        for (int n = 0; n < NF; ++n) acc[m][n] = (f32x4){0.f, 0.f, 0.f, 0.f};

    int ready = (tl.fA == nullptr && tl.fB == nullptr) ? NCH : 0;
#define STAGE_TILE(st, sb) do { _Pragma("unroll") for (int _j = 0; _j < KT; ++_j) { const int _kt = (st) * KT + _j; \
        const char* _a = tl.A + (size_t)(_kt >> 1) * tl.csA + (_kt & 1) * 128; const char* _b = tl.B + (size_t)_kt * kstepB; \
        _Pragma("unroll") for (int _i = 0; _i < RA; ++_i) GLDS(_a + voffA[_i], lds + (sb) + _j * TBYTES + ldsw + _i * 8192); \
        _Pragma("unroll") for (int _i = 0; _i < 2; ++_i) GLDS(_b + voffB[_i], lds + (sb) + _j * TBYTES + ABYTES + ldsw + _i * 8192); } } while (0)

    if (wid == 0) ensure(tl, 0, ready, call, tmo, lane);
    BAR;
    STAMP(wsl_dbg, sbase);
    STAGE_TILE(0, 0);
    if (KT == 2 && wid == 0) ensure(tl, 1, ready, call, tmo, lane);
    if (KT == 2) BAR;
    STAGE_TILE(1, SBYTES);
    unsigned sb_rd = 0, sb_wr = 2 * SBYTES;
    for (int t = 0; t < NS; ++t) {
        const int tn = t + 2 < NS ? t + 2 : NS - 1;
        if (wid == 0 && (KT == 2 || (t & 1) == 0)) ensure(tl, KT == 2 ? tn : tn >> 1, ready, call, tmo, lane);
        WAIT_V(6);
        static_assert((RA + 2) * KT == 6, "vmcnt");
        BAR;
        STAGE_TILE(tn, sb_wr);
        s16x8 Af[2 * KT][MF]; s16x4 Bl[2 * KT][NF], Bh[2 * KT][NF];
        const unsigned a = aoff + sb_rd;
#pragma unroll
        for (int k = 0; k < 2 * KT; ++k) {
#pragma unroll
            for (int m = 0; m < MF; ++m) RD128(Af[k][m], a, (k >> 1) * TBYTES + m * 2048 + (k & 1) * 1024);
#pragma unroll
            for (int n = 0; n < NF; ++n) { TRRD(Bl[k][n], boff[n][0] + sb_rd, (k >> 1) * TBYTES + (k & 1) * 8192); TRRD(Bh[k][n], boff[n][1] + sb_rd, (k >> 1) * TBYTES + (k & 1) * 8192); }
        }
#pragma unroll
        for (int k = 0; k < 2 * KT; ++k) {
            if constexpr (BM == 256) { if (k == 0) WAIT_L(12); else WAIT_L(0); }
            else { if (k == 0) WAIT_L(15); else if (k == 1) WAIT_L(12); else if (k == 2) WAIT_L(6); else WAIT_L(0); }
            SCHED;
            __builtin_amdgcn_s_setprio(1);
#pragma unroll
            for (int m = 0; m < MF; ++m)
#pragma unroll
                for (int n = 0; n < NF; ++n) {
                    const s16x8 bf = __builtin_shufflevector(Bl[k][n], Bh[k][n], 0, 1, 2, 3, 4, 5, 6, 7);
                    acc[m][n] = __builtin_amdgcn_mfma_f32_16x16x32_bf16(__builtin_bit_cast(bf16x8, bf), __builtin_bit_cast(bf16x8, Af[k][m]), acc[m][n], 0, 0, 0);
                }
            __builtin_amdgcn_s_setprio(0);
            SCHED;
        }
        sb_rd = sb_rd == 2 * SBYTES ? 0 : sb_rd + SBYTES;
        sb_wr = sb_wr == 2 * SBYTES ? 0 : sb_wr + SBYTES;
    }
#undef STAGE_TILE
    WAIT_V(0);
    BAR;
    STAMP(wsl_dbg, sbase + 1);
    unsigned bad = 0;
    if (tl.fA != nullptr || tl.fB != nullptr) bad = timed_out(tmo, call);
    const int tz = opaque_tid();
    const int lane2 = tz & 63, fr2 = lane2 & 15, fq2 = lane2 >> 4;
#pragma unroll
    for (int m = 0; m < MF; ++m) { const int row = wr * (MF * 16) + m * 16 + fr2;
#pragma unroll
        for (int n = 0; n < NF; ++n) {
            const int col = wc * (NF * 16) + n * 16 + fq2 * 4; const int ch = col >> 3, hf = (col >> 2) & 1;
            *(LAS u32x2*)(lds + row * 256 + ((ch ^ fr2) << 4) + hf * 8) = gelu_pack(acc[m][n], bad); } }
    WAIT_L(0); BAR;
    constexpr int RPW = BM / 8;
    if (tl.fO != nullptr) {
#pragma unroll 4
        for (int i = 0; i < RPW / 4; ++i) { const int row = wid * RPW + i * 4 + (lane2 >> 4), chp = lane2 & 15;
            const u32x4 v = *(const LAS u32x4*)(lds + row * 256 + chp * 16);
            st16_sys(tl.O + (size_t)row * 256 + ((chp ^ (row & 15)) << 4), v); }
        WAIT_V(0);
        BAR;
        const int tf = opaque_tid();
        if (tf < tl.nfO) st_sys(tl.fO + tf, call);
    } else {
#pragma unroll 4
        for (int i = 0; i < RPW / 4; ++i) { const int row = wid * RPW + i * 4 + (lane2 >> 4), chp = lane2 & 15;
            const u32x4 v = *(const LAS u32x4*)(lds + row * 256 + chp * 16);
            *(u32x4*)(tl.O + (size_t)row * 2048 + ((chp ^ (row & 15)) << 4)) = v; }
        WAIT_L(0);
        BAR;
    }
    STAMP(wsl_dbg, sbase + 2);
}

__device__ __forceinline__ void gemm_tile_256(LAS unsigned char* lds, const Tile& tl, const unsigned call, unsigned* tmo) {
    char* const wsl_dbg = (char*)tmo - WS_TMO; (void)wsl_dbg;
    constexpr int HALF = 128, HTB = HALF * 64 * 2;
    const int tid = threadIdx.x, wid = __builtin_amdgcn_readfirstlane(tid >> 6), lane = tid & 63, wr = wid >> 2, wc = wid & 3, fr = lane & 15, fq = lane >> 4;
    unsigned voffA[2], voffB[2];
#pragma unroll
    for (int i = 0; i < 2; ++i) { int R, C; stage_rc(tid * 16 + i * 8192, R, C); voffA[i] = (unsigned)R * tl.lda + (unsigned)C * 2u;
        const int b = tid * 16 + i * 8192, row = b >> 8, chpos = (b >> 4) & 15, ch = chpos ^ (((row & 3) << 2) | ((row >> 2) & 3));
        voffB[i] = (unsigned)row * tl.ldb + (unsigned)ch * 16u; }
    const size_t kstepB = (size_t)64 * tl.ldb;
    const size_t hstepA = (size_t)HALF * tl.lda, hstepB = 256;
    const unsigned ldsw = (unsigned)wid * 1024u;
    const unsigned ldsbase = (unsigned)(size_t)lds;
    const unsigned aoff = ldsbase + (unsigned)lds_byte(wr * 64 + fr, fq * 8);
    unsigned boff[2][2];
    { const int g = lane >> 4, q = (lane & 15) >> 2, p = lane & 3;
#pragma unroll
      for (int n = 0; n < 2; ++n)
#pragma unroll
        for (int t = 0; t < 2; ++t) { const int row = 8 * g + 4 * t + q, ch = 4 * wc + 2 * n + (p >> 1);
            boff[n][t] = ldsbase + 4 * HTB + 256 * row + 16 * (ch ^ (((row & 3) << 2) | ((row >> 2) & 3))) + 8 * (p & 1); } }
#define SA(b, h) (((b) * 2 + (h)) * HTB)
#define SB(b, h) ((4 + (b) * 2 + (h)) * HTB)
#define STAGE(bufoff, gbase, voff) do { _Pragma("unroll") for (int _i = 0; _i < 2; ++_i) GLDS((const char*)(gbase) + (voff)[_i], lds + (bufoff) + ldsw + _i * 8192); } while (0)
#define LDA(dst, b, h) do { _Pragma("unroll") for (int m = 0; m < 4; ++m) _Pragma("unroll") for (int k = 0; k < 2; ++k) RD128(dst[m][k], aoff, SA(b, h) + m * 2048 + k * 1024); } while (0)
#define LDB(dst, b, h) do { _Pragma("unroll") for (int n = 0; n < 2; ++n) _Pragma("unroll") for (int k = 0; k < 2; ++k) { \
        s16x4 lo, hi; TRRD(lo, boff[n][0], ((b) * 2 + (h)) * HTB + k * 8192); TRRD(hi, boff[n][1], ((b) * 2 + (h)) * HTB + k * 8192); \
        dst[n][k] = __builtin_shufflevector(lo, hi, 0, 1, 2, 3, 4, 5, 6, 7); } } while (0)
#define MMA(ai, bj, At, Bt) do { __builtin_amdgcn_s_setprio(1); _Pragma("unroll") for (int m = 0; m < 4; ++m) _Pragma("unroll") for (int n = 0; n < 2; ++n) _Pragma("unroll") for (int k = 0; k < 2; ++k) \
        acc[ai][bj][m][n] = __builtin_amdgcn_mfma_f32_16x16x32_bf16(__builtin_bit_cast(bf16x8, Bt[n][k]), __builtin_bit_cast(bf16x8, At[m][k]), acc[ai][bj][m][n], 0, 0, 0); __builtin_amdgcn_s_setprio(0); } while (0)
#define KOFFA(t) ((size_t)((t) >> 1) * tl.csA + ((t) & 1) * 128)
    f32x4 acc[2][2][4][2];
#pragma unroll
    for (int a = 0; a < 2; ++a)
#pragma unroll
        for (int b = 0; b < 2; ++b)
#pragma unroll
            for (int m = 0; m < 4; ++m)
#pragma unroll
                for (int n = 0; n < 2; ++n) acc[a][b][m][n] = (f32x4){0.f, 0.f, 0.f, 0.f};
    s16x8 At[4][2], B0[2][2], B1[2][2];
    const char* cA = tl.A; const char* cB = tl.B;
    int ready = tl.fA == nullptr ? NCH : 0;
    if (wid == 0) ensure(tl, 0, ready, call, tmo, lane);
    BAR;
    STAMP(wsl_dbg, 1);
    STAGE(SB(0, 0), cB, voffB); STAGE(SA(0, 0), cA, voffA); STAGE(SB(0, 1), cB + hstepB, voffB); STAGE(SA(0, 1), cA + hstepA, voffA);
    if (wr == 1) BAR;
    WAIT_V(4); BAR;
    STAGE(SB(1, 0), cB + kstepB, voffB); STAGE(SA(1, 0), cA + 128, voffA); STAGE(SB(1, 1), cB + hstepB + kstepB, voffB);
    WAIT_V(6); BAR;
    for (int t = 0; t < NT; t += 2) {
        const int t2 = (t == NT - 2) ? 0 : t + 2;
        if (wid == 0) ensure(tl, t2 >> 1, ready, call, tmo, lane);
        const char* a1 = cA + KOFFA(t + 1);
        const char* a2 = cA + KOFFA(t2); const char* b2 = cB + (size_t)t2 * kstepB;
        const char* a3 = a2 + 128; const char* b3 = b2 + kstepB;
        LDB(B0, 0, 0); SCHED; LDA(At, 0, 0); STAGE(SA(1, 1), a1 + hstepA, voffA);
        WAIT_L(8); BAR; WAIT_L(0); SCHED; MMA(0, 0, At, B0); BAR; SCHED;
        LDB(B1, 0, 1); STAGE(SB(0, 0), b2, voffB);
        BAR; WAIT_L(0); SCHED; MMA(0, 1, At, B1); BAR; SCHED;
        LDA(At, 0, 1); STAGE(SA(0, 0), a2, voffA);
        BAR; WAIT_L(0); SCHED; MMA(1, 0, At, B0); BAR; SCHED;
        STAGE(SB(0, 1), b2 + hstepB, voffB);
        WAIT_V(6); BAR; MMA(1, 1, At, B1); BAR; SCHED;
        LDB(B0, 1, 0); SCHED; LDA(At, 1, 0); STAGE(SA(0, 1), a2 + hstepA, voffA);
        WAIT_L(8); BAR; WAIT_L(0); SCHED; MMA(0, 0, At, B0); BAR; SCHED;
        LDB(B1, 1, 1); STAGE(SB(1, 0), b3, voffB);
        BAR; WAIT_L(0); SCHED; MMA(0, 1, At, B1); BAR; SCHED;
        LDA(At, 1, 1); STAGE(SA(1, 0), a3, voffA);
        BAR; WAIT_L(0); SCHED; MMA(1, 0, At, B0); BAR; SCHED;
        STAGE(SB(1, 1), b3 + hstepB, voffB);
        WAIT_V(6); BAR; MMA(1, 1, At, B1); BAR; SCHED;
    }
    WAIT_V(0);
    if (wr == 0) BAR;
    BAR;
    STAMP(wsl_dbg, 2);
    unsigned bad = 0;
    if (tl.fA != nullptr) bad = timed_out(tmo, call);
    const int tz = opaque_tid();
    const int lane2 = tz & 63, fr2 = lane2 & 15, fq2 = lane2 >> 4;
#pragma unroll
    for (int ai = 0; ai < 2; ++ai)
#pragma unroll
        for (int m = 0; m < 4; ++m) { const int row = ai * HALF + wr * 64 + m * 16 + fr2;
#pragma unroll
            for (int bj = 0; bj < 2; ++bj)
#pragma unroll
                for (int n = 0; n < 2; ++n) {
                    const int col = bj * HALF + wc * 32 + n * 16 + fq2 * 4; const int ch = col >> 3, hf = (col >> 2) & 1;
                    *(LAS u32x2*)(lds + row * 512 + ((ch ^ fr2) << 4) + hf * 8) = gelu_pack(acc[ai][bj][m][n], bad); } }
    WAIT_L(0); BAR;
#pragma unroll 4
    for (int i = 0; i < 16; ++i) { const int row = wid * 32 + i * 2 + (lane2 >> 5), chp = lane2 & 31;
        const u32x4 v = *(const LAS u32x4*)(lds + row * 512 + chp * 16);
        *(u32x4*)(tl.O + (size_t)row * 2048 + ((chp ^ (row & 15)) << 4)) = v; }
    WAIT_L(0);
    BAR;
    STAMP(wsl_dbg, 3);
#undef SA
#undef SB
#undef STAGE
#undef LDA
#undef LDB
#undef MMA
#undef KOFFA
}

__device__ __forceinline__ int block_row(int r, int rb) {
    if (rb < 4) return r * 1024 + rb * 256;
    if (rb < 8) return (r ^ 1) * 1024 + (rb - 4) * 256;
    const int j = (rb - 8) >> 1, xv = (j & 1) + 2 + (j >> 1) * 4;
    return (r ^ xv) * 1024 + (r >> 2) * 512 + (rb & 1) * 256;
}
__device__ __forceinline__ int g1_block(int g1) { return g1 < 5 ? g1 : (g1 == 5 ? 6 : g1 + 2); }
__device__ __forceinline__ Tile make_tile(const Params& p, char* wsl, char* par, int rb, int sub, int cb, int ou) {
    Tile t; const int ro = sub < 0 ? 0 : sub * 64;
    if (rb < 4) { t.A = p.x + (size_t)(rb * 256 + ro) * XROW; t.lda = XROW; t.csA = 256; t.fA = nullptr; }
    else { t.A = par + (size_t)(rb - 4) * NCH * UNIT + ro * 256; t.lda = 256; t.csA = (unsigned)UNIT; t.fA = (const unsigned*)(wsl + WS_FA) + rb * NCH * 8; }
    if (cb < 8) {
        t.B = p.w + cb * 256; t.ldb = 2048; t.fB = nullptr;
        t.O = p.out + (size_t)(block_row(p.rank, rb) + ro) * 2048 + cb * 256; t.fO = nullptr; t.nfO = 0;
    } else {
        const int j = (cb - 8) >> 2, cl = cb & 3;
        const int cp = j * 2 + (cl >> 1);
        t.B = par + XR_BYTES + (size_t)cp * NCH * UNIT + (cl & 1) * 256; t.ldb = 512;
        t.fB = (const unsigned*)(wsl + WS_FB) + cp * NCH * 8;
        const int uo = j * 64 + ou * 4 + cl;
        t.O = wsl + WS_STG + (size_t)uo * UNIT + ro * 256;
        t.fO = (unsigned*)(wsl + WS_FS) + uo * 4 + (sub < 0 ? 0 : sub); t.nfO = sub < 0 ? 4 : 1;
    }
    return t;
}

struct Slice { const char* src; char* dst; unsigned sstride, dstride; int sh; unsigned* flag; const unsigned* dep; };

__device__ __forceinline__ int link_late(int r, int L) { return L == 0 ? 26 : 26 - (r & 1); }
__device__ __forceinline__ int link_total(int r, int L) { return 8 * (L >= 5 ? 128 + NDIR1 + 8 : 128 + link_late(r, L) + (L == 0 ? 0 : 4 * RPF)); }
__device__ __forceinline__ int link_xor(int L) { return L == 0 ? 1 : (L < 5 ? ((L - 1) & 1) + 2 + ((L - 1) >> 1) * 4 : L - 1); }
__device__ __forceinline__ int xlink_index(int xv) { return (xv & 1) + ((xv >> 2) & 1) * 2; }
__device__ __forceinline__ void make_slice(const Params& p, char* wsl, size_t paroff, int L, int tk, Slice& u) {
    const int r = p.rank;
    const int xv = link_xor(L);
    const int peer = r ^ xv;
    char* pw = p.ws[peer]; char* ppar = pw + paroff;
    const int un = tk >> 3, sl = tk & 7;
    u.dep = nullptr; u.sh = 4;
    if (un < 128) {
        const int c = un >> 1, b = un & 1;
        if (L < 5) {
            const int srow = L == 0 ? b * 512 : (peer >> 2) * 512 + b * 256;
            const int rb = L == 0 ? 4 + 2 * b : 8 + 2 * (L - 1) + b;
            u.src = p.x + (size_t)(srow + sl * 32) * XROW + c * 256; u.sstride = XROW;
            u.dst = ppar + ((size_t)(rb - 4) * NCH + c) * UNIT + sl * 8192; u.dstride = 256;
            u.flag = (unsigned*)(pw + WS_FA) + (rb * NCH + c) * 8 + sl;
        } else {
            const int h = L - 5, cp = h * 2 + b;
            u.src = p.w + (size_t)(c * 128 + sl * 16) * 2048 + h * 1024 + b * 512; u.sstride = 2048;
            u.dst = ppar + XR_BYTES + ((size_t)cp * NCH + c) * UNIT + sl * 8192; u.dstride = 512; u.sh = 5;
            u.flag = (unsigned*)(pw + WS_FB) + (cp * NCH + c) * 8 + sl;
        }
        return;
    }
    int m = un - 128;
    if (L >= 5) {
        const int h = L - 5, g = m < NDIR1 ? m : m + 4 * RPF, ou = g >> 2, cl = g & 3, uo = h * 64 + g;
        const int rb = ou < 14 ? g1_block(ou) : 5 + 2 * (ou - 14);
        u.src = wsl + WS_STG + (size_t)uo * UNIT + sl * 8192; u.sstride = 256;
        u.dst = p.outs[peer] + (size_t)(block_row(r, rb) + sl * 32) * 2048 + h * 1024 + cl * 256; u.dstride = 2048;
        u.dep = (const unsigned*)(wsl + WS_FS) + uo * 4 + (sl >> 1);
        u.flag = (unsigned*)(pw + WS_FO) + uo * 8 + sl;
        return;
    }
    const int nlate = link_late(r, L);
    if (m < nlate) {
        if (L == 0) {
            const int c = (m >> 1) * 5, blk = 1 + 2 * (m & 1), rb = 4 + blk;
            u.src = p.x + (size_t)(blk * 256 + sl * 32) * XROW + c * 256; u.sstride = XROW;
            u.dst = ppar + ((size_t)blk * NCH + c) * UNIT + sl * 8192; u.dstride = 256;
            u.flag = (unsigned*)(pw + WS_FA) + (rb * NCH + c) * 8 + sl;
        } else {
            const int c = (m >> 1) * 5 + 1 + (r & 1) + 2 * (m & 1);
            const int I = peer ^ 1;
            const int rbl = 8 + 2 * xlink_index(r ^ I) + 1;
            const int blk = 2 * (r >> 2) + 1, rb = 4 + blk;
            u.src = wsl + paroff + ((size_t)(rbl - 4) * NCH + c) * UNIT + sl * 8192; u.sstride = 256;
            u.dep = (const unsigned*)(wsl + WS_FA) + (rbl * NCH + c) * 8 + sl;
            u.dst = ppar + ((size_t)blk * NCH + c) * UNIT + sl * 8192; u.dstride = 256;
            u.flag = (unsigned*)(pw + WS_FA) + (rb * NCH + c) * 8 + sl;
        }
        return;
    }
    m -= nlate;
    const int h = m & 1;
    if (m < 2 * RPF) {
        const int k = m >> 1, g = NDIR1 + k * 4 + (L - 1), uo = h * 64 + g, slot = ((L - 1) * 2 + h) * RPF + k;
        u.src = wsl + WS_STG + (size_t)uo * UNIT + sl * 8192; u.sstride = 256;
        u.dep = (const unsigned*)(wsl + WS_FS) + uo * 4 + (sl >> 1);
        u.dst = pw + WS_RB + (size_t)slot * UNIT + sl * 8192; u.dstride = 256;
        u.flag = (unsigned*)(pw + WS_FR) + slot * 8 + sl;
    } else {
        const int k = (m - 2 * RPF) >> 1, rs = peer ^ (4 + h), jx = xlink_index(rs ^ r);
        const int g = NDIR1 + k * 4 + jx, ou = g >> 2, cl = g & 3, slot = (jx * 2 + h) * RPF + k;
        u.src = wsl + WS_RB + (size_t)slot * UNIT + sl * 8192; u.sstride = 256;
        u.dep = (const unsigned*)(wsl + WS_FR) + slot * 8 + sl;
        u.dst = p.outs[peer] + (size_t)(block_row(rs, g1_block(ou)) + sl * 32) * 2048 + h * 1024 + cl * 256; u.dstride = 2048;
        u.flag = (unsigned*)(pw + WS_FO) + (h * 64 + g) * 8 + sl;
    }
}
__device__ __forceinline__ void slice_load(const Slice& u, u32x4 (&v)[8], int lane) {
    const char* s = u.src + (size_t)(lane >> u.sh) * u.sstride + (lane & ((1 << u.sh) - 1)) * 16;
    const size_t st = (size_t)(64 >> u.sh) * u.sstride;
    if (u.dep != nullptr) {
#pragma unroll
        for (int i = 0; i < 8; ++i) asm volatile("global_load_dwordx4 %0, %1, off sc0 sc1" : "=v"(v[i]) : "v"(s + i * st) : "memory");
    } else {
#pragma unroll
        for (int i = 0; i < 8; ++i) asm volatile("global_load_dwordx4 %0, %1, off" : "=v"(v[i]) : "v"(s + i * st) : "memory");
    }
}
__device__ __forceinline__ void slice_store(const Slice& u, u32x4 (&v)[8], int lane) {
    char* d = u.dst + (size_t)(lane >> u.sh) * u.dstride + (lane & ((1 << u.sh) - 1)) * 16;
    const size_t st = (size_t)(64 >> u.sh) * u.dstride;
#pragma unroll
    for (int i = 0; i < 8; ++i) st16_sys(d + i * st, v[i]);
}
#define PIN8(v) asm volatile("" : "+v"(v[0]), "+v"(v[1]), "+v"(v[2]), "+v"(v[3]), "+v"(v[4]), "+v"(v[5]), "+v"(v[6]), "+v"(v[7]))

__device__ __forceinline__ void copy_wave(const Params& p, char* wsl, size_t paroff, int gw, unsigned call, unsigned* tmo) {
    const int lane = threadIdx.x & 63;
    const int L = gw % 7, j = gw / 7;
    const int nw = (NCW - L + 6) / 7;
    const int total = link_total(p.rank, L);
    unsigned* ctr = (unsigned*)(wsl + WS_Q + 128 * (1 + L));
    const unsigned tbase = (call - 1u) * (unsigned)total - (unsigned)nw;
    u32x4 va[8], vb[8];
    Slice ua, ub;
    unsigned tkv = 0;
#define DRAW() do { if (lane0()) asm volatile("global_atomic_add %0, %1, %2, off sc0" : "=v"(tkv) : "v"(ctr), "v"(1u) : "memory"); } while (0)
    unsigned* fprev = nullptr;
    make_slice(p, wsl, paroff, L, j, ua);
    DRAW();
    slice_load(ua, va, lane);
#define STEP(cur, vcur, nxt, vnxt) { \
        WAIT_V(0); \
        PIN8(vcur); asm volatile("" : "+v"(tkv)); \
        if (fprev != nullptr && lane0()) st_sys(fprev, call); \
        const int tk = (int)((unsigned)__builtin_amdgcn_readfirstlane(tkv) - tbase); \
        const bool hn = tk < total; \
        unsigned dv = 0; bool chk = false; \
        if (hn) { make_slice(p, wsl, paroff, L, tk, nxt); chk = nxt.dep != nullptr; \
            if (chk) asm volatile("global_load_dword %0, %1, off sc0 sc1" : "=v"(dv) : "v"(nxt.dep) : "memory"); } \
        slice_store(cur, vcur, lane); \
        fprev = cur.flag; \
        if (!hn) break; \
        if (chk) { \
            WAIT_V(8); asm volatile("" : "+v"(dv)); \
            if (!geq((unsigned)__builtin_amdgcn_readfirstlane(dv), call)) { \
                WAIT_V(0); if (lane0()) st_sys(fprev, call); fprev = nullptr; \
                wait_all(nxt.dep, true, call, tmo); } } \
        DRAW(); \
        slice_load(nxt, vnxt, lane); \
        if (tk >= 1000 && tk < 1024) STAMP(wsl, 2); \
        if (L < 5 && tk >= (128 + 25) * 8 - 24 && tk < (128 + 25) * 8) STAMP(wsl, 4); \
        if (L >= 1 && L < 5 && tk >= (128 + 26 + 2 * RPF) * 8 - 24 && tk < (128 + 26 + 2 * RPF) * 8) STAMP(wsl, 5); \
        if (L >= 5 && tk >= (128 + NDIR1) * 8 - 24 && tk < (128 + NDIR1) * 8) STAMP(wsl, 5); }
    for (;;) {
        STEP(ua, va, ub, vb)
        STEP(ub, vb, ua, va)
    }
#undef STEP
#undef DRAW
    WAIT_V(0);
    if (lane0()) st_sys(fprev, call);
    STAMP(wsl, 3);
}

__global__ __launch_bounds__(NTHREADS, 2) void ag_gemm_kernel(const Params p) {
    extern __shared__ __attribute__((aligned(16))) unsigned char shm[];
    LAS unsigned char* lds = (LAS unsigned char*)shm;
    char* wsl = p.ws[p.rank];
    unsigned* tmo = (unsigned*)(wsl + WS_TMO);
    const int bid = blockIdx.x, tid = threadIdx.x;
    unsigned call;
    {
        unsigned* ctr = (unsigned*)(wsl + WS_CTR + (size_t)bid * 64);
        unsigned v;
        asm volatile("global_load_dword %0, %1, off sc1\n\ts_waitcnt vmcnt(0)" : "=v"(v) : "v"(ctr) : "memory");
        call = __builtin_amdgcn_readfirstlane(v) + 1u;
        if (call == 0u) call = 1u;
        __syncthreads();
        if (thread0()) st_agent(ctr, call);
    }
    const size_t paroff = WS_DATA + (size_t)(call & 1u) * PAR_BYTES;
    char* par = wsl + paroff;
    STAMP(wsl, 0);
    if (bid < FOR0) {
        const int wid = __builtin_amdgcn_readfirstlane(tid >> 6);
        if (bid < NCOPY) { if (wid < AW) copy_wave(p, wsl, paroff, bid * AW + wid, call, tmo); }
        else {
            const unsigned* fo = (const unsigned*)(wsl + WS_FO);
            wait_all(fo + tid, true, call, tmo);
            wait_all(fo + 512 + tid, true, call, tmo);
            if (timed_out(tmo, call) && thread0()) *(u16*)(p.out + (size_t)p.rank * 1024 * 2048) = 0x7fc0;
            STAMP(wsl, 1);
        }
        return;
    }
    if (bid < OWN0) {
        const int g = bid - FOR0, xcd = g & 7, slot = g >> 3;
        const int g1 = (xcd >> 2) * 7 + slot % 7, cb = 8 + (xcd & 3) * 2 + slot / 7;
        const Tile t = make_tile(p, wsl, par, g1_block(g1), -1, cb, g1);
        gemm_tile<256, 1>(lds, t, call, tmo, 1);
    } else {
        const int g = bid - OWN0, xcd = g & 7, slot = g >> 3;
        const int g1 = (xcd >> 2) * 7 + slot, cb = (xcd & 3) * 2;
        const Tile t = make_tile(p, wsl, par, g1_block(g1), -1, cb, g1);
        gemm_tile_256(lds, t, call, tmo);
    }
    const int xq = bid & 7;
    unsigned* q = (unsigned*)(wsl + WS_Q + 128 * (8 + xq));
    const unsigned qbase = (call - 1u) * (unsigned)(NG2 / 8 + NGEMM / 8);
    for (;;) {
        LAS unsigned* sw = (LAS unsigned*)(lds + LDS_BYTES - 16);
        if (thread0()) *sw = __hip_atomic_fetch_add(q, 1u, __ATOMIC_RELAXED, __HIP_MEMORY_SCOPE_AGENT) - qbase;
        __syncthreads();
        const unsigned tk = __builtin_amdgcn_readfirstlane(*sw);
        __syncthreads();
        if (tk >= (unsigned)(NG2 / 8)) break;
        const int cb = tk < 8 ? 8 + xq : xq, g2 = tk & 7;
        const Tile t = make_tile(p, wsl, par, 5 + 2 * (g2 >> 2), g2 & 3, cb, 14 + (g2 >> 2));
        gemm_tile<64, 2>(lds, t, call, tmo, 4);
    }
}

}

extern "C" void kernel_launch(int rank, int nranks, ncclComm_t comm,
                              hipStream_t stream,
                              void* const* d_in, const int* in_sizes, int n_in,
                              void* d_out, int out_size,
                              const int* in_dtypes, int out_dtype,
                              void* const* d_out_all,
                              void* const* d_ws_all, size_t ws_bytes) {
    using namespace agk;
    if (nranks != NR || (unsigned)rank >= (unsigned)NR || ws_bytes < WS_NEED) { fprintf(stderr, "unsupported configuration\n"); return; }
    static bool attr_set[64] = {};
    int dev = 0; HIP_CHECK(hipGetDevice(&dev));
    if (dev >= 0 && dev < 64 && !attr_set[dev]) {
        HIP_CHECK(hipFuncSetAttribute((const void*)ag_gemm_kernel, hipFuncAttributeMaxDynamicSharedMemorySize, LDS_BYTES));
        attr_set[dev] = true;
    }
    Params p;
    p.x = (const char*)d_in[0]; p.w = (const char*)d_in[1]; p.out = (char*)d_out; p.rank = rank;
    for (int i = 0; i < NR; ++i) { p.ws[i] = (char*)d_ws_all[i]; p.outs[i] = (char*)d_out_all[i]; }
    hipLaunchKernelGGL(ag_gemm_kernel, dim3(NWG), dim3(NTHREADS), LDS_BYTES, stream, p);
    HIP_CHECK(hipGetLastError());
}
```

```cpp
#include <hip/hip_runtime.h>
#include <rccl/rccl.h>
#include <cstdio>

#define HIP_CHECK(expr) do { hipError_t _e = (expr); if (_e != hipSuccess) { \
    fprintf(stderr, "HIP error %s at %s:%d\n", hipGetErrorString(_e), __FILE__, __LINE__); \
    return; } } while (0)

namespace agk {

#define LAS __attribute__((address_space(3)))
typedef unsigned short u16;
typedef short s16x4 __attribute__((ext_vector_type(4)));
typedef short s16x8 __attribute__((ext_vector_type(8)));
typedef __bf16 bf16x8 __attribute__((ext_vector_type(8)));
typedef float f32x4 __attribute__((ext_vector_type(4)));
typedef unsigned u32x4 __attribute__((ext_vector_type(4)));
typedef unsigned u32x2 __attribute__((ext_vector_type(2)));

constexpr int NR = 8;
constexpr int KDIM = 8192;
constexpr unsigned XROW = KDIM * 2;
constexpr int NT = KDIM / 64;
constexpr int NCH = KDIM / 128;
constexpr int NTHREADS = 512;
constexpr int NWG = 256;
constexpr int NCOPY = 88;
#define AWP 2
#define NPUB 1
#define HS 4
#define NPW 147
#define NPUBW (88 * NPUB)
#define NPUSH 1
#define RPF 7
constexpr int NDIR1 = 56 - 4 * RPF;
constexpr int FOR0 = 88;
constexpr int OWN0 = 200;
constexpr int NGEMM = NWG - FOR0;
constexpr int NG2 = 128;
constexpr int LDS_BYTES = 3 * (256 * 128 + 64 * 256);
constexpr unsigned long long DEADLINE = 50000000ull;

constexpr size_t WS_CTR = 0;
constexpr size_t WS_TMO = 0x4000;
constexpr size_t WS_Q = 0x4100;
constexpr size_t WS_FA = 0x10000;
constexpr size_t WS_FB = 0x18000;
constexpr size_t WS_AU = 0x1A000;
constexpr size_t WS_AB = 0x1E000;
constexpr size_t WS_DBG = 0x20000;
constexpr size_t WS_AA = 0x28000;
constexpr size_t WS_DATA = 1u << 20;
constexpr size_t UNIT = 65536;
constexpr size_t D_XR = 0;
constexpr size_t D_WR = D_XR + (size_t)12 * NCH * UNIT;
constexpr size_t D_PX = D_WR + (size_t)4 * NCH * UNIT;
constexpr size_t D_PW = D_PX + (size_t)4 * NCH * UNIT;
constexpr size_t D_STG = D_PW + (size_t)4 * NCH * UNIT;
constexpr size_t D_RB = D_STG + (size_t)128 * UNIT;
constexpr size_t PAR_BYTES = D_RB + (size_t)8 * 14 * UNIT;
constexpr size_t WS_NEED = WS_DATA + 2 * PAR_BYTES;
static_assert(RPF >= 0 && RPF <= 14, "relay count");

struct Params {
    const char* x; const char* w; char* out;
    char* ws[NR]; char* outs[NR];
    int rank;
};

__device__ __forceinline__ void st_sys(unsigned* p, unsigned v) { __hip_atomic_store(p, v, __ATOMIC_RELAXED, __HIP_MEMORY_SCOPE_SYSTEM); }
__device__ __forceinline__ unsigned ld_agent(const unsigned* p) { return __hip_atomic_load(p, __ATOMIC_RELAXED, __HIP_MEMORY_SCOPE_AGENT); }
__device__ __forceinline__ void st_agent(unsigned* p, unsigned v) { __hip_atomic_store(p, v, __ATOMIC_RELAXED, __HIP_MEMORY_SCOPE_AGENT); }
__device__ __forceinline__ bool geq(unsigned v, unsigned call) { return (int)(v - call) >= 0; }
__device__ __forceinline__ int opaque_tid() { int t = threadIdx.x; asm volatile("" : "+v"(t)); return t; }
__device__ __forceinline__ bool thread0() { return opaque_tid() == 0; }
__device__ __forceinline__ bool lane0() { return (opaque_tid() & 63) == 0; }
#define WAIT_V_(n) asm volatile("s_waitcnt vmcnt(" #n ")" ::: "memory")
#define WAIT_V(n) WAIT_V_(n)
#define WAIT_L(n) asm volatile("s_waitcnt lgkmcnt(" #n ")" ::: "memory")
#define BAR __builtin_amdgcn_s_barrier()
#define SCHED __builtin_amdgcn_sched_barrier(0)
#define STAMP(wsl, i) do { } while (0)

__device__ __forceinline__ void st16_sys(void* p, u32x4 v) {
    asm volatile("global_store_dwordx4 %0, %1, off sc0 sc1\n\ts_nop 1" :: "v"(p), "v"(v) : "memory");
}
__device__ __forceinline__ unsigned long long poll_once(const unsigned* p, bool need, unsigned call) {
    unsigned v;
    asm volatile("global_load_dword %0, %1, off sc0 sc1\n\ts_waitcnt vmcnt(0)" : "=v"(v) : "v"(p) : "memory");
    return __ballot(!need || geq(v, call));
}
__device__ __forceinline__ bool timed_out(const unsigned* tmo, unsigned call) {
    unsigned tv;
    asm volatile("global_load_dword %0, %1, off sc1\n\ts_waitcnt vmcnt(0)" : "=v"(tv) : "v"(tmo) : "memory");
    return __builtin_amdgcn_readfirstlane(tv) == call;
}
__device__ __forceinline__ bool spin_check(unsigned* tmo, unsigned call, unsigned long long t0, unsigned& spins) {
    if ((++spins & 7u) == 0u) {
        if (timed_out(tmo, call)) return true;
        if (__builtin_amdgcn_s_memrealtime() - t0 > DEADLINE) { st_agent(tmo, call); return true; }
    }
    __builtin_amdgcn_s_sleep(1);
    return false;
}
__device__ __forceinline__ void wait_all(const unsigned* p, bool need, unsigned call, unsigned* tmo) {
    if (poll_once(p, need, call) == ~0ull) return;
    const unsigned long long t0 = __builtin_amdgcn_s_memrealtime();
    unsigned spins = 7;
    for (;;) {
        if (spin_check(tmo, call, t0, spins)) return;
        if (poll_once(p, need, call) == ~0ull) return;
    }
}

__host__ __device__ __forceinline__ int lds_byte(int r, int c) { const int st = (r >> 4) * 2 + (c >> 5), rr = r & 15, cc = c & 31, ob = rr * 64 + cc * 2; return st * 1024 + (ob ^ (((ob >> 9) & 1) << 5)); }
__host__ __device__ __forceinline__ void stage_rc(int b, int& R, int& C) { const int st = b / 1024, sb = b % 1024, swz = sb ^ (((sb >> 9) & 1) << 5); R = (st >> 1) * 16 + swz / 64; C = (st & 1) * 32 + (swz % 64) / 2; }
__device__ __forceinline__ unsigned cvt_pk_bf16(float lo, float hi) { unsigned r; asm volatile("v_cvt_pk_bf16_f32 %0, %1, %2" : "=v"(r) : "v"(lo), "v"(hi)); return r; }
__device__ __forceinline__ float gelu_tanh(float y) {
    const float k1 = -2.0f * 0.7978845608028654f * 1.4426950408889634f;
    const float k2 = k1 * 0.044715f;
    const float t = y * (k2 * (y * y) + k1);
    return y * __builtin_amdgcn_rcpf(1.0f + __builtin_amdgcn_exp2f(t));
}
__device__ __forceinline__ u32x2 gelu_pack(f32x4 v, unsigned bad) {
    u32x2 w; w.x = cvt_pk_bf16(gelu_tanh(v[0]), gelu_tanh(v[1])); w.y = cvt_pk_bf16(gelu_tanh(v[2]), gelu_tanh(v[3]));
    if (bad) { w.x = 0x7fc07fc0u; w.y = 0x7fc07fc0u; }
    return w;
}

struct Tile {
    const char* A; const char* B; char* O;
    unsigned lda, csA;
    unsigned ldb;
    const unsigned* fA; const unsigned* fB;
    unsigned* fO; int nfO;
};

__device__ __forceinline__ unsigned poll_chunks(const unsigned* p, bool need, unsigned call) {
    u32x4 v;
    asm volatile("global_load_dwordx4 %0, %1, off sc0 sc1\n\ts_waitcnt vmcnt(0)" : "=v"(v) : "v"(p) : "memory");
    const unsigned long long mk = __ballot(!need || (geq(v.x, call) && geq(v.y, call) && geq(v.z, call) && geq(v.w, call)));
    const unsigned both = (unsigned)mk & (unsigned)(mk >> 32);
    return both & (both >> 1) & 0x55555555u;
}
__device__ __forceinline__ void ensure2(const unsigned* fA, const unsigned* fB, int c, int& ready, unsigned call, unsigned* tmo, int lane) {
    if (c >= ready) {
        const int cc = c + ((lane >> 1) & 15);
        const unsigned* base = lane < 32 ? fA : fB;
        const bool need = base != nullptr && cc < NCH;
        const unsigned* p = need ? base + cc * 8 + (lane & 1) * 4 : (const unsigned*)((const char*)tmo - WS_TMO + WS_FA);
        unsigned m = poll_chunks(p, need, call);
        if (!(m & 1u)) {
            const unsigned long long t0 = __builtin_amdgcn_s_memrealtime();
            unsigned spins = 7;
            for (;;) {
                if (spin_check(tmo, call, t0, spins)) { m = 0x55555555u; break; }
                m = poll_chunks(p, need, call);
                if (m & 1u) break;
            }
        }
        const unsigned miss = ~m & 0x55555555u;
        ready = c + (miss ? (__builtin_ctz(miss) >> 1) : 16);
    }
}
__device__ __forceinline__ void ensure(const Tile& tl, int c, int& ready, unsigned call, unsigned* tmo, int lane) { ensure2(tl.fA, tl.fB, c, ready, call, tmo, lane); }
#define RD128(dst, addr, off) asm volatile("ds_read_b128 %0, %1 offset:%2" : "=v"(dst) : "v"(addr), "i"(off) : "memory")
#define TRRD(dst, addr, off) asm volatile("ds_read_b64_tr_b16 %0, %1 offset:%2" : "=v"(dst) : "v"(addr), "i"(off) : "memory")
#define GLDS(g, l) __builtin_amdgcn_global_load_lds((const unsigned*)(g), (LAS unsigned*)(l), 16, 0, 0)

template <int BM, int KT>
__device__ __forceinline__ void gemm_tile(LAS unsigned char* lds, const Tile& tl, const unsigned call, unsigned* tmo, int sbase) {
    char* const wsl_dbg = (char*)tmo - WS_TMO; (void)wsl_dbg; (void)sbase;
    constexpr int WAVES_M = BM == 256 ? 4 : 2, WAVES_N = 8 / WAVES_M;
    constexpr int MF = BM / WAVES_M / 16, NF = 128 / WAVES_N / 16;
    constexpr int ABYTES = BM * 128, TBYTES = ABYTES + 16384, SBYTES = KT * TBYTES;
    constexpr int RA = ABYTES / 8192;
    constexpr int NS = NT / KT;
    static_assert(3 * SBYTES <= LDS_BYTES, "LDS");
    const int tid = threadIdx.x, wid = __builtin_amdgcn_readfirstlane(tid >> 6), lane = tid & 63;
    const int wr = wid / WAVES_N, wc = wid % WAVES_N, fr = lane & 15, fq = lane >> 4;
    unsigned voffA[RA], voffB[2];
#pragma unroll
    for (int i = 0; i < RA; ++i) { int R, C; stage_rc(tid * 16 + i * 8192, R, C); voffA[i] = (unsigned)R * tl.lda + (unsigned)C * 2u; }
#pragma unroll
    for (int i = 0; i < 2; ++i) { const int b = tid * 16 + i * 8192, row = b >> 8, chpos = (b >> 4) & 15, ch = chpos ^ (((row & 3) << 2) | ((row >> 2) & 3));
        voffB[i] = (unsigned)row * tl.ldb + (unsigned)ch * 16u; }
    const size_t kstepB = (size_t)64 * tl.ldb;
    const unsigned ldsbase = (unsigned)(size_t)lds;
    const unsigned ldsw = (unsigned)wid * 1024u;
    const unsigned aoff = ldsbase + (unsigned)lds_byte(wr * (MF * 16) + fr, fq * 8);
    unsigned boff0;
    { const int g = lane >> 4, q = (lane & 15) >> 2, p = lane & 3; const int row = 8 * g + q, ch = 2 * (wc * NF) + (p >> 1);
      boff0 = (unsigned)(ABYTES + 256 * row + 16 * (ch ^ (((row & 3) << 2) | ((row >> 2) & 3))) + 8 * (p & 1)); }
    unsigned boff[NF][2];
#pragma unroll
    for (int n = 0; n < NF; ++n)
#pragma unroll
        for (int t = 0; t < 2; ++t) boff[n][t] = ldsbase + (boff0 ^ (unsigned)(n * 32 + t * 16)) + (unsigned)(t * 1024);

    f32x4 acc[MF][NF];
#pragma unroll
    for (int m = 0; m < MF; ++m)
#pragma unroll
        for (int n = 0; n < NF; ++n) acc[m][n] = (f32x4){0.f, 0.f, 0.f, 0.f};

    int ready = (tl.fA == nullptr && tl.fB == nullptr) ? NCH : 0;
#define STAGE_TILE(st, sb) do { _Pragma("unroll") for (int _j = 0; _j < KT; ++_j) { const int _kt = (st) * KT + _j; \
        const char* _a = tl.A + (size_t)(_kt >> 1) * tl.csA + (_kt & 1) * 128; const char* _b = tl.B + (size_t)_kt * kstepB; \
        _Pragma("unroll") for (int _i = 0; _i < RA; ++_i) GLDS(_a + voffA[_i], lds + (sb) + _j * TBYTES + ldsw + _i * 8192); \
        _Pragma("unroll") for (int _i = 0; _i < 2; ++_i) GLDS(_b + voffB[_i], lds + (sb) + _j * TBYTES + ABYTES + ldsw + _i * 8192); } } while (0)

    if (wid == 0) ensure(tl, 0, ready, call, tmo, lane);
    BAR;
    STAMP(wsl_dbg, sbase);
    STAGE_TILE(0, 0);
    if (KT == 2 && wid == 0) ensure(tl, 1, ready, call, tmo, lane);
    if (KT == 2) BAR;
    STAGE_TILE(1, SBYTES);
    unsigned sb_rd = 0, sb_wr = 2 * SBYTES;
    for (int t = 0; t < NS; ++t) {
        const int tn = t + 2 < NS ? t + 2 : NS - 1;
        if (wid == 0 && (KT == 2 || (t & 1) == 0)) ensure(tl, KT == 2 ? tn : tn >> 1, ready, call, tmo, lane);
        WAIT_V(6);
        static_assert((RA + 2) * KT == 6, "vmcnt");
        BAR;
        STAGE_TILE(tn, sb_wr);
        s16x8 Af[2 * KT][MF]; s16x4 Bl[2 * KT][NF], Bh[2 * KT][NF];
        const unsigned a = aoff + sb_rd;
#pragma unroll
        for (int k = 0; k < 2 * KT; ++k) {
#pragma unroll
            for (int m = 0; m < MF; ++m) RD128(Af[k][m], a, (k >> 1) * TBYTES + m * 2048 + (k & 1) * 1024);
#pragma unroll
            for (int n = 0; n < NF; ++n) { TRRD(Bl[k][n], boff[n][0] + sb_rd, (k >> 1) * TBYTES + (k & 1) * 8192); TRRD(Bh[k][n], boff[n][1] + sb_rd, (k >> 1) * TBYTES + (k & 1) * 8192); }
        }
#pragma unroll
        for (int k = 0; k < 2 * KT; ++k) {
            if constexpr (BM == 256) { if (k == 0) WAIT_L(12); else WAIT_L(0); }
            else { if (k == 0) WAIT_L(15); else if (k == 1) WAIT_L(12); else if (k == 2) WAIT_L(6); else WAIT_L(0); }
            SCHED;
            __builtin_amdgcn_s_setprio(1);
#pragma unroll
            for (int m = 0; m < MF; ++m)
#pragma unroll
                for (int n = 0; n < NF; ++n) {
                    const s16x8 bf = __builtin_shufflevector(Bl[k][n], Bh[k][n], 0, 1, 2, 3, 4, 5, 6, 7);
                    acc[m][n] = __builtin_amdgcn_mfma_f32_16x16x32_bf16(__builtin_bit_cast(bf16x8, bf), __builtin_bit_cast(bf16x8, Af[k][m]), acc[m][n], 0, 0, 0);
                }
            __builtin_amdgcn_s_setprio(0);
            SCHED;
        }
        sb_rd = sb_rd == 2 * SBYTES ? 0 : sb_rd + SBYTES;
        sb_wr = sb_wr == 2 * SBYTES ? 0 : sb_wr + SBYTES;
    }
#undef STAGE_TILE
    WAIT_V(0);
    BAR;
    STAMP(wsl_dbg, sbase + 1);
    unsigned bad = 0;
    if (tl.fA != nullptr || tl.fB != nullptr) bad = timed_out(tmo, call);
    const int tz = opaque_tid();
    const int lane2 = tz & 63, fr2 = lane2 & 15, fq2 = lane2 >> 4;
#pragma unroll
    for (int m = 0; m < MF; ++m) { const int row = wr * (MF * 16) + m * 16 + fr2;
#pragma unroll
        for (int n = 0; n < NF; ++n) {
            const int col = wc * (NF * 16) + n * 16 + fq2 * 4; const int ch = col >> 3, hf = (col >> 2) & 1;
            *(LAS u32x2*)(lds + row * 256 + ((ch ^ fr2) << 4) + hf * 8) = gelu_pack(acc[m][n], bad); } }
    WAIT_L(0); BAR;
    constexpr int RPW = BM / 8;
    if (tl.fO != nullptr) {
#pragma unroll 4
        for (int i = 0; i < RPW / 4; ++i) { const int row = wid * RPW + i * 4 + (lane2 >> 4), chp = lane2 & 15;
            const u32x4 v = *(const LAS u32x4*)(lds + row * 256 + chp * 16);
            st16_sys(tl.O + (size_t)row * 256 + ((chp ^ (row & 15)) << 4), v); }
        WAIT_V(0);
        BAR;
        const int tf = opaque_tid();
        if (tf < tl.nfO) st_sys(tl.fO + tf, call);
    } else {
#pragma unroll 4
        for (int i = 0; i < RPW / 4; ++i) { const int row = wid * RPW + i * 4 + (lane2 >> 4), chp = lane2 & 15;
            const u32x4 v = *(const LAS u32x4*)(lds + row * 256 + chp * 16);
            *(u32x4*)(tl.O + (size_t)row * 2048 + ((chp ^ (row & 15)) << 4)) = v; }
        WAIT_L(0);
        BAR;
    }
    STAMP(wsl_dbg, sbase + 2);
}

__device__ __forceinline__ void gemm_tile_256(LAS unsigned char* lds, const Tile& tl, const unsigned call, unsigned* tmo) {
    char* const wsl_dbg = (char*)tmo - WS_TMO; (void)wsl_dbg;
    constexpr int HALF = 128, HTB = HALF * 64 * 2;
    const int tid = threadIdx.x, wid = __builtin_amdgcn_readfirstlane(tid >> 6), lane = tid & 63, wr = wid >> 2, wc = wid & 3, fr = lane & 15, fq = lane >> 4;
    unsigned voffA[2], voffB[2];
#pragma unroll
    for (int i = 0; i < 2; ++i) { int R, C; stage_rc(tid * 16 + i * 8192, R, C); voffA[i] = (unsigned)R * tl.lda + (unsigned)C * 2u;
        const int b = tid * 16 + i * 8192, row = b >> 8, chpos = (b >> 4) & 15, ch = chpos ^ (((row & 3) << 2) | ((row >> 2) & 3));
        voffB[i] = (unsigned)row * tl.ldb + (unsigned)ch * 16u; }
    const size_t kstepB = (size_t)64 * tl.ldb;
    const size_t hstepA = (size_t)HALF * tl.lda, hstepB = 256;
    const unsigned ldsw = (unsigned)wid * 1024u;
    const unsigned ldsbase = (unsigned)(size_t)lds;
    const unsigned aoff = ldsbase + (unsigned)lds_byte(wr * 64 + fr, fq * 8);
    unsigned boff[2][2];
    { const int g = lane >> 4, q = (lane & 15) >> 2, p = lane & 3;
#pragma unroll
      for (int n = 0; n < 2; ++n)
#pragma unroll
        for (int t = 0; t < 2; ++t) { const int row = 8 * g + 4 * t + q, ch = 4 * wc + 2 * n + (p >> 1);
            boff[n][t] = ldsbase + 4 * HTB + 256 * row + 16 * (ch ^ (((row & 3) << 2) | ((row >> 2) & 3))) + 8 * (p & 1); } }
#define SA(b, h) (((b) * 2 + (h)) * HTB)
#define SB(b, h) ((4 + (b) * 2 + (h)) * HTB)
#define STAGE(bufoff, gbase, voff) do { _Pragma("unroll") for (int _i = 0; _i < 2; ++_i) GLDS((const char*)(gbase) + (voff)[_i], lds + (bufoff) + ldsw + _i * 8192); } while (0)
#define LDA(dst, b, h) do { _Pragma("unroll") for (int m = 0; m < 4; ++m) _Pragma("unroll") for (int k = 0; k < 2; ++k) RD128(dst[m][k], aoff, SA(b, h) + m * 2048 + k * 1024); } while (0)
#define LDB(dst, b, h) do { _Pragma("unroll") for (int n = 0; n < 2; ++n) _Pragma("unroll") for (int k = 0; k < 2; ++k) { \
        s16x4 lo, hi; TRRD(lo, boff[n][0], ((b) * 2 + (h)) * HTB + k * 8192); TRRD(hi, boff[n][1], ((b) * 2 + (h)) * HTB + k * 8192); \
        dst[n][k] = __builtin_shufflevector(lo, hi, 0, 1, 2, 3, 4, 5, 6, 7); } } while (0)
#define MMA(ai, bj, At, Bt) do { __builtin_amdgcn_s_setprio(1); _Pragma("unroll") for (int m = 0; m < 4; ++m) _Pragma("unroll") for (int n = 0; n < 2; ++n) _Pragma("unroll") for (int k = 0; k < 2; ++k) \
        acc[ai][bj][m][n] = __builtin_amdgcn_mfma_f32_16x16x32_bf16(__builtin_bit_cast(bf16x8, Bt[n][k]), __builtin_bit_cast(bf16x8, At[m][k]), acc[ai][bj][m][n], 0, 0, 0); __builtin_amdgcn_s_setprio(0); } while (0)
#define KOFFA(t) ((size_t)((t) >> 1) * tl.csA + ((t) & 1) * 128)
    f32x4 acc[2][2][4][2];
#pragma unroll
    for (int a = 0; a < 2; ++a)
#pragma unroll
        for (int b = 0; b < 2; ++b)
#pragma unroll
            for (int m = 0; m < 4; ++m)
#pragma unroll
                for (int n = 0; n < 2; ++n) acc[a][b][m][n] = (f32x4){0.f, 0.f, 0.f, 0.f};
    s16x8 At[4][2], B0[2][2], B1[2][2];
    const char* cA = tl.A; const char* cB = tl.B;
    int ready = tl.fA == nullptr ? NCH : 0;
    if (wid == 0) ensure(tl, 0, ready, call, tmo, lane);
    BAR;
    STAMP(wsl_dbg, 1);
    STAGE(SB(0, 0), cB, voffB); STAGE(SA(0, 0), cA, voffA); STAGE(SB(0, 1), cB + hstepB, voffB); STAGE(SA(0, 1), cA + hstepA, voffA);
    if (wr == 1) BAR;
    WAIT_V(4); BAR;
    STAGE(SB(1, 0), cB + kstepB, voffB); STAGE(SA(1, 0), cA + 128, voffA); STAGE(SB(1, 1), cB + hstepB + kstepB, voffB);
    WAIT_V(6); BAR;
    for (int t = 0; t < NT; t += 2) {
        const int t2 = (t == NT - 2) ? 0 : t + 2;
        if (wid == 0) ensure(tl, t2 >> 1, ready, call, tmo, lane);
        const char* a1 = cA + KOFFA(t + 1);
        const char* a2 = cA + KOFFA(t2); const char* b2 = cB + (size_t)t2 * kstepB;
        const char* a3 = a2 + 128; const char* b3 = b2 + kstepB;
        LDB(B0, 0, 0); SCHED; LDA(At, 0, 0); STAGE(SA(1, 1), a1 + hstepA, voffA);
        WAIT_L(8); BAR; WAIT_L(0); SCHED; MMA(0, 0, At, B0); BAR; SCHED;
        LDB(B1, 0, 1); STAGE(SB(0, 0), b2, voffB);
        BAR; WAIT_L(0); SCHED; MMA(0, 1, At, B1); BAR; SCHED;
        LDA(At, 0, 1); STAGE(SA(0, 0), a2, voffA);
        BAR; WAIT_L(0); SCHED; MMA(1, 0, At, B0); BAR; SCHED;
        STAGE(SB(0, 1), b2 + hstepB, voffB);
        WAIT_V(6); BAR; MMA(1, 1, At, B1); BAR; SCHED;
        LDB(B0, 1, 0); SCHED; LDA(At, 1, 0); STAGE(SA(0, 1), a2 + hstepA, voffA);
        WAIT_L(8); BAR; WAIT_L(0); SCHED; MMA(0, 0, At, B0); BAR; SCHED;
        LDB(B1, 1, 1); STAGE(SB(1, 0), b3, voffB);
        BAR; WAIT_L(0); SCHED; MMA(0, 1, At, B1); BAR; SCHED;
        LDA(At, 1, 1); STAGE(SA(1, 0), a3, voffA);
        BAR; WAIT_L(0); SCHED; MMA(1, 0, At, B0); BAR; SCHED;
        STAGE(SB(1, 1), b3 + hstepB, voffB);
        WAIT_V(6); BAR; MMA(1, 1, At, B1); BAR; SCHED;
    }
    WAIT_V(0);
    if (wr == 0) BAR;
    BAR;
    STAMP(wsl_dbg, 2);
    unsigned bad = 0;
    if (tl.fA != nullptr) bad = timed_out(tmo, call);
    const int tz = opaque_tid();
    const int lane2 = tz & 63, fr2 = lane2 & 15, fq2 = lane2 >> 4;
#pragma unroll
    for (int ai = 0; ai < 2; ++ai)
#pragma unroll
        for (int m = 0; m < 4; ++m) { const int row = ai * HALF + wr * 64 + m * 16 + fr2;
#pragma unroll
            for (int bj = 0; bj < 2; ++bj)
#pragma unroll
                for (int n = 0; n < 2; ++n) {
                    const int col = bj * HALF + wc * 32 + n * 16 + fq2 * 4; const int ch = col >> 3, hf = (col >> 2) & 1;
                    *(LAS u32x2*)(lds + row * 512 + ((ch ^ fr2) << 4) + hf * 8) = gelu_pack(acc[ai][bj][m][n], bad); } }
    WAIT_L(0); BAR;
#pragma unroll 4
    for (int i = 0; i < 16; ++i) { const int row = wid * 32 + i * 2 + (lane2 >> 5), chp = lane2 & 31;
        const u32x4 v = *(const LAS u32x4*)(lds + row * 512 + chp * 16);
        *(u32x4*)(tl.O + (size_t)row * 2048 + ((chp ^ (row & 15)) << 4)) = v; }
    WAIT_L(0);
    BAR;
    STAMP(wsl_dbg, 3);
#undef SA
#undef SB
#undef STAGE
#undef LDA
#undef LDB
#undef MMA
#undef KOFFA
}

__device__ __forceinline__ int block_row(int r, int rb) {
    if (rb < 4) return r * 1024 + rb * 256;
    if (rb < 8) return (r ^ 1) * 1024 + (rb - 4) * 256;
    const int j = (rb - 8) >> 1, xv = (j & 1) + 2 + (j >> 1) * 4;
    return (r ^ xv) * 1024 + (r >> 2) * 512 + (rb & 1) * 256;
}
__device__ __forceinline__ int g1_block(int g1) { return g1 < 5 ? g1 : (g1 == 5 ? 6 : g1 + 2); }
__device__ __forceinline__ int link_xor(int L) { return L == 0 ? 1 : (L < 5 ? ((L - 1) & 1) + 2 + ((L - 1) >> 1) * 4 : L - 1); }
__device__ __forceinline__ int xlink_index(int xv) { return (xv & 1) + ((xv >> 2) & 1) * 2; }
__device__ __forceinline__ Tile make_tile(const Params& p, char* wsl, size_t paroff, int rb, int sub, int cb, int ou) {
    Tile t; const int ro = sub < 0 ? 0 : sub * 64; char* par = wsl + paroff; const int r = p.rank;
    if (rb < 4) { t.A = p.x + (size_t)(rb * 256 + ro) * XROW; t.lda = XROW; t.csA = 256; t.fA = nullptr; }
    else { t.A = par + D_XR + (size_t)(rb - 4) * NCH * UNIT + ro * 256; t.lda = 256; t.csA = (unsigned)UNIT; t.fA = (const unsigned*)(wsl + WS_FA) + rb * NCH * 8; }
    if (cb < 8) {
        t.B = p.w + cb * 256; t.ldb = 2048; t.fB = nullptr;
        t.O = p.out + (size_t)(block_row(r, rb) + ro) * 2048 + cb * 256; t.fO = nullptr; t.nfO = 0;
    } else {
        const int h = (cb - 8) >> 2, cl = cb & 3;
        const int cp = h * 2 + (cl >> 1);
        t.B = par + D_WR + (size_t)cp * NCH * UNIT + (cl & 1) * 256; t.ldb = 512;
        t.fB = (const unsigned*)(wsl + WS_FB) + cp * NCH * 8;
        const int g = ou * 4 + cl, uo = h * 64 + g;
        t.O = par + D_STG + (size_t)uo * UNIT + ro * 256;
        t.nfO = sub < 0 ? 8 : 2;
        if (g < NDIR1 || g >= 56)
            t.fO = (unsigned*)(p.ws[r ^ (4 + h)] + WS_AU) + ((5 + h) * 64 + (g < NDIR1 ? g : g - 4 * RPF)) * 8 + (sub < 0 ? 0 : sub * 2);
        else { const int jx = (g - NDIR1) & 3, k = (g - NDIR1) >> 2;
            t.fO = (unsigned*)(p.ws[r ^ link_xor(jx + 1)] + WS_AU) + ((jx + 1) * 64 + (26 - (r & 1)) + k * 2 + h) * 8; }
    }
    return t;
}

struct Slice { const char* src; char* dst; char* dst1; char* dst2; unsigned sstride, dstride; int sh; int dep; unsigned* f0; unsigned* f1; unsigned* f2; };

constexpr int NS1 = 2 * (NCH - NPUSH);
__device__ __forceinline__ int link_late(int r, int L) { return L == 0 ? 26 : 26 - ((r ^ link_xor(L)) & 1); }
__device__ __forceinline__ int link_total(int r, int L) { return 8 * (L == 7 ? 512 : (L >= 5 ? NS1 + NDIR1 + 8 : NS1 + link_late(r, L) + (L == 0 ? 0 : 4 * RPF))); }
__device__ __forceinline__ void make_slice(const Params& p, char* wsl, size_t paroff, int L, int tk, Slice& u) {
    const int r = p.rank;
    char* par = wsl + paroff;
    const int un = tk >> 3, sl = tk & 7;
    unsigned* const fA = (unsigned*)(wsl + WS_FA);
    u.dep = -1; u.sh = 4; u.sstride = 256; u.dstride = 256; u.f0 = nullptr; u.f1 = nullptr; u.f2 = nullptr; u.dst1 = nullptr; u.dst2 = nullptr;
    if (L == 7) {
        const int c = un >> 3, b = un & 7;
        if (b < 4) {
            u.src = p.x + (size_t)(b * 256 + sl * 32) * XROW + c * 256; u.sstride = XROW;
            const int hi = ((r >> 2) ^ (b >> 1)) & 1;
            const int a1 = r ^ (2 + 4 * hi), a2 = r ^ (3 + 4 * hi), rb1 = 8 + 4 * hi + (b & 1), rb2 = rb1 + 2;
            unsigned* mate_late = (unsigned*)(p.ws[r ^ 1] + WS_AU) + (2 * (c / 5) + (b >> 1)) * 8 + sl;
            char* pub = par + D_PX + ((size_t)b * NCH + c) * UNIT + sl * 8192;
            if (c >= NPUSH) {
                u.dst = pub;
                if (!(b & 1)) u.f0 = (unsigned*)(p.ws[r ^ 1] + WS_AA) + ((4 + b) * NCH + c) * 8 + sl;
                else if (c % 5 == 0) u.f0 = mate_late;
                u.f1 = (unsigned*)(p.ws[a1] + WS_AA) + (rb1 * NCH + c) * 8 + sl;
                u.f2 = (unsigned*)(p.ws[a2] + WS_AA) + (rb2 * NCH + c) * 8 + sl;
            } else {
                u.dst = p.ws[a1] + paroff + D_XR + ((size_t)(rb1 - 4) * NCH + c) * UNIT + sl * 8192;
                u.dst1 = p.ws[a2] + paroff + D_XR + ((size_t)(rb2 - 4) * NCH + c) * UNIT + sl * 8192;
                u.f1 = (unsigned*)(p.ws[a1] + WS_FA) + (rb1 * NCH + c) * 8 + sl;
                u.f2 = (unsigned*)(p.ws[a2] + WS_FA) + (rb2 * NCH + c) * 8 + sl;
                if (!(b & 1)) { u.dst2 = p.ws[r ^ 1] + paroff + D_XR + ((size_t)b * NCH + c) * UNIT + sl * 8192;
                    u.f0 = (unsigned*)(p.ws[r ^ 1] + WS_FA) + ((4 + b) * NCH + c) * 8 + sl; }
                else if (c % 5 == 0) { u.dst2 = pub; u.f0 = mate_late; }
                else {
                    const int a = ((c % 5 - 1) & 1) == (a1 & 1) ? a1 : a2;
                    u.f0 = (unsigned*)(p.ws[r ^ 1] + WS_AU) + ((1 + xlink_index(r ^ 1 ^ a)) * 64 + 2 * (c / 5) + ((c % 5 - 1) >> 1)) * 8 + sl; }
            }
        } else {
            const int cp = b - 4, J = r ^ (4 + (cp >> 1));
            u.src = p.w + (size_t)(c * 128 + sl * 16) * 2048 + cp * 512; u.sstride = 2048; u.dstride = 512; u.sh = 5;
            if (c >= NPUSH) {
                u.dst = par + D_PW + ((size_t)cp * NCH + c) * UNIT + sl * 8192;
                u.f0 = (unsigned*)(p.ws[J] + WS_AB) + (cp * NCH + c) * 8 + sl;
            } else {
                u.dst = p.ws[J] + paroff + D_WR + ((size_t)cp * NCH + c) * UNIT + sl * 8192;
                u.f0 = (unsigned*)(p.ws[J] + WS_FB) + (cp * NCH + c) * 8 + sl;
            }
        }
        return;
    }
    const int xv = link_xor(L);
    const int peer = r ^ xv;
    const char* ppar = p.ws[peer] + paroff;
    if (un < NS1) {
        const int c = NPUSH + (un >> 1), b = un & 1;
        if (L < 5) {
            const int blk = L == 0 ? 2 * b : 2 * (r >> 2) + b;
            const int rb = L == 0 ? 4 + 2 * b : 8 + 2 * (L - 1) + b;
            u.src = ppar + D_PX + ((size_t)blk * NCH + c) * UNIT + sl * 8192;
            u.dst = par + D_XR + ((size_t)(rb - 4) * NCH + c) * UNIT + sl * 8192;
            u.f0 = fA + (rb * NCH + c) * 8 + sl;
            if (L != 0 && b == 1) { const int c5 = c % 5 - 1 - (r & 1);
                if (c5 == 0 || c5 == 2) u.f1 = (unsigned*)(p.ws[peer ^ 1] + WS_AU) + ((1 + xlink_index(peer ^ 1 ^ r)) * 64 + 2 * (c / 5) + (c5 >> 1)) * 8 + sl; }
        } else {
            const int cp = (L - 5) * 2 + b;
            u.src = ppar + D_PW + ((size_t)cp * NCH + c) * UNIT + sl * 8192;
            u.dst = par + D_WR + ((size_t)cp * NCH + c) * UNIT + sl * 8192;
            u.f0 = (unsigned*)(wsl + WS_FB) + (cp * NCH + c) * 8 + sl;
        }
        return;
    }
    int m = un - NS1;
    u.dep = m;
    if (L >= 5) {
        const int h = L - 5, g = m < NDIR1 ? m : m + 4 * RPF, ou = g >> 2, cl = g & 3, uo = h * 64 + g;
        const int rb = ou < 14 ? g1_block(ou) : 5 + 2 * (ou - 14);
        u.src = ppar + D_STG + (size_t)uo * UNIT + sl * 8192;
        u.dst = p.out + (size_t)(block_row(peer, rb) + sl * 32) * 2048 + h * 1024 + cl * 256; u.dstride = 2048;
        return;
    }
    const int nlate = link_late(r, L);
    if (m < nlate) {
        int c, blk;
        if (L == 0) { c = (m >> 1) * 5; blk = 1 + 2 * (m & 1); u.src = ppar + D_PX + ((size_t)blk * NCH + c) * UNIT + sl * 8192; }
        else { c = (m >> 1) * 5 + 1 + (peer & 1) + 2 * (m & 1); blk = 2 * (peer >> 2) + 1;
            const int rbl = 8 + 2 * xlink_index(peer ^ r ^ 1) + 1;
            u.src = ppar + D_XR + ((size_t)(rbl - 4) * NCH + c) * UNIT + sl * 8192; }
        u.dst = par + D_XR + ((size_t)blk * NCH + c) * UNIT + sl * 8192;
        u.f0 = fA + ((4 + blk) * NCH + c) * 8 + sl;
        return;
    }
    m -= nlate;
    const int h = m & 1;
    if (m < 2 * RPF) {
        const int k = m >> 1, g = NDIR1 + k * 4 + (L - 1), uo = h * 64 + g, slot = ((L - 1) * 2 + h) * RPF + k, J = peer ^ (4 + h);
        u.src = ppar + D_STG + (size_t)uo * UNIT + sl * 8192;
        u.dst = par + D_RB + (size_t)slot * UNIT + sl * 8192;
        u.f0 = (unsigned*)(p.ws[J] + WS_AU) + ((1 + xlink_index(J ^ r)) * 64 + (26 - (r & 1)) + 2 * RPF + m) * 8 + sl;
    } else {
        const int k = (m - 2 * RPF) >> 1, rs = r ^ (4 + h), jx = xlink_index(rs ^ peer);
        const int g = NDIR1 + k * 4 + jx, ou = g >> 2, cl = g & 3, slot = (jx * 2 + h) * RPF + k;
        u.src = ppar + D_RB + (size_t)slot * UNIT + sl * 8192;
        u.dst = p.out + (size_t)(block_row(rs, g1_block(ou)) + sl * 32) * 2048 + h * 1024 + cl * 256; u.dstride = 2048;
    }
}
template <int I0, int I1>
__device__ __forceinline__ void slice_load(const Slice& u, u32x4 (&v)[8], int lane) {
    const char* s = u.src + (size_t)(lane >> u.sh) * u.sstride + (lane & ((1 << u.sh) - 1)) * 16;
    const size_t st = (size_t)(64 >> u.sh) * u.sstride;
#pragma unroll
    for (int i = I0; i < I1; ++i) asm volatile("global_load_dwordx4 %0, %1, off sc0 sc1" : "=v"(v[i]) : "v"(s + i * st) : "memory");
}
__device__ __forceinline__ void slice_store(const Slice& u, u32x4 (&v)[8], int lane) {
    const size_t lo = (size_t)(lane >> u.sh) * u.dstride + (lane & ((1 << u.sh) - 1)) * 16;
    const size_t st = (size_t)(64 >> u.sh) * u.dstride;
#pragma unroll
    for (int i = 0; i < 8; ++i) st16_sys(u.dst + lo + i * st, v[i]);
    if (u.dst1 != nullptr) {
#pragma unroll
        for (int i = 0; i < 8; ++i) st16_sys(u.dst1 + lo + i * st, v[i]);
    }
    if (u.dst2 != nullptr) {
#pragma unroll
        for (int i = 0; i < 8; ++i) st16_sys(u.dst2 + lo + i * st, v[i]);
    }
}
#define PIN8(v) asm volatile("" : "+v"(v[0]), "+v"(v[1]), "+v"(v[2]), "+v"(v[3]), "+v"(v[4]), "+v"(v[5]), "+v"(v[6]), "+v"(v[7]))

__device__ __forceinline__ void copy_wave(const Params& p, char* wsl, size_t paroff, int L, int j, int nw, unsigned call, unsigned* tmo) {
    const int lane = threadIdx.x & 63;
    const int total = link_total(p.rank, L);
    unsigned* ctr = (unsigned*)(wsl + WS_Q + 128 * L);
    const unsigned tbase = (call - 1u) * (unsigned)total - (unsigned)nw;
    u32x4 va[8], vb[8];
    Slice ua, ub;
    unsigned tkv = 0;
#define DRAW() do { if (lane0()) asm volatile("global_atomic_add %0, %1, %2, off sc0" : "=v"(tkv) : "v"(ctr), "v"(1u) : "memory"); } while (0)
#define PUBLISH(a, b, c) do { const int _l = opaque_tid() & 63; unsigned* _f = _l == 0 ? (a) : (_l == 1 ? (b) : (c)); if (_l < 3 && _f != nullptr) st_sys(_f, call); } while (0)
    unsigned* p0 = nullptr; unsigned* p1 = nullptr; unsigned* p2 = nullptr;
    const unsigned* av0 = L < 5 ? (const unsigned*)(wsl + WS_AA) + (L == 0 ? 4 : 8 + 2 * (L - 1)) * NCH * 8 : (const unsigned*)(wsl + WS_AB) + (L - 5) * 2 * NCH * 8;
    const unsigned* av1 = av0 + (L == 0 ? 2 : 1) * NCH * 8;
    const unsigned* avU = (const unsigned*)(wsl + WS_AU) + L * 64 * 8;
    int aready = L == 7 ? NCH : 0, uready = 0;
    make_slice(p, wsl, paroff, L, j, ua);
    DRAW();
    if (L < 7) ensure2(av0, av1, NPUSH + (j >> 4), aready, call, tmo, lane);
    slice_load<0, 8>(ua, va, lane);
#define STEP(cur, vcur, nxt, vnxt) { \
        WAIT_V(0); \
        PIN8(vcur); asm volatile("" : "+v"(tkv)); \
        PUBLISH(p0, p1, p2); \
        const int tk = (int)((unsigned)__builtin_amdgcn_readfirstlane(tkv) - tbase); \
        slice_store(cur, vcur, lane); \
        p0 = cur.f0; p1 = cur.f1; p2 = cur.f2; \
        if (tk >= total) break; \
        make_slice(p, wsl, paroff, L, tk, nxt); \
        const bool w1 = L < 7 && tk < NS1 * 8 && NPUSH + (tk >> 4) >= aready, w2 = nxt.dep >= uready; \
        if (w1 || w2) { \
            WAIT_V(0); PUBLISH(p0, p1, p2); p0 = nullptr; p1 = nullptr; p2 = nullptr; \
            if (w1) ensure2(av0, av1, NPUSH + (tk >> 4), aready, call, tmo, lane); \
            else ensure2(avU, nullptr, nxt.dep, uready, call, tmo, lane); } \
        DRAW(); \
        if (HS == 8 || L == 7) slice_load<0, 8>(nxt, vnxt, lane); \
        else if (HS == 4) { slice_load<0, 4>(nxt, vnxt, lane); WAIT_V(0); slice_load<4, 8>(nxt, vnxt, lane); } \
        else { slice_load<0, 2>(nxt, vnxt, lane); WAIT_V(0); slice_load<2, 4>(nxt, vnxt, lane); WAIT_V(0); slice_load<4, 6>(nxt, vnxt, lane); WAIT_V(0); slice_load<6, 8>(nxt, vnxt, lane); } \
        if (L < 7 && tk >= NS1 * 8 - 24 && tk < NS1 * 8) STAMP(wsl, 2); \
        if (L < 5 && tk >= (NS1 + 25) * 8 - 24 && tk < (NS1 + 25) * 8) STAMP(wsl, 4); \
        if (L >= 1 && L < 5 && tk >= (NS1 + 26 + 2 * RPF) * 8 - 24 && tk < (NS1 + 26 + 2 * RPF) * 8) STAMP(wsl, 5); \
        if (L >= 5 && L < 7 && tk >= (NS1 + NDIR1) * 8 - 24 && tk < (NS1 + NDIR1) * 8) STAMP(wsl, 5); }
    for (;;) {
        STEP(ua, va, ub, vb)
        STEP(ub, vb, ua, va)
    }
#undef STEP
#undef DRAW
    WAIT_V(0);
    PUBLISH(p0, p1, p2);
#undef PUBLISH
    if (L < 7) STAMP(wsl, 3);
}

__global__ __launch_bounds__(NTHREADS, 2) void ag_gemm_kernel(const Params p) {
    extern __shared__ __attribute__((aligned(16))) unsigned char shm[];
    LAS unsigned char* lds = (LAS unsigned char*)shm;
    char* wsl = p.ws[p.rank];
    unsigned* tmo = (unsigned*)(wsl + WS_TMO);
    const int bid = blockIdx.x, tid = threadIdx.x;
    unsigned call;
    {
        unsigned* ctr = (unsigned*)(wsl + WS_CTR + (size_t)bid * 64);
        unsigned v;
        asm volatile("global_load_dword %0, %1, off sc1\n\ts_waitcnt vmcnt(0)" : "=v"(v) : "v"(ctr) : "memory");
        call = __builtin_amdgcn_readfirstlane(v) + 1u;
        if (call == 0u) call = 1u;
        __syncthreads();
        if (thread0()) st_agent(ctr, call);
    }
    const size_t paroff = WS_DATA + (size_t)(call & 1u) * PAR_BYTES;
    STAMP(wsl, 0);
    if (bid < FOR0) {
        const int wid = __builtin_amdgcn_readfirstlane(tid >> 6);
        if (wid < AWP) { const int gw = bid * AWP + wid, L = gw % 7; if (gw >= NPW) return; copy_wave(p, wsl, paroff, L, gw / 7, (NPW - L + 6) / 7, call, tmo);
            if (L >= 1 && timed_out(tmo, call) && lane0()) *(u16*)(p.out + (size_t)p.rank * 1024 * 2048) = 0x7fc0; }
        else if (wid < AWP + NPUB && bid * NPUB + wid - AWP < NPUBW) copy_wave(p, wsl, paroff, 7, bid * NPUB + wid - AWP, NPUBW, call, tmo);
        return;
    }
    if (bid < OWN0) {
        const int g = bid - FOR0, xcd = g & 7, slot = g >> 3;
        const int g1 = (xcd >> 2) * 7 + slot % 7, cb = 8 + (xcd & 3) * 2 + slot / 7;
        const Tile t = make_tile(p, wsl, paroff, g1_block(g1), -1, cb, g1);
        gemm_tile<256, 1>(lds, t, call, tmo, 1);
    } else {
        const int g = bid - OWN0, xcd = g & 7, slot = g >> 3;
        const int g1 = (xcd >> 2) * 7 + slot, cb = (xcd & 3) * 2;
        const Tile t = make_tile(p, wsl, paroff, g1_block(g1), -1, cb, g1);
        gemm_tile_256(lds, t, call, tmo);
    }
    const int xq = bid & 7;
    unsigned* q = (unsigned*)(wsl + WS_Q + 128 * (8 + xq));
    const unsigned qbase = (call - 1u) * (unsigned)(NG2 / 8 + NGEMM / 8);
    for (;;) {
        LAS unsigned* sw = (LAS unsigned*)(lds + LDS_BYTES - 16);
        if (thread0()) *sw = __hip_atomic_fetch_add(q, 1u, __ATOMIC_RELAXED, __HIP_MEMORY_SCOPE_AGENT) - qbase;
        __syncthreads();
        const unsigned tk = __builtin_amdgcn_readfirstlane(*sw);
        __syncthreads();
        if (tk >= (unsigned)(NG2 / 8)) break;
        const int cb = tk < 8 ? 8 + xq : xq, g2 = tk & 7;
        const Tile t = make_tile(p, wsl, paroff, 5 + 2 * (g2 >> 2), g2 & 3, cb, 14 + (g2 >> 2));
        gemm_tile<64, 2>(lds, t, call, tmo, 4);
    }
}

}

extern "C" void kernel_launch(int rank, int nranks, ncclComm_t comm,
                              hipStream_t stream,
                              void* const* d_in, const int* in_sizes, int n_in,
                              void* d_out, int out_size,
                              const int* in_dtypes, int out_dtype,
                              void* const* d_out_all,
                              void* const* d_ws_all, size_t ws_bytes) {
    using namespace agk;
    if (nranks != NR || (unsigned)rank >= (unsigned)NR || ws_bytes < WS_NEED) { fprintf(stderr, "unsupported configuration\n"); return; }
    static bool attr_set[64] = {};
    int dev = 0; HIP_CHECK(hipGetDevice(&dev));
    if (dev >= 0 && dev < 64 && !attr_set[dev]) {
        HIP_CHECK(hipFuncSetAttribute((const void*)ag_gemm_kernel, hipFuncAttributeMaxDynamicSharedMemorySize, LDS_BYTES));
        attr_set[dev] = true;
    }
    Params p;
    p.x = (const char*)d_in[0]; p.w = (const char*)d_in[1]; p.out = (char*)d_out; p.rank = rank;
    for (int i = 0; i < NR; ++i) { p.ws[i] = (char*)d_ws_all[i]; p.outs[i] = (char*)d_out_all[i]; }
    hipLaunchKernelGGL(ag_gemm_kernel, dim3(NWG), dim3(NTHREADS), LDS_BYTES, stream, p);
    HIP_CHECK(hipGetLastError());
}
```

```cpp
#include <hip/hip_runtime.h>
#include <rccl/rccl.h>
#include <cstdio>

#define HIP_CHECK(expr) do { hipError_t _e = (expr); if (_e != hipSuccess) { \
    fprintf(stderr, "HIP error %s at %s:%d\n", hipGetErrorString(_e), __FILE__, __LINE__); \
    return; } } while (0)

namespace agk {

#define LAS __attribute__((address_space(3)))
typedef unsigned short u16;
typedef short s16x4 __attribute__((ext_vector_type(4)));
typedef short s16x8 __attribute__((ext_vector_type(8)));
typedef __bf16 bf16x8 __attribute__((ext_vector_type(8)));
typedef float f32x4 __attribute__((ext_vector_type(4)));
typedef unsigned u32x4 __attribute__((ext_vector_type(4)));
typedef unsigned u32x2 __attribute__((ext_vector_type(2)));

constexpr int NR = 8;
constexpr int KDIM = 8192;
constexpr unsigned XROW = KDIM * 2;
constexpr int NT = KDIM / 64;
constexpr int NCH = KDIM / 128;
constexpr int NTHREADS = 512;
constexpr int NWG = 256;
#define NCOPY 56
#define AWP 3
#define NPUB 2
#define HS 4
#define NPW 147
#define NPUBW 112
#define NPUSH 1
#define RPF 6
constexpr int NDIR1 = 56 - 4 * RPF;
constexpr int FOR0 = NCOPY;
constexpr int OWN0 = FOR0 + 112;
constexpr int EXT0 = OWN0 + 56;
constexpr int NGEMM = NWG - FOR0;
static_assert(NCOPY % 8 == 0 && EXT0 <= NWG && NPW <= NCOPY * AWP && NPUBW <= NCOPY * NPUB && AWP + NPUB <= 8, "roles");
constexpr int NG2 = 128;
constexpr int LDS_BYTES = 3 * (256 * 128 + 64 * 256);
constexpr unsigned long long DEADLINE = 50000000ull;

constexpr size_t WS_CTR = 0;
constexpr size_t WS_TMO = 0x4000;
constexpr size_t WS_Q = 0x4100;
constexpr size_t WS_FA = 0x10000;
constexpr size_t WS_FB = 0x18000;
constexpr size_t WS_AU = 0x1A000;
constexpr size_t WS_AB = 0x1E000;
constexpr size_t WS_DBG = 0x20000;
constexpr size_t WS_AA = 0x28000;
constexpr size_t WS_DATA = 1u << 20;
constexpr size_t UNIT = 65536;
constexpr size_t D_XR = 0;
constexpr size_t D_WR = D_XR + (size_t)12 * NCH * UNIT;
constexpr size_t D_PX = D_WR + (size_t)4 * NCH * UNIT;
constexpr size_t D_PW = D_PX + (size_t)4 * NCH * UNIT;
constexpr size_t D_STG = D_PW + (size_t)4 * NCH * UNIT;
constexpr size_t D_RB = D_STG + (size_t)128 * UNIT;
constexpr size_t PAR_BYTES = D_RB + (size_t)8 * 14 * UNIT;
constexpr size_t WS_NEED = WS_DATA + 2 * PAR_BYTES;
static_assert(RPF >= 0 && RPF <= 14, "relay count");

struct Params {
    const char* x; const char* w; char* out;
    char* ws[NR]; char* outs[NR];
    int rank;
};

__device__ __forceinline__ void st_sys(unsigned* p, unsigned v) { __hip_atomic_store(p, v, __ATOMIC_RELAXED, __HIP_MEMORY_SCOPE_SYSTEM); }
__device__ __forceinline__ unsigned ld_agent(const unsigned* p) { return __hip_atomic_load(p, __ATOMIC_RELAXED, __HIP_MEMORY_SCOPE_AGENT); }
__device__ __forceinline__ void st_agent(unsigned* p, unsigned v) { __hip_atomic_store(p, v, __ATOMIC_RELAXED, __HIP_MEMORY_SCOPE_AGENT); }
__device__ __forceinline__ bool geq(unsigned v, unsigned call) { return (int)(v - call) >= 0; }
__device__ __forceinline__ int opaque_tid() { int t = threadIdx.x; asm volatile("" : "+v"(t)); return t; }
__device__ __forceinline__ bool thread0() { return opaque_tid() == 0; }
__device__ __forceinline__ bool lane0() { return (opaque_tid() & 63) == 0; }
#define WAIT_V_(n) asm volatile("s_waitcnt vmcnt(" #n ")" ::: "memory")
#define WAIT_V(n) WAIT_V_(n)
#define WAIT_L(n) asm volatile("s_waitcnt lgkmcnt(" #n ")" ::: "memory")
#define BAR __builtin_amdgcn_s_barrier()
#define SCHED __builtin_amdgcn_sched_barrier(0)
#define STAMP(wsl, i) do { } while (0)

__device__ __forceinline__ void st16_sys(void* p, u32x4 v) {
    asm volatile("global_store_dwordx4 %0, %1, off sc0 sc1\n\ts_nop 1" :: "v"(p), "v"(v) : "memory");
}
__device__ __forceinline__ unsigned long long poll_once(const unsigned* p, bool need, unsigned call) {
    unsigned v;
    asm volatile("global_load_dword %0, %1, off sc0 sc1\n\ts_waitcnt vmcnt(0)" : "=v"(v) : "v"(p) : "memory");
    return __ballot(!need || geq(v, call));
}
__device__ __forceinline__ bool timed_out(const unsigned* tmo, unsigned call) {
    unsigned tv;
    asm volatile("global_load_dword %0, %1, off sc1\n\ts_waitcnt vmcnt(0)" : "=v"(tv) : "v"(tmo) : "memory");
    return __builtin_amdgcn_readfirstlane(tv) == call;
}
__device__ __forceinline__ bool spin_check(unsigned* tmo, unsigned call, unsigned long long t0, unsigned& spins) {
    if ((++spins & 7u) == 0u) {
        if (timed_out(tmo, call)) return true;
        if (__builtin_amdgcn_s_memrealtime() - t0 > DEADLINE) { st_agent(tmo, call); return true; }
    }
    __builtin_amdgcn_s_sleep(1);
    return false;
}
__device__ __forceinline__ void wait_all(const unsigned* p, bool need, unsigned call, unsigned* tmo) {
    if (poll_once(p, need, call) == ~0ull) return;
    const unsigned long long t0 = __builtin_amdgcn_s_memrealtime();
    unsigned spins = 7;
    for (;;) {
        if (spin_check(tmo, call, t0, spins)) return;
        if (poll_once(p, need, call) == ~0ull) return;
    }
}

__host__ __device__ __forceinline__ int lds_byte(int r, int c) { const int st = (r >> 4) * 2 + (c >> 5), rr = r & 15, cc = c & 31, ob = rr * 64 + cc * 2; return st * 1024 + (ob ^ (((ob >> 9) & 1) << 5)); }
__host__ __device__ __forceinline__ void stage_rc(int b, int& R, int& C) { const int st = b / 1024, sb = b % 1024, swz = sb ^ (((sb >> 9) & 1) << 5); R = (st >> 1) * 16 + swz / 64; C = (st & 1) * 32 + (swz % 64) / 2; }
__device__ __forceinline__ unsigned cvt_pk_bf16(float lo, float hi) { unsigned r; asm volatile("v_cvt_pk_bf16_f32 %0, %1, %2" : "=v"(r) : "v"(lo), "v"(hi)); return r; }
__device__ __forceinline__ float gelu_tanh(float y) {
    const float k1 = -2.0f * 0.7978845608028654f * 1.4426950408889634f;
    const float k2 = k1 * 0.044715f;
    const float t = y * (k2 * (y * y) + k1);
    return y * __builtin_amdgcn_rcpf(1.0f + __builtin_amdgcn_exp2f(t));
}
__device__ __forceinline__ u32x2 gelu_pack(f32x4 v, unsigned bad) {
    u32x2 w; w.x = cvt_pk_bf16(gelu_tanh(v[0]), gelu_tanh(v[1])); w.y = cvt_pk_bf16(gelu_tanh(v[2]), gelu_tanh(v[3]));
    if (bad) { w.x = 0x7fc07fc0u; w.y = 0x7fc07fc0u; }
    return w;
}

struct Tile {
    const char* A; const char* B; char* O;
    unsigned lda, csA;
    unsigned ldb;
    const unsigned* fA; const unsigned* fB;
    unsigned* fO; int nfO;
};

__device__ __forceinline__ unsigned poll_chunks(const unsigned* p, bool need, unsigned call) {
    u32x4 v;
    asm volatile("global_load_dwordx4 %0, %1, off sc0 sc1\n\ts_waitcnt vmcnt(0)" : "=v"(v) : "v"(p) : "memory");
    const unsigned long long mk = __ballot(!need || (geq(v.x, call) && geq(v.y, call) && geq(v.z, call) && geq(v.w, call)));
    const unsigned both = (unsigned)mk & (unsigned)(mk >> 32);
    return both & (both >> 1) & 0x55555555u;
}
__device__ __forceinline__ void ensure2(const unsigned* fA, const unsigned* fB, int c, int& ready, unsigned call, unsigned* tmo, int lane) {
    if (c >= ready) {
        const int cc = c + ((lane >> 1) & 15);
        const unsigned* base = lane < 32 ? fA : fB;
        const bool need = base != nullptr && cc < NCH;
        const unsigned* p = need ? base + cc * 8 + (lane & 1) * 4 : (const unsigned*)((const char*)tmo - WS_TMO + WS_FA);
        unsigned m = poll_chunks(p, need, call);
        if (!(m & 1u)) {
            const unsigned long long t0 = __builtin_amdgcn_s_memrealtime();
            unsigned spins = 7;
            for (;;) {
                if (spin_check(tmo, call, t0, spins)) { m = 0x55555555u; break; }
                m = poll_chunks(p, need, call);
                if (m & 1u) break;
            }
        }
        const unsigned miss = ~m & 0x55555555u;
        ready = c + (miss ? (__builtin_ctz(miss) >> 1) : 16);
    }
}
__device__ __forceinline__ void ensure(const Tile& tl, int c, int& ready, unsigned call, unsigned* tmo, int lane) { ensure2(tl.fA, tl.fB, c, ready, call, tmo, lane); }
#define RD128(dst, addr, off) asm volatile("ds_read_b128 %0, %1 offset:%2" : "=v"(dst) : "v"(addr), "i"(off) : "memory")
#define TRRD(dst, addr, off) asm volatile("ds_read_b64_tr_b16 %0, %1 offset:%2" : "=v"(dst) : "v"(addr), "i"(off) : "memory")
#define GLDS(g, l) __builtin_amdgcn_global_load_lds((const unsigned*)(g), (LAS unsigned*)(l), 16, 0, 0)

template <int N> __device__ __forceinline__ void wait_v() { asm volatile("s_waitcnt vmcnt(%0)" :: "n"(N) : "memory"); }
template <int N> __device__ __forceinline__ void wait_l() { asm volatile("s_waitcnt lgkmcnt(%0)" :: "n"(N > 15 ? 15 : N) : "memory"); }
template <int BM, int BN, int KT, int STG>
__device__ __forceinline__ void gemm_tile(LAS unsigned char* lds, const Tile& tl, const unsigned call, unsigned* tmo, int sbase) {
    char* const wsl_dbg = (char*)tmo - WS_TMO; (void)wsl_dbg; (void)sbase;
    constexpr int WAVES_M = BM == 256 ? 4 : 2, WAVES_N = 8 / WAVES_M;
    constexpr int MF = BM / WAVES_M / 16, NF = BN / WAVES_N / 16;
    constexpr int BROW = BN * 2, ABYTES = BM * 128, BBYTES = 64 * BROW, TBYTES = ABYTES + BBYTES, SBYTES = KT * TBYTES;
    constexpr int RA = ABYTES / 8192, RB = BBYTES / 8192;
    constexpr int LPS = (RA + RB) * KT;
    constexpr int NS = NT / KT;
    constexpr int KS = 2 * KT, RPK = MF + 2 * NF;
    static_assert(STG * SBYTES <= LDS_BYTES, "LDS");
    const int tid = threadIdx.x, wid = __builtin_amdgcn_readfirstlane(tid >> 6), lane = tid & 63;
    const int wr = wid / WAVES_N, wc = wid % WAVES_N, fr = lane & 15, fq = lane >> 4;
    unsigned voffA[RA], voffB[RB];
#pragma unroll
    for (int i = 0; i < RA; ++i) { int R, C; stage_rc(tid * 16 + i * 8192, R, C); voffA[i] = (unsigned)R * tl.lda + (unsigned)C * 2u; }
#pragma unroll
    for (int i = 0; i < RB; ++i) { const int b = tid * 16 + i * 8192;
        if (BN == 128) { const int row = b >> 8, chpos = (b >> 4) & 15, ch = chpos ^ (((row & 3) << 2) | ((row >> 2) & 3)); voffB[i] = (unsigned)row * tl.ldb + (unsigned)ch * 16u; }
        else { const int row = b >> 7, chpos = (b >> 4) & 7, ch = chpos ^ ((((row >> 1) & 1) | (((row >> 3) & 1) << 1)) << 1); voffB[i] = (unsigned)row * tl.ldb + (unsigned)ch * 16u; } }
    const size_t kstepB = (size_t)64 * tl.ldb;
    const unsigned ldsbase = (unsigned)(size_t)lds;
    const unsigned ldsw = (unsigned)wid * 1024u;
    const unsigned aoff = ldsbase + (unsigned)lds_byte(wr * (MF * 16) + fr, fq * 8);
    unsigned boff[NF][2];
    { const int g = lane >> 4, q = (lane & 15) >> 2, p = lane & 3;
#pragma unroll
      for (int n = 0; n < NF; ++n)
#pragma unroll
        for (int t = 0; t < 2; ++t) { const int row = 8 * g + 4 * t + q, ch = 2 * (wc * NF + n) + (p >> 1);
            const int sw = BN == 128 ? (((row & 3) << 2) | ((row >> 2) & 3)) : ((((row >> 1) & 1) | (((row >> 3) & 1) << 1)) << 1);
            boff[n][t] = ldsbase + (unsigned)(ABYTES + BROW * row + 16 * (ch ^ sw) + 8 * (p & 1)); } }

    f32x4 acc[MF][NF];
#pragma unroll
    for (int m = 0; m < MF; ++m)
#pragma unroll
        for (int n = 0; n < NF; ++n) acc[m][n] = (f32x4){0.f, 0.f, 0.f, 0.f};

    int ready = (tl.fA == nullptr && tl.fB == nullptr) ? NCH : 0;
#define STAGE_TILE(st, sb) do { _Pragma("unroll") for (int _j = 0; _j < KT; ++_j) { const int _kt = (st) * KT + _j; \
        const char* _a = tl.A + (size_t)(_kt >> 1) * tl.csA + (_kt & 1) * 128; const char* _b = tl.B + (size_t)_kt * kstepB; \
        _Pragma("unroll") for (int _i = 0; _i < RA; ++_i) GLDS(_a + voffA[_i], lds + (sb) + _j * TBYTES + ldsw + _i * 8192); \
        _Pragma("unroll") for (int _i = 0; _i < RB; ++_i) GLDS(_b + voffB[_i], lds + (sb) + _j * TBYTES + ABYTES + ldsw + _i * 8192); } } while (0)

    if (wid == 0) { for (int c = 0; c <= ((STG - 2) * KT) >> 1; ++c) ensure(tl, c, ready, call, tmo, lane); }
    BAR;
    STAMP(wsl_dbg, sbase);
#pragma unroll
    for (int st = 0; st < STG - 1; ++st) STAGE_TILE(st, st * SBYTES);
    unsigned sb_rd = 0, sb_wr = (STG - 1) * SBYTES;
    for (int t = 0; t < NS; ++t) {
        const int tn = t + STG - 1 < NS ? t + STG - 1 : NS - 1;
        if (wid == 0 && (KT == 2 || (tn & 1) == 0)) ensure(tl, (tn * KT) >> 1, ready, call, tmo, lane);
        wait_v<LPS * (STG - 2)>();
        BAR;
        STAGE_TILE(tn, sb_wr);
        s16x8 Af[KS][MF]; s16x4 Bl[KS][NF], Bh[KS][NF];
        const unsigned a = aoff + sb_rd;
#pragma unroll
        for (int k = 0; k < KS; ++k) {
#pragma unroll
            for (int m = 0; m < MF; ++m) RD128(Af[k][m], a, (k >> 1) * TBYTES + m * 2048 + (k & 1) * 1024);
#pragma unroll
            for (int n = 0; n < NF; ++n) { TRRD(Bl[k][n], boff[n][0] + sb_rd, (k >> 1) * TBYTES + (k & 1) * 32 * BROW); TRRD(Bh[k][n], boff[n][1] + sb_rd, (k >> 1) * TBYTES + (k & 1) * 32 * BROW); }
        }
#pragma unroll
        for (int k = 0; k < KS; ++k) {
            if (k == 0) wait_l<(KS - 1) * RPK>(); else if (k == 1) wait_l<(KS - 2) * RPK>(); else if (k == 2) wait_l<(KS > 3 ? KS - 3 : 0) * RPK>(); else wait_l<0>();
            SCHED;
            __builtin_amdgcn_s_setprio(1);
#pragma unroll
            for (int m = 0; m < MF; ++m)
#pragma unroll
                for (int n = 0; n < NF; ++n) {
                    const s16x8 bf = __builtin_shufflevector(Bl[k][n], Bh[k][n], 0, 1, 2, 3, 4, 5, 6, 7);
                    acc[m][n] = __builtin_amdgcn_mfma_f32_16x16x32_bf16(__builtin_bit_cast(bf16x8, bf), __builtin_bit_cast(bf16x8, Af[k][m]), acc[m][n], 0, 0, 0);
                }
            __builtin_amdgcn_s_setprio(0);
            SCHED;
        }
        sb_rd = sb_rd == (STG - 1) * SBYTES ? 0 : sb_rd + SBYTES;
        sb_wr = sb_wr == (STG - 1) * SBYTES ? 0 : sb_wr + SBYTES;
    }
#undef STAGE_TILE
    WAIT_V(0);
    BAR;
    STAMP(wsl_dbg, sbase + 1);
    unsigned bad = 0;
    if (tl.fA != nullptr || tl.fB != nullptr) bad = timed_out(tmo, call);
    const int tz = opaque_tid();
    const int lane2 = tz & 63, fr2 = lane2 & 15, fq2 = lane2 >> 4;
    constexpr int CPR = BN / 8;
#pragma unroll
    for (int m = 0; m < MF; ++m) { const int row = wr * (MF * 16) + m * 16 + fr2;
#pragma unroll
        for (int n = 0; n < NF; ++n) {
            const int col = wc * (NF * 16) + n * 16 + fq2 * 4; const int ch = col >> 3, hf = (col >> 2) & 1;
            *(LAS u32x2*)(lds + row * BROW + ((ch ^ (fr2 & (CPR - 1))) << 4) + hf * 8) = gelu_pack(acc[m][n], bad); } }
    WAIT_L(0); BAR;
    constexpr int RPW = BM / 8, RPI = 64 / CPR;
    if (tl.fO != nullptr) {
#pragma unroll 4
        for (int i = 0; i < RPW / RPI; ++i) { const int row = wid * RPW + i * RPI + lane2 / CPR, chp = lane2 & (CPR - 1);
            const u32x4 v = *(const LAS u32x4*)(lds + row * BROW + chp * 16);
            st16_sys(tl.O + (size_t)row * 256 + ((chp ^ (row & (CPR - 1))) << 4), v); }
        WAIT_V(0);
        BAR;
        const int tf = opaque_tid();
        if (tf < tl.nfO) st_sys(tl.fO + tf, call);
    } else {
#pragma unroll 4
        for (int i = 0; i < RPW / RPI; ++i) { const int row = wid * RPW + i * RPI + lane2 / CPR, chp = lane2 & (CPR - 1);
            const u32x4 v = *(const LAS u32x4*)(lds + row * BROW + chp * 16);
            *(u32x4*)(tl.O + (size_t)row * 2048 + ((chp ^ (row & (CPR - 1))) << 4)) = v; }
        WAIT_L(0);
        BAR;
    }
    STAMP(wsl_dbg, sbase + 2);
}

__device__ __forceinline__ void gemm_tile_256(LAS unsigned char* lds, const Tile& tl, const unsigned call, unsigned* tmo) {
    char* const wsl_dbg = (char*)tmo - WS_TMO; (void)wsl_dbg;
    constexpr int HALF = 128, HTB = HALF * 64 * 2;
    const int tid = threadIdx.x, wid = __builtin_amdgcn_readfirstlane(tid >> 6), lane = tid & 63, wr = wid >> 2, wc = wid & 3, fr = lane & 15, fq = lane >> 4;
    unsigned voffA[2], voffB[2];
#pragma unroll
    for (int i = 0; i < 2; ++i) { int R, C; stage_rc(tid * 16 + i * 8192, R, C); voffA[i] = (unsigned)R * tl.lda + (unsigned)C * 2u;
        const int b = tid * 16 + i * 8192, row = b >> 8, chpos = (b >> 4) & 15, ch = chpos ^ (((row & 3) << 2) | ((row >> 2) & 3));
        voffB[i] = (unsigned)row * tl.ldb + (unsigned)ch * 16u; }
    const size_t kstepB = (size_t)64 * tl.ldb;
    const size_t hstepA = (size_t)HALF * tl.lda, hstepB = 256;
    const unsigned ldsw = (unsigned)wid * 1024u;
    const unsigned ldsbase = (unsigned)(size_t)lds;
    const unsigned aoff = ldsbase + (unsigned)lds_byte(wr * 64 + fr, fq * 8);
    unsigned boff[2][2];
    { const int g = lane >> 4, q = (lane & 15) >> 2, p = lane & 3;
#pragma unroll
      for (int n = 0; n < 2; ++n)
#pragma unroll
        for (int t = 0; t < 2; ++t) { const int row = 8 * g + 4 * t + q, ch = 4 * wc + 2 * n + (p >> 1);
            boff[n][t] = ldsbase + 4 * HTB + 256 * row + 16 * (ch ^ (((row & 3) << 2) | ((row >> 2) & 3))) + 8 * (p & 1); } }
#define SA(b, h) (((b) * 2 + (h)) * HTB)
#define SB(b, h) ((4 + (b) * 2 + (h)) * HTB)
#define STAGE(bufoff, gbase, voff) do { _Pragma("unroll") for (int _i = 0; _i < 2; ++_i) GLDS((const char*)(gbase) + (voff)[_i], lds + (bufoff) + ldsw + _i * 8192); } while (0)
#define LDA(dst, b, h) do { _Pragma("unroll") for (int m = 0; m < 4; ++m) _Pragma("unroll") for (int k = 0; k < 2; ++k) RD128(dst[m][k], aoff, SA(b, h) + m * 2048 + k * 1024); } while (0)
#define LDB(dst, b, h) do { _Pragma("unroll") for (int n = 0; n < 2; ++n) _Pragma("unroll") for (int k = 0; k < 2; ++k) { \
        s16x4 lo, hi; TRRD(lo, boff[n][0], ((b) * 2 + (h)) * HTB + k * 8192); TRRD(hi, boff[n][1], ((b) * 2 + (h)) * HTB + k * 8192); \
        dst[n][k] = __builtin_shufflevector(lo, hi, 0, 1, 2, 3, 4, 5, 6, 7); } } while (0)
#define MMA(ai, bj, At, Bt) do { __builtin_amdgcn_s_setprio(1); _Pragma("unroll") for (int m = 0; m < 4; ++m) _Pragma("unroll") for (int n = 0; n < 2; ++n) _Pragma("unroll") for (int k = 0; k < 2; ++k) \
        acc[ai][bj][m][n] = __builtin_amdgcn_mfma_f32_16x16x32_bf16(__builtin_bit_cast(bf16x8, Bt[n][k]), __builtin_bit_cast(bf16x8, At[m][k]), acc[ai][bj][m][n], 0, 0, 0); __builtin_amdgcn_s_setprio(0); } while (0)
#define KOFFA(t) ((size_t)((t) >> 1) * tl.csA + ((t) & 1) * 128)
    f32x4 acc[2][2][4][2];
#pragma unroll
    for (int a = 0; a < 2; ++a)
#pragma unroll
        for (int b = 0; b < 2; ++b)
#pragma unroll
            for (int m = 0; m < 4; ++m)
#pragma unroll
                for (int n = 0; n < 2; ++n) acc[a][b][m][n] = (f32x4){0.f, 0.f, 0.f, 0.f};
    s16x8 At[4][2], B0[2][2], B1[2][2];
    const char* cA = tl.A; const char* cB = tl.B;
    int ready = tl.fA == nullptr ? NCH : 0;
    if (wid == 0) ensure(tl, 0, ready, call, tmo, lane);
    BAR;
    STAMP(wsl_dbg, 1);
    STAGE(SB(0, 0), cB, voffB); STAGE(SA(0, 0), cA, voffA); STAGE(SB(0, 1), cB + hstepB, voffB); STAGE(SA(0, 1), cA + hstepA, voffA);
    if (wr == 1) BAR;
    WAIT_V(4); BAR;
    STAGE(SB(1, 0), cB + kstepB, voffB); STAGE(SA(1, 0), cA + 128, voffA); STAGE(SB(1, 1), cB + hstepB + kstepB, voffB);
    WAIT_V(6); BAR;
    for (int t = 0; t < NT; t += 2) {
        const int t2 = (t == NT - 2) ? 0 : t + 2;
        if (wid == 0) ensure(tl, t2 >> 1, ready, call, tmo, lane);
        const char* a1 = cA + KOFFA(t + 1);
        const char* a2 = cA + KOFFA(t2); const char* b2 = cB + (size_t)t2 * kstepB;
        const char* a3 = a2 + 128; const char* b3 = b2 + kstepB;
        LDB(B0, 0, 0); SCHED; LDA(At, 0, 0); STAGE(SA(1, 1), a1 + hstepA, voffA);
        WAIT_L(8); BAR; WAIT_L(0); SCHED; MMA(0, 0, At, B0); BAR; SCHED;
        LDB(B1, 0, 1); STAGE(SB(0, 0), b2, voffB);
        BAR; WAIT_L(0); SCHED; MMA(0, 1, At, B1); BAR; SCHED;
        LDA(At, 0, 1); STAGE(SA(0, 0), a2, voffA);
        BAR; WAIT_L(0); SCHED; MMA(1, 0, At, B0); BAR; SCHED;
        STAGE(SB(0, 1), b2 + hstepB, voffB);
        WAIT_V(6); BAR; MMA(1, 1, At, B1); BAR; SCHED;
        LDB(B0, 1, 0); SCHED; LDA(At, 1, 0); STAGE(SA(0, 1), a2 + hstepA, voffA);
        WAIT_L(8); BAR; WAIT_L(0); SCHED; MMA(0, 0, At, B0); BAR; SCHED;
        LDB(B1, 1, 1); STAGE(SB(1, 0), b3, voffB);
        BAR; WAIT_L(0); SCHED; MMA(0, 1, At, B1); BAR; SCHED;
        LDA(At, 1, 1); STAGE(SA(1, 0), a3, voffA);
        BAR; WAIT_L(0); SCHED; MMA(1, 0, At, B0); BAR; SCHED;
        STAGE(SB(1, 1), b3 + hstepB, voffB);
        WAIT_V(6); BAR; MMA(1, 1, At, B1); BAR; SCHED;
    }
    WAIT_V(0);
    if (wr == 0) BAR;
    BAR;
    STAMP(wsl_dbg, 2);
    unsigned bad = 0;
    if (tl.fA != nullptr) bad = timed_out(tmo, call);
    const int tz = opaque_tid();
    const int lane2 = tz & 63, fr2 = lane2 & 15, fq2 = lane2 >> 4;
#pragma unroll
    for (int ai = 0; ai < 2; ++ai)
#pragma unroll
        for (int m = 0; m < 4; ++m) { const int row = ai * HALF + wr * 64 + m * 16 + fr2;
#pragma unroll
            for (int bj = 0; bj < 2; ++bj)
#pragma unroll
                for (int n = 0; n < 2; ++n) {
                    const int col = bj * HALF + wc * 32 + n * 16 + fq2 * 4; const int ch = col >> 3, hf = (col >> 2) & 1;
                    *(LAS u32x2*)(lds + row * 512 + ((ch ^ fr2) << 4) + hf * 8) = gelu_pack(acc[ai][bj][m][n], bad); } }
    WAIT_L(0); BAR;
#pragma unroll 4
    for (int i = 0; i < 16; ++i) { const int row = wid * 32 + i * 2 + (lane2 >> 5), chp = lane2 & 31;
        const u32x4 v = *(const LAS u32x4*)(lds + row * 512 + chp * 16);
        *(u32x4*)(tl.O + (size_t)row * 2048 + ((chp ^ (row & 15)) << 4)) = v; }
    WAIT_L(0);
    BAR;
    STAMP(wsl_dbg, 3);
#undef SA
#undef SB
#undef STAGE
#undef LDA
#undef LDB
#undef MMA
#undef KOFFA
}

__device__ __forceinline__ int block_row(int r, int rb) {
    if (rb < 4) return r * 1024 + rb * 256;
    if (rb < 8) return (r ^ 1) * 1024 + (rb - 4) * 256;
    const int j = (rb - 8) >> 1, xv = (j & 1) + 2 + (j >> 1) * 4;
    return (r ^ xv) * 1024 + (r >> 2) * 512 + (rb & 1) * 256;
}
__device__ __forceinline__ int g1_block(int g1) { return g1 < 2 ? g1 : (g1 == 2 ? 4 : (g1 < 7 ? g1 + 5 : (g1 < 9 ? g1 - 5 : (g1 == 9 ? 6 : g1 + 2)))); }
__device__ __forceinline__ int link_xor(int L) { return L == 0 ? 1 : (L < 5 ? ((L - 1) & 1) + 2 + ((L - 1) >> 1) * 4 : L - 1); }
__device__ __forceinline__ int xlink_index(int xv) { return (xv & 1) + ((xv >> 2) & 1) * 2; }
__device__ __forceinline__ Tile make_tile(const Params& p, char* wsl, size_t paroff, int rb, int sub, int cb, int ou) {
    Tile t; const int ro = sub < 0 ? 0 : sub * 64; char* par = wsl + paroff; const int r = p.rank;
    if (rb < 4) { t.A = p.x + (size_t)(rb * 256 + ro) * XROW; t.lda = XROW; t.csA = 256; t.fA = nullptr; }
    else { t.A = par + D_XR + (size_t)(rb - 4) * NCH * UNIT + ro * 256; t.lda = 256; t.csA = (unsigned)UNIT; t.fA = (const unsigned*)(wsl + WS_FA) + rb * NCH * 8; }
    if (cb < 8) {
        t.B = p.w + cb * 256; t.ldb = 2048; t.fB = nullptr;
        t.O = p.out + (size_t)(block_row(r, rb) + ro) * 2048 + cb * 256; t.fO = nullptr; t.nfO = 0;
    } else {
        const int h = (cb - 8) >> 2, cl = cb & 3;
        const int cp = h * 2 + (cl >> 1);
        t.B = par + D_WR + (size_t)cp * NCH * UNIT + (cl & 1) * 256; t.ldb = 512;
        t.fB = (const unsigned*)(wsl + WS_FB) + cp * NCH * 8;
        const int g = ou * 4 + cl, uo = h * 64 + g;
        t.O = par + D_STG + (size_t)uo * UNIT + ro * 256;
        t.nfO = sub < 0 ? 8 : 2;
        if (g < NDIR1 || g >= 56)
            t.fO = (unsigned*)(p.ws[r ^ (4 + h)] + WS_AU) + ((5 + h) * 64 + (g < NDIR1 ? g : g - 4 * RPF)) * 8 + (sub < 0 ? 0 : sub * 2);
        else { const int jx = (g - NDIR1) & 3, k = (g - NDIR1) >> 2;
            t.fO = (unsigned*)(p.ws[r ^ link_xor(jx + 1)] + WS_AU) + ((jx + 1) * 64 + (26 - (r & 1)) + k * 2 + h) * 8; }
    }
    return t;
}

struct Slice { const char* src; char* dst; unsigned sstride, dstride; int sh; int dep; unsigned* f0; unsigned* f1; unsigned* f2; };

constexpr int NS1 = 2 * NCH;
__device__ __forceinline__ int link_late(int r, int L) { return L == 0 ? 26 : 26 - ((r ^ link_xor(L)) & 1); }
__device__ __forceinline__ int link_total(int r, int L) { return 8 * (L == 7 ? 512 : (L >= 5 ? NS1 + NDIR1 + 8 : NS1 + link_late(r, L) + (L == 0 ? 0 : 4 * RPF))); }
__device__ __forceinline__ void make_slice(const Params& p, char* wsl, size_t paroff, int L, int tk, Slice& u) {
    const int r = p.rank;
    char* par = wsl + paroff;
    const int un = tk >> 3, sl = tk & 7;
    unsigned* const fA = (unsigned*)(wsl + WS_FA);
    u.dep = -1; u.sh = 4; u.sstride = 256; u.dstride = 256; u.f0 = nullptr; u.f1 = nullptr; u.f2 = nullptr;
    if (L == 7) {
        const int c = ((un >> 3) + NPUSH) & (NCH - 1), b = un & 7;
        if (b < 4) {
            u.src = p.x + (size_t)(b * 256 + sl * 32) * XROW + c * 256; u.sstride = XROW;
            const int hi = ((r >> 2) ^ (b >> 1)) & 1;
            const int a1 = r ^ (2 + 4 * hi), a2 = r ^ (3 + 4 * hi), rb1 = 8 + 4 * hi + (b & 1), rb2 = rb1 + 2;
            unsigned* mate_late = (unsigned*)(p.ws[r ^ 1] + WS_AU) + (2 * (c / 5) + (b >> 1)) * 8 + sl;
            u.dst = par + D_PX + ((size_t)b * NCH + c) * UNIT + sl * 8192;
            if ((b & 1) && c % 5 == 0) u.f0 = mate_late;
            if (c >= NPUSH) {
                if (!(b & 1)) u.f0 = (unsigned*)(p.ws[r ^ 1] + WS_AA) + ((4 + b) * NCH + c) * 8 + sl;
                u.f1 = (unsigned*)(p.ws[a1] + WS_AA) + (rb1 * NCH + c) * 8 + sl;
                u.f2 = (unsigned*)(p.ws[a2] + WS_AA) + (rb2 * NCH + c) * 8 + sl;
            }
        } else {
            const int cp = b - 4, J = r ^ (4 + (cp >> 1));
            u.src = p.w + (size_t)(c * 128 + sl * 16) * 2048 + cp * 512; u.sstride = 2048; u.dstride = 512; u.sh = 5;
            u.dst = par + D_PW + ((size_t)cp * NCH + c) * UNIT + sl * 8192;
            if (c >= NPUSH) u.f0 = (unsigned*)(p.ws[J] + WS_AB) + (cp * NCH + c) * 8 + sl;
        }
        return;
    }
    const int xv = link_xor(L);
    const int peer = r ^ xv;
    const char* ppar = p.ws[peer] + paroff;
    if (un < 2 * NPUSH) {
        const int c = un >> 1, b = un & 1;
        char* pp = p.ws[peer] + paroff;
        if (L < 5) {
            const int blk = L == 0 ? 2 * b : 2 * (peer >> 2) + b;
            const int rb = L == 0 ? 4 + 2 * b : 8 + 2 * (L - 1) + b;
            u.src = p.x + (size_t)(blk * 256 + sl * 32) * XROW + c * 256; u.sstride = XROW;
            u.dst = pp + D_XR + ((size_t)(rb - 4) * NCH + c) * UNIT + sl * 8192;
            u.f0 = (unsigned*)(p.ws[peer] + WS_FA) + (rb * NCH + c) * 8 + sl;
            if (L != 0 && b == 1) { const int c5 = c % 5 - 1 - (peer & 1);
                if (c5 == 0 || c5 == 2) u.f1 = (unsigned*)(p.ws[r ^ 1] + WS_AU) + ((1 + xlink_index(r ^ 1 ^ peer)) * 64 + 2 * (c / 5) + (c5 >> 1)) * 8 + sl; }
        } else {
            const int cp = (L - 5) * 2 + b;
            u.src = p.w + (size_t)(c * 128 + sl * 16) * 2048 + cp * 512; u.sstride = 2048; u.dstride = 512; u.sh = 5;
            u.dst = pp + D_WR + ((size_t)cp * NCH + c) * UNIT + sl * 8192;
            u.f0 = (unsigned*)(p.ws[peer] + WS_FB) + (cp * NCH + c) * 8 + sl;
        }
        return;
    }
    if (un < NS1) {
        const int c = un >> 1, b = un & 1;
        if (L < 5) {
            const int blk = L == 0 ? 2 * b : 2 * (r >> 2) + b;
            const int rb = L == 0 ? 4 + 2 * b : 8 + 2 * (L - 1) + b;
            u.src = ppar + D_PX + ((size_t)blk * NCH + c) * UNIT + sl * 8192;
            u.dst = par + D_XR + ((size_t)(rb - 4) * NCH + c) * UNIT + sl * 8192;
            u.f0 = fA + (rb * NCH + c) * 8 + sl;
            if (L != 0 && b == 1) { const int c5 = c % 5 - 1 - (r & 1);
                if (c5 == 0 || c5 == 2) u.f1 = (unsigned*)(p.ws[peer ^ 1] + WS_AU) + ((1 + xlink_index(peer ^ 1 ^ r)) * 64 + 2 * (c / 5) + (c5 >> 1)) * 8 + sl; }
        } else {
            const int cp = (L - 5) * 2 + b;
            u.src = ppar + D_PW + ((size_t)cp * NCH + c) * UNIT + sl * 8192;
            u.dst = par + D_WR + ((size_t)cp * NCH + c) * UNIT + sl * 8192;
            u.f0 = (unsigned*)(wsl + WS_FB) + (cp * NCH + c) * 8 + sl;
        }
        return;
    }
    int m = un - NS1;
    u.dep = m;
    if (L >= 5) {
        const int h = L - 5, g = m < NDIR1 ? m : m + 4 * RPF, ou = g >> 2, cl = g & 3, uo = h * 64 + g;
        const int rb = ou < 14 ? g1_block(ou) : 5 + 2 * (ou - 14);
        u.src = ppar + D_STG + (size_t)uo * UNIT + sl * 8192;
        u.dst = p.out + (size_t)(block_row(peer, rb) + sl * 32) * 2048 + h * 1024 + cl * 256; u.dstride = 2048;
        return;
    }
    const int nlate = link_late(r, L);
    if (m < nlate) {
        int c, blk;
        if (L == 0) { c = (m >> 1) * 5; blk = 1 + 2 * (m & 1); u.src = ppar + D_PX + ((size_t)blk * NCH + c) * UNIT + sl * 8192; }
        else { c = (m >> 1) * 5 + 1 + (peer & 1) + 2 * (m & 1); blk = 2 * (peer >> 2) + 1;
            const int rbl = 8 + 2 * xlink_index(peer ^ r ^ 1) + 1;
            u.src = ppar + D_XR + ((size_t)(rbl - 4) * NCH + c) * UNIT + sl * 8192; }
        u.dst = par + D_XR + ((size_t)blk * NCH + c) * UNIT + sl * 8192;
        u.f0 = fA + ((4 + blk) * NCH + c) * 8 + sl;
        return;
    }
    m -= nlate;
    const int h = m & 1;
    if (m < 2 * RPF) {
        const int k = m >> 1, g = NDIR1 + k * 4 + (L - 1), uo = h * 64 + g, slot = ((L - 1) * 2 + h) * RPF + k, J = peer ^ (4 + h);
        u.src = ppar + D_STG + (size_t)uo * UNIT + sl * 8192;
        u.dst = par + D_RB + (size_t)slot * UNIT + sl * 8192;
        u.f0 = (unsigned*)(p.ws[J] + WS_AU) + ((1 + xlink_index(J ^ r)) * 64 + (26 - (r & 1)) + 2 * RPF + m) * 8 + sl;
    } else {
        const int k = (m - 2 * RPF) >> 1, rs = r ^ (4 + h), jx = xlink_index(rs ^ peer);
        const int g = NDIR1 + k * 4 + jx, ou = g >> 2, cl = g & 3, slot = (jx * 2 + h) * RPF + k;
        u.src = ppar + D_RB + (size_t)slot * UNIT + sl * 8192;
        u.dst = p.out + (size_t)(block_row(rs, g1_block(ou)) + sl * 32) * 2048 + h * 1024 + cl * 256; u.dstride = 2048;
    }
}
template <int I0, int I1>
__device__ __forceinline__ void slice_load(const Slice& u, u32x4 (&v)[8], int lane) {
    const char* s = u.src + (size_t)(lane >> u.sh) * u.sstride + (lane & ((1 << u.sh) - 1)) * 16;
    const size_t st = (size_t)(64 >> u.sh) * u.sstride;
#pragma unroll
    for (int i = I0; i < I1; ++i) asm volatile("global_load_dwordx4 %0, %1, off sc0 sc1" : "=v"(v[i]) : "v"(s + i * st) : "memory");
}
__device__ __forceinline__ void slice_store(const Slice& u, u32x4 (&v)[8], int lane) {
    char* d = u.dst + (size_t)(lane >> u.sh) * u.dstride + (lane & ((1 << u.sh) - 1)) * 16;
    const size_t st = (size_t)(64 >> u.sh) * u.dstride;
#pragma unroll
    for (int i = 0; i < 8; ++i) st16_sys(d + i * st, v[i]);
}
#define PIN8(v) asm volatile("" : "+v"(v[0]), "+v"(v[1]), "+v"(v[2]), "+v"(v[3]), "+v"(v[4]), "+v"(v[5]), "+v"(v[6]), "+v"(v[7]))

__device__ __forceinline__ void copy_wave(const Params& p, char* wsl, size_t paroff, int L, int j, int nw, unsigned call, unsigned* tmo) {
    const int lane = threadIdx.x & 63;
    const int total = link_total(p.rank, L);
    unsigned* ctr = (unsigned*)(wsl + WS_Q + 128 * L);
    const unsigned tbase = (call - 1u) * (unsigned)total - (unsigned)nw;
    u32x4 va[8], vb[8];
    Slice ua, ub;
    unsigned tkv = 0;
#define DRAW() do { if (lane0()) asm volatile("global_atomic_add %0, %1, %2, off sc0" : "=v"(tkv) : "v"(ctr), "v"(1u) : "memory"); } while (0)
#define DBG_PF true
#define PUBLISH(a, b, c) do { if (DBG_PF) { const int _l = opaque_tid() & 63; unsigned* _f = _l == 0 ? (a) : (_l == 1 ? (b) : (c)); if (_l < 3 && _f != nullptr) st_sys(_f, call); } } while (0)
    unsigned* p0 = nullptr; unsigned* p1 = nullptr; unsigned* p2 = nullptr;
    const unsigned* av0 = L < 5 ? (const unsigned*)(wsl + WS_AA) + (L == 0 ? 4 : 8 + 2 * (L - 1)) * NCH * 8 : (const unsigned*)(wsl + WS_AB) + (L - 5) * 2 * NCH * 8;
    const unsigned* av1 = av0 + (L == 0 ? 2 : 1) * NCH * 8;
    const unsigned* avU = (const unsigned*)(wsl + WS_AU) + L * 64 * 8;
    int aready = L == 7 ? NCH : 0, uready = 0;
    make_slice(p, wsl, paroff, L, j, ua);
    DRAW();
    if (L < 7 && j >= 16 * NPUSH) ensure2(av0, av1, j >> 4, aready, call, tmo, lane);
    slice_load<0, 8>(ua, va, lane);
#define STEP(cur, vcur, nxt, vnxt) { \
        WAIT_V(0); \
        PIN8(vcur); asm volatile("" : "+v"(tkv)); \
        PUBLISH(p0, p1, p2); \
        const int tk = (int)((unsigned)__builtin_amdgcn_readfirstlane(tkv) - tbase); \
        slice_store(cur, vcur, lane); \
        p0 = cur.f0; p1 = cur.f1; p2 = cur.f2; \
        if (tk >= total) break; \
        make_slice(p, wsl, paroff, L, tk, nxt); \
        const bool w1 = L < 7 && tk >= 16 * NPUSH && tk < NS1 * 8 && (tk >> 4) >= aready, w2 = nxt.dep >= uready; \
        if (w1 || w2) { \
            WAIT_V(0); PUBLISH(p0, p1, p2); p0 = nullptr; p1 = nullptr; p2 = nullptr; \
            if (w1) ensure2(av0, av1, tk >> 4, aready, call, tmo, lane); \
            else ensure2(avU, nullptr, nxt.dep, uready, call, tmo, lane); } \
        DRAW(); \
        if (HS == 8 || L == 7) slice_load<0, 8>(nxt, vnxt, lane); \
        else if (HS == 4) { slice_load<0, 4>(nxt, vnxt, lane); WAIT_V(0); slice_load<4, 8>(nxt, vnxt, lane); } \
        else { slice_load<0, 2>(nxt, vnxt, lane); WAIT_V(0); slice_load<2, 4>(nxt, vnxt, lane); WAIT_V(0); slice_load<4, 6>(nxt, vnxt, lane); WAIT_V(0); slice_load<6, 8>(nxt, vnxt, lane); } \
        if (L < 7 && tk >= NS1 * 8 - 24 && tk < NS1 * 8) STAMP(wsl, 2); \
        if (L < 5 && tk >= (NS1 + 25) * 8 - 24 && tk < (NS1 + 25) * 8) STAMP(wsl, 4); \
        if (L >= 1 && L < 5 && tk >= (NS1 + 26 + 2 * RPF) * 8 - 24 && tk < (NS1 + 26 + 2 * RPF) * 8) STAMP(wsl, 5); \
        if (L >= 5 && L < 7 && tk >= (NS1 + NDIR1) * 8 - 24 && tk < (NS1 + NDIR1) * 8) STAMP(wsl, 5); }
    for (;;) {
        STEP(ua, va, ub, vb)
        STEP(ub, vb, ua, va)
    }
#undef STEP
#undef DRAW
    WAIT_V(0);
    PUBLISH(p0, p1, p2);
#undef PUBLISH
    if (L < 7) STAMP(wsl, 3);
}

__global__ __launch_bounds__(NTHREADS, 2) void ag_gemm_kernel(const Params p) {
    extern __shared__ __attribute__((aligned(16))) unsigned char shm[];
    LAS unsigned char* lds = (LAS unsigned char*)shm;
    char* wsl = p.ws[p.rank];
    unsigned* tmo = (unsigned*)(wsl + WS_TMO);
    const int bid = blockIdx.x, tid = threadIdx.x;
    unsigned call;
    {
        unsigned* ctr = (unsigned*)(wsl + WS_CTR + (size_t)bid * 64);
        unsigned v;
        asm volatile("global_load_dword %0, %1, off sc1\n\ts_waitcnt vmcnt(0)" : "=v"(v) : "v"(ctr) : "memory");
        call = __builtin_amdgcn_readfirstlane(v) + 1u;
        if (call == 0u) call = 1u;
        __syncthreads();
        if (thread0()) st_agent(ctr, call);
    }
    const size_t paroff = WS_DATA + (size_t)(call & 1u) * PAR_BYTES;
    STAMP(wsl, 0);
    if (bid < FOR0) {
        const int wid = __builtin_amdgcn_readfirstlane(tid >> 6);
        if (wid < AWP) { const int gw = bid * AWP + wid, L = gw % 7; if (gw >= NPW) return; copy_wave(p, wsl, paroff, L, gw / 7, (NPW - L + 6) / 7, call, tmo);
            if (L >= 1 && timed_out(tmo, call) && lane0()) *(u16*)(p.out + (size_t)p.rank * 1024 * 2048) = 0x7fc0; }
        else if (wid < AWP + NPUB && bid * NPUB + wid - AWP < NPUBW) copy_wave(p, wsl, paroff, 7, bid * NPUB + wid - AWP, NPUBW, call, tmo);
        return;
    }
    if (bid < OWN0) {
        const int g = bid - FOR0, xcd = g & 7, slot = g >> 3;
        const int g1 = (xcd >> 2) * 7 + slot % 7, cb = 8 + (xcd & 3) * 2 + slot / 7;
        const Tile t = make_tile(p, wsl, paroff, g1_block(g1), -1, cb, g1);
        gemm_tile<256, 128, 1, 3>(lds, t, call, tmo, 1);
    } else if (bid < EXT0) {
        const int g = bid - OWN0, xcd = g & 7, slot = g >> 3;
        const int g1 = (xcd >> 2) * 7 + slot, cb = (xcd & 3) * 2;
        const Tile t = make_tile(p, wsl, paroff, g1_block(g1), -1, cb, g1);
        gemm_tile_256(lds, t, call, tmo);
    }
    const int xq = bid & 7;
    unsigned* q = (unsigned*)(wsl + WS_Q + 128 * (8 + xq));
    const unsigned qbase = (call - 1u) * (unsigned)(24 + NGEMM / 8);
    for (;;) {
        LAS unsigned* sw = (LAS unsigned*)(lds + LDS_BYTES - 16);
        if (thread0()) *sw = __hip_atomic_fetch_add(q, 1u, __ATOMIC_RELAXED, __HIP_MEMORY_SCOPE_AGENT) - qbase;
        __syncthreads();
        const unsigned tk = __builtin_amdgcn_readfirstlane(*sw);
        __syncthreads();
        if (tk >= 24u) break;
        if (tk >= 8u) {
            const int g2 = (tk - 8) >> 1, hf = tk & 1;
            Tile t = make_tile(p, wsl, paroff, 5 + 2 * (g2 >> 2), g2 & 3, 8 + xq, 14 + (g2 >> 2));
            t.B += hf * 128; t.O += hf * 128; t.fO += hf; t.nfO = 1;
            gemm_tile<64, 64, 2, 4>(lds, t, call, tmo, 4);
        } else {
            const Tile t = make_tile(p, wsl, paroff, 5 + 2 * (tk >> 2), tk & 3, xq, 14 + (tk >> 2));
            gemm_tile<64, 128, 2, 3>(lds, t, call, tmo, 9);
        }
    }
}

}

extern "C" void kernel_launch(int rank, int nranks, ncclComm_t comm,
                              hipStream_t stream,
                              void* const* d_in, const int* in_sizes, int n_in,
                              void* d_out, int out_size,
                              const int* in_dtypes, int out_dtype,
                              void* const* d_out_all,
                              void* const* d_ws_all, size_t ws_bytes) {
    using namespace agk;
    if (nranks != NR || (unsigned)rank >= (unsigned)NR || ws_bytes < WS_NEED) { fprintf(stderr, "unsupported configuration\n"); return; }
    static bool attr_set[64] = {};
    int dev = 0; HIP_CHECK(hipGetDevice(&dev));
    if (dev >= 0 && dev < 64 && !attr_set[dev]) {
        HIP_CHECK(hipFuncSetAttribute((const void*)ag_gemm_kernel, hipFuncAttributeMaxDynamicSharedMemorySize, LDS_BYTES));
        attr_set[dev] = true;
    }
    Params p;
    p.x = (const char*)d_in[0]; p.w = (const char*)d_in[1]; p.out = (char*)d_out; p.rank = rank;
    for (int i = 0; i < NR; ++i) { p.ws[i] = (char*)d_ws_all[i]; p.outs[i] = (char*)d_out_all[i]; }
    hipLaunchKernelGGL(ag_gemm_kernel, dim3(NWG), dim3(NTHREADS), LDS_BYTES, stream, p);
    HIP_CHECK(hipGetLastError());
}
```

```cpp
#include <hip/hip_runtime.h>
#include <rccl/rccl.h>
#include <cstdio>

#define HIP_CHECK(expr) do { hipError_t _e = (expr); if (_e != hipSuccess) { \
    fprintf(stderr, "HIP error %s at %s:%d\n", hipGetErrorString(_e), __FILE__, __LINE__); \
    return; } } while (0)

namespace agk {

#define LAS __attribute__((address_space(3)))
typedef unsigned short u16;
typedef short s16x4 __attribute__((ext_vector_type(4)));
typedef short s16x8 __attribute__((ext_vector_type(8)));
typedef __bf16 bf16x8 __attribute__((ext_vector_type(8)));
typedef float f32x4 __attribute__((ext_vector_type(4)));
typedef unsigned u32x4 __attribute__((ext_vector_type(4)));
typedef unsigned u32x2 __attribute__((ext_vector_type(2)));

constexpr int NR = 8;
constexpr int KDIM = 8192;
constexpr unsigned XROW = KDIM * 2;
constexpr int NT = KDIM / 64;
constexpr int NCH = KDIM / 128;
constexpr int NTHREADS = 512;
constexpr int NWG = 256;
#define NCOPY 56
#define AWP 3
#define NPUB 2
#define HS 4
#define NPW 147
#define NPUBW 112
#define NPUSH 1
#define RPF 6
constexpr int NDIR1 = 56 - 4 * RPF;
constexpr int FOR0 = NCOPY;
constexpr int OWN0 = FOR0 + 112;
constexpr int EXT0 = OWN0 + 56;
constexpr int NGEMM = NWG - FOR0;
static_assert(NCOPY % 8 == 0 && EXT0 <= NWG && NPW <= NCOPY * AWP && NPUBW <= NCOPY * NPUB && AWP + NPUB <= 8, "roles");
constexpr int LDS_BYTES = 3 * (256 * 128 + 64 * 256);
constexpr unsigned long long DEADLINE = 50000000ull;

constexpr size_t WS_CTR = 0;
constexpr size_t WS_TMO = 0x4000;
constexpr size_t WS_Q = 0x4100;
constexpr size_t WS_FA = 0x10000;
constexpr size_t WS_FB = 0x18000;
constexpr size_t WS_AU = 0x1A000;
constexpr size_t WS_AB = 0x1E000;
constexpr size_t WS_DBG = 0x20000;
constexpr size_t WS_AA = 0x28000;
constexpr size_t WS_DATA = 1u << 20;
constexpr size_t UNIT = 65536;
constexpr size_t D_XR = 0;
constexpr size_t D_WR = D_XR + (size_t)12 * NCH * UNIT;
constexpr size_t D_PX = D_WR + (size_t)4 * NCH * UNIT;
constexpr size_t D_PW = D_PX + (size_t)4 * NCH * UNIT;
constexpr size_t D_STG = D_PW + (size_t)4 * NCH * UNIT;
constexpr size_t D_RB = D_STG + (size_t)128 * UNIT;
constexpr size_t PAR_BYTES = D_RB + (size_t)8 * 14 * UNIT;
constexpr size_t WS_NEED = WS_DATA + 2 * PAR_BYTES;
static_assert(RPF >= 0 && RPF <= 14, "relay count");

struct Params {
    const char* x; const char* w; char* out;
    char* ws[NR]; char* outs[NR];
    int rank;
};

__device__ __forceinline__ void st_sys(unsigned* p, unsigned v) { __hip_atomic_store(p, v, __ATOMIC_RELAXED, __HIP_MEMORY_SCOPE_SYSTEM); }
__device__ __forceinline__ void st_agent(unsigned* p, unsigned v) { __hip_atomic_store(p, v, __ATOMIC_RELAXED, __HIP_MEMORY_SCOPE_AGENT); }
__device__ __forceinline__ bool geq(unsigned v, unsigned call) { return (int)(v - call) >= 0; }
__device__ __forceinline__ int opaque_tid() { int t = threadIdx.x; asm volatile("" : "+v"(t)); return t; }
__device__ __forceinline__ bool thread0() { return opaque_tid() == 0; }
__device__ __forceinline__ bool lane0() { return (opaque_tid() & 63) == 0; }
#define WAIT_V_(n) asm volatile("s_waitcnt vmcnt(" #n ")" ::: "memory")
#define WAIT_V(n) WAIT_V_(n)
#define WAIT_L(n) asm volatile("s_waitcnt lgkmcnt(" #n ")" ::: "memory")
#define BAR __builtin_amdgcn_s_barrier()
#define SCHED __builtin_amdgcn_sched_barrier(0)
#define STAMP(wsl, i) do { } while (0)

__device__ __forceinline__ void st16_sys(void* p, u32x4 v) {
    asm volatile("global_store_dwordx4 %0, %1, off sc0 sc1\n\ts_nop 1" :: "v"(p), "v"(v) : "memory");
}
#define OUT_ST(p, v) st16_sys(p, v)
__device__ __forceinline__ unsigned long long poll_once(const unsigned* p, bool need, unsigned call) {
    unsigned v;
    asm volatile("global_load_dword %0, %1, off sc0 sc1\n\ts_waitcnt vmcnt(0)" : "=v"(v) : "v"(p) : "memory");
    return __ballot(!need || geq(v, call));
}
__device__ __forceinline__ bool timed_out(const unsigned* tmo, unsigned call) {
    unsigned tv;
    asm volatile("global_load_dword %0, %1, off sc1\n\ts_waitcnt vmcnt(0)" : "=v"(tv) : "v"(tmo) : "memory");
    return __builtin_amdgcn_readfirstlane(tv) == call;
}
__device__ __forceinline__ bool spin_check(unsigned* tmo, unsigned call, unsigned long long t0, unsigned& spins) {
    if ((++spins & 7u) == 0u) {
        if (timed_out(tmo, call)) return true;
        if (__builtin_amdgcn_s_memrealtime() - t0 > DEADLINE) { st_agent(tmo, call); return true; }
    }
    __builtin_amdgcn_s_sleep(1);
    return false;
}
__device__ __forceinline__ void wait_all(const unsigned* p, bool need, unsigned call, unsigned* tmo) {
    if (poll_once(p, need, call) == ~0ull) return;
    const unsigned long long t0 = __builtin_amdgcn_s_memrealtime();
    unsigned spins = 0;
    for (;;) {
        if (spin_check(tmo, call, t0, spins)) return;
        if (poll_once(p, need, call) == ~0ull) return;
    }
}

__host__ __device__ __forceinline__ int lds_byte(int r, int c) { const int st = (r >> 4) * 2 + (c >> 5), rr = r & 15, cc = c & 31, ob = rr * 64 + cc * 2; return st * 1024 + (ob ^ (((ob >> 9) & 1) << 5)); }
__host__ __device__ __forceinline__ void stage_rc(int b, int& R, int& C) { const int st = b / 1024, sb = b % 1024, swz = sb ^ (((sb >> 9) & 1) << 5); R = (st >> 1) * 16 + swz / 64; C = (st & 1) * 32 + (swz % 64) / 2; }
__device__ __forceinline__ unsigned cvt_pk_bf16(float lo, float hi) { unsigned r; asm volatile("v_cvt_pk_bf16_f32 %0, %1, %2" : "=v"(r) : "v"(lo), "v"(hi)); return r; }
typedef float f32x2 __attribute__((ext_vector_type(2)));
__device__ __forceinline__ f32x2 gelu_tanh2(f32x2 y) {
    const float k1 = -2.0f * 0.7978845608028654f * 1.4426950408889634f;
    const float k2 = k1 * 0.044715f;
    const f32x2 t = y * ((y * y) * k2 + k1);
    f32x2 d; d.x = __builtin_amdgcn_exp2f(t.x); d.y = __builtin_amdgcn_exp2f(t.y);
    d = d + 1.0f;
    f32x2 r; r.x = __builtin_amdgcn_rcpf(d.x); r.y = __builtin_amdgcn_rcpf(d.y);
    return y * r;
}
__device__ __forceinline__ u32x2 gelu_pack(f32x4 v, unsigned bad) {
    const f32x2 a = gelu_tanh2((f32x2){v[0], v[1]}), b = gelu_tanh2((f32x2){v[2], v[3]});
    u32x2 w; w.x = cvt_pk_bf16(a.x, a.y); w.y = cvt_pk_bf16(b.x, b.y);
    if (bad) { w.x = 0x7fc07fc0u; w.y = 0x7fc07fc0u; }
    return w;
}

struct Tile {
    const char* A; const char* B; char* O;
    unsigned lda, csA;
    unsigned ldb;
    const unsigned* fA; const unsigned* fB;
    unsigned* fO; int nfO;
};

__device__ __forceinline__ unsigned poll_chunks(const unsigned* p, bool need, unsigned call) {
    u32x4 v;
    asm volatile("global_load_dwordx4 %0, %1, off sc0 sc1\n\ts_waitcnt vmcnt(0)" : "=v"(v) : "v"(p) : "memory");
    const unsigned long long mk = __ballot(!need || (geq(v.x, call) && geq(v.y, call) && geq(v.z, call) && geq(v.w, call)));
    const unsigned both = (unsigned)mk & (unsigned)(mk >> 32);
    return both & (both >> 1) & 0x55555555u;
}
__device__ __forceinline__ void ensure2(const unsigned* fA, const unsigned* fB, int c, int& ready, unsigned call, unsigned* tmo, int lane) {
    if (c >= ready) {
        const int cc = c + ((lane >> 1) & 15);
        const unsigned* base = lane < 32 ? fA : fB;
        const bool need = base != nullptr && cc < NCH;
        const unsigned* p = need ? base + cc * 8 + (lane & 1) * 4 : (const unsigned*)((const char*)tmo - WS_TMO + WS_FA);
        unsigned m = poll_chunks(p, need, call);
        if (!(m & 1u)) {
            const unsigned long long t0 = __builtin_amdgcn_s_memrealtime();
            unsigned spins = 0;
            for (;;) {
                if (spin_check(tmo, call, t0, spins)) { m = 0x55555555u; break; }
                m = poll_chunks(p, need, call);
                if (m & 1u) break;
            }
        }
        const unsigned miss = ~m & 0x55555555u;
        ready = c + (miss ? (__builtin_ctz(miss) >> 1) : 16);
    }
}
__device__ __forceinline__ void ensure(const Tile& tl, int c, int& ready, unsigned call, unsigned* tmo, int lane) { ensure2(tl.fA, tl.fB, c, ready, call, tmo, lane); }
#define RD128(dst, addr, off) asm volatile("ds_read_b128 %0, %1 offset:%2" : "=v"(dst) : "v"(addr), "i"(off) : "memory")
#define TRRD(dst, addr, off) asm volatile("ds_read_b64_tr_b16 %0, %1 offset:%2" : "=v"(dst) : "v"(addr), "i"(off) : "memory")
#define GLDS(g, l) __builtin_amdgcn_global_load_lds((const unsigned*)(g), (LAS unsigned*)(l), 16, 0, 0)

template <int N> __device__ __forceinline__ void wait_v() { asm volatile("s_waitcnt vmcnt(%0)" :: "n"(N) : "memory"); }
template <int N> __device__ __forceinline__ void wait_l() { asm volatile("s_waitcnt lgkmcnt(%0)" :: "n"(N > 15 ? 15 : N) : "memory"); }
template <int BM, int BN, int KT, int STG>
__device__ __forceinline__ void gemm_tile(LAS unsigned char* lds, const Tile& tl, const unsigned call, unsigned* tmo, int sbase) {
    char* const wsl_dbg = (char*)tmo - WS_TMO; (void)wsl_dbg; (void)sbase;
    constexpr int WAVES_M = BM == 256 ? 4 : 2, WAVES_N = 8 / WAVES_M;
    constexpr int MF = BM / WAVES_M / 16, NF = BN / WAVES_N / 16;
    constexpr int BROW = BN * 2, ABYTES = BM * 128, BBYTES = 64 * BROW, TBYTES = ABYTES + BBYTES, SBYTES = KT * TBYTES;
    constexpr int RA = ABYTES / 8192, RB = BBYTES / 8192;
    constexpr int LPS = (RA + RB) * KT;
    constexpr int NS = NT / KT;
    constexpr int KS = 2 * KT, RPK = MF + 2 * NF;
    static_assert(STG * SBYTES <= LDS_BYTES, "LDS");
    const int tid = threadIdx.x, wid = __builtin_amdgcn_readfirstlane(tid >> 6), lane = tid & 63;
    const int wr = wid / WAVES_N, wc = wid % WAVES_N, fr = lane & 15, fq = lane >> 4;
    unsigned voffA[RA], voffB[RB];
#pragma unroll
    for (int i = 0; i < RA; ++i) { int R, C; stage_rc(tid * 16 + i * 8192, R, C); voffA[i] = (unsigned)R * tl.lda + (unsigned)C * 2u; }
#pragma unroll
    for (int i = 0; i < RB; ++i) { const int b = tid * 16 + i * 8192;
        if (BN == 128) { const int row = b >> 8, chpos = (b >> 4) & 15, ch = chpos ^ (((row & 3) << 2) | ((row >> 2) & 3)); voffB[i] = (unsigned)row * tl.ldb + (unsigned)ch * 16u; }
        else { const int row = b >> 7, chpos = (b >> 4) & 7, ch = chpos ^ ((((row >> 1) & 1) | (((row >> 3) & 1) << 1)) << 1); voffB[i] = (unsigned)row * tl.ldb + (unsigned)ch * 16u; } }
    const size_t kstepB = (size_t)64 * tl.ldb;
    const unsigned ldsbase = (unsigned)(size_t)lds;
    const unsigned ldsw = (unsigned)wid * 1024u;
    const unsigned aoff = ldsbase + (unsigned)lds_byte(wr * (MF * 16) + fr, fq * 8);
    unsigned boff[NF][2];
    { const int g = lane >> 4, q = (lane & 15) >> 2, p = lane & 3;
#pragma unroll
      for (int n = 0; n < NF; ++n)
#pragma unroll
        for (int t = 0; t < 2; ++t) { const int row = 8 * g + 4 * t + q, ch = 2 * (wc * NF + n) + (p >> 1);
            const int sw = BN == 128 ? (((row & 3) << 2) | ((row >> 2) & 3)) : ((((row >> 1) & 1) | (((row >> 3) & 1) << 1)) << 1);
            boff[n][t] = ldsbase + (unsigned)(ABYTES + BROW * row + 16 * (ch ^ sw) + 8 * (p & 1)); } }

    f32x4 acc[MF][NF];
#pragma unroll
    for (int m = 0; m < MF; ++m)
#pragma unroll
        for (int n = 0; n < NF; ++n) acc[m][n] = (f32x4){0.f, 0.f, 0.f, 0.f};

    int ready = (tl.fA == nullptr && tl.fB == nullptr) ? NCH : 0;
#define STAGE_TILE(st, sb) do { _Pragma("unroll") for (int _j = 0; _j < KT; ++_j) { const int _kt = (st) * KT + _j; \
        const char* _a = tl.A + (size_t)(_kt >> 1) * tl.csA + (_kt & 1) * 128; const char* _b = tl.B + (size_t)_kt * kstepB; \
        _Pragma("unroll") for (int _i = 0; _i < RA; ++_i) GLDS(_a + voffA[_i], lds + (sb) + _j * TBYTES + ldsw + _i * 8192); \
        _Pragma("unroll") for (int _i = 0; _i < RB; ++_i) GLDS(_b + voffB[_i], lds + (sb) + _j * TBYTES + ABYTES + ldsw + _i * 8192); } } while (0)

    if (wid == 0) { for (int c = 0; c <= ((STG - 2) * KT) >> 1; ++c) ensure(tl, c, ready, call, tmo, lane); }
    BAR;
    STAMP(wsl_dbg, sbase);
#pragma unroll
    for (int st = 0; st < STG - 1; ++st) STAGE_TILE(st, st * SBYTES);
    unsigned sb_rd = 0, sb_wr = (STG - 1) * SBYTES;
    unsigned tmov = 0;
    for (int t = 0; t < NS; ++t) {
        const int tn = t + STG - 1 < NS ? t + STG - 1 : NS - 1;
        if (wid == 0 && (KT == 2 || (tn & 1) == 0)) ensure(tl, (tn * KT) >> 1, ready, call, tmo, lane);
        wait_v<LPS * (STG - 2)>();
        BAR;
        if (t == NS - STG + 1) asm volatile("global_load_dword %0, %1, off sc1" : "+v"(tmov) : "v"(tmo) : "memory");
        STAGE_TILE(tn, sb_wr);
        s16x8 Af[KS][MF]; s16x4 Bl[KS][NF], Bh[KS][NF];
        const unsigned a = aoff + sb_rd;
#pragma unroll
        for (int k = 0; k < KS; ++k) {
#pragma unroll
            for (int m = 0; m < MF; ++m) RD128(Af[k][m], a, (k >> 1) * TBYTES + m * 2048 + (k & 1) * 1024);
#pragma unroll
            for (int n = 0; n < NF; ++n) { TRRD(Bl[k][n], boff[n][0] + sb_rd, (k >> 1) * TBYTES + (k & 1) * 32 * BROW); TRRD(Bh[k][n], boff[n][1] + sb_rd, (k >> 1) * TBYTES + (k & 1) * 32 * BROW); }
        }
#pragma unroll
        for (int k = 0; k < KS; ++k) {
            if (k == 0) wait_l<(KS - 1) * RPK>(); else if (k == 1) wait_l<(KS - 2) * RPK>(); else if (k == 2) wait_l<(KS > 3 ? KS - 3 : 0) * RPK>(); else wait_l<0>();
            SCHED;
            __builtin_amdgcn_s_setprio(1);
#pragma unroll
            for (int m = 0; m < MF; ++m)
#pragma unroll
                for (int n = 0; n < NF; ++n) {
                    const s16x8 bf = __builtin_shufflevector(Bl[k][n], Bh[k][n], 0, 1, 2, 3, 4, 5, 6, 7);
                    acc[m][n] = __builtin_amdgcn_mfma_f32_16x16x32_bf16(__builtin_bit_cast(bf16x8, bf), __builtin_bit_cast(bf16x8, Af[k][m]), acc[m][n], 0, 0, 0);
                }
            __builtin_amdgcn_s_setprio(0);
            SCHED;
        }
        sb_rd = sb_rd == (STG - 1) * SBYTES ? 0 : sb_rd + SBYTES;
        sb_wr = sb_wr == (STG - 1) * SBYTES ? 0 : sb_wr + SBYTES;
    }
#undef STAGE_TILE
    WAIT_V(0);
    BAR;
    STAMP(wsl_dbg, sbase + 1);
    asm volatile("" : "+v"(tmov));
    const unsigned bad = (tl.fA != nullptr || tl.fB != nullptr) && (unsigned)__builtin_amdgcn_readfirstlane(tmov) == call;
    const int tz = opaque_tid();
    const int lane2 = tz & 63, fr2 = lane2 & 15, fq2 = lane2 >> 4;
    constexpr int CPR = BN / 8;
#pragma unroll
    for (int m = 0; m < MF; ++m) { const int row = wr * (MF * 16) + m * 16 + fr2;
#pragma unroll
        for (int n = 0; n < NF; ++n) {
            const int col = wc * (NF * 16) + n * 16 + fq2 * 4; const int ch = col >> 3, hf = (col >> 2) & 1;
            *(LAS u32x2*)(lds + row * BROW + ((ch ^ (fr2 & (CPR - 1))) << 4) + hf * 8) = gelu_pack(acc[m][n], bad); } }
    WAIT_L(0); BAR;
    constexpr int RPW = BM / 8, RPI = 64 / CPR;
    if (tl.fO != nullptr) {
#pragma unroll 4
        for (int i = 0; i < RPW / RPI; ++i) { const int row = wid * RPW + i * RPI + lane2 / CPR, chp = lane2 & (CPR - 1);
            const u32x4 v = *(const LAS u32x4*)(lds + row * BROW + chp * 16);
            st16_sys(tl.O + (size_t)row * 256 + ((chp ^ (row & (CPR - 1))) << 4), v); }
        WAIT_V(0);
        BAR;
        const int tf = opaque_tid();
        if (tf < tl.nfO) st_sys(tl.fO + tf, call);
    } else {
#pragma unroll 4
        for (int i = 0; i < RPW / RPI; ++i) { const int row = wid * RPW + i * RPI + lane2 / CPR, chp = lane2 & (CPR - 1);
            const u32x4 v = *(const LAS u32x4*)(lds + row * BROW + chp * 16);
            OUT_ST(tl.O + (size_t)row * 2048 + ((chp ^ (row & (CPR - 1))) << 4), v); }
        WAIT_L(0);
        BAR;
    }
    STAMP(wsl_dbg, sbase + 2);
}

__device__ __forceinline__ void gemm_tile_256(LAS unsigned char* lds, const Tile& tl, const unsigned call, unsigned* tmo) {
    char* const wsl_dbg = (char*)tmo - WS_TMO; (void)wsl_dbg;
    constexpr int HALF = 128, HTB = HALF * 64 * 2;
    const int tid = threadIdx.x, wid = __builtin_amdgcn_readfirstlane(tid >> 6), lane = tid & 63, wr = wid >> 2, wc = wid & 3, fr = lane & 15, fq = lane >> 4;
    unsigned voffA[2], voffB[2];
#pragma unroll
    for (int i = 0; i < 2; ++i) { int R, C; stage_rc(tid * 16 + i * 8192, R, C); voffA[i] = (unsigned)R * tl.lda + (unsigned)C * 2u;
        const int b = tid * 16 + i * 8192, row = b >> 8, chpos = (b >> 4) & 15, ch = chpos ^ (((row & 3) << 2) | ((row >> 2) & 3));
        voffB[i] = (unsigned)row * tl.ldb + (unsigned)ch * 16u; }
    const size_t kstepB = (size_t)64 * tl.ldb;
    const size_t hstepA = (size_t)HALF * tl.lda, hstepB = 256;
    const unsigned ldsw = (unsigned)wid * 1024u;
    const unsigned ldsbase = (unsigned)(size_t)lds;
    const unsigned aoff = ldsbase + (unsigned)lds_byte(wr * 64 + fr, fq * 8);
    unsigned boff[2][2];
    { const int g = lane >> 4, q = (lane & 15) >> 2, p = lane & 3;
#pragma unroll
      for (int n = 0; n < 2; ++n)
#pragma unroll
        for (int t = 0; t < 2; ++t) { const int row = 8 * g + 4 * t + q, ch = 4 * wc + 2 * n + (p >> 1);
            boff[n][t] = ldsbase + 4 * HTB + 256 * row + 16 * (ch ^ (((row & 3) << 2) | ((row >> 2) & 3))) + 8 * (p & 1); } }
#define SA(b, h) (((b) * 2 + (h)) * HTB)
#define SB(b, h) ((4 + (b) * 2 + (h)) * HTB)
#define STAGE(bufoff, gbase, voff) do { _Pragma("unroll") for (int _i = 0; _i < 2; ++_i) GLDS((const char*)(gbase) + (voff)[_i], lds + (bufoff) + ldsw + _i * 8192); } while (0)
#define LDA(dst, b, h) do { _Pragma("unroll") for (int m = 0; m < 4; ++m) _Pragma("unroll") for (int k = 0; k < 2; ++k) RD128(dst[m][k], aoff, SA(b, h) + m * 2048 + k * 1024); } while (0)
#define LDB(dst, b, h) do { _Pragma("unroll") for (int n = 0; n < 2; ++n) _Pragma("unroll") for (int k = 0; k < 2; ++k) { \
        s16x4 lo, hi; TRRD(lo, boff[n][0], ((b) * 2 + (h)) * HTB + k * 8192); TRRD(hi, boff[n][1], ((b) * 2 + (h)) * HTB + k * 8192); \
        dst[n][k] = __builtin_shufflevector(lo, hi, 0, 1, 2, 3, 4, 5, 6, 7); } } while (0)
#define MMA(ai, bj, At, Bt) do { __builtin_amdgcn_s_setprio(1); _Pragma("unroll") for (int m = 0; m < 4; ++m) _Pragma("unroll") for (int n = 0; n < 2; ++n) _Pragma("unroll") for (int k = 0; k < 2; ++k) \
        acc[ai][bj][m][n] = __builtin_amdgcn_mfma_f32_16x16x32_bf16(__builtin_bit_cast(bf16x8, Bt[n][k]), __builtin_bit_cast(bf16x8, At[m][k]), acc[ai][bj][m][n], 0, 0, 0); __builtin_amdgcn_s_setprio(0); } while (0)
#define KOFFA(t) ((size_t)((t) >> 1) * tl.csA + ((t) & 1) * 128)
    f32x4 acc[2][2][4][2];
#pragma unroll
    for (int a = 0; a < 2; ++a)
#pragma unroll
        for (int b = 0; b < 2; ++b)
#pragma unroll
            for (int m = 0; m < 4; ++m)
#pragma unroll
                for (int n = 0; n < 2; ++n) acc[a][b][m][n] = (f32x4){0.f, 0.f, 0.f, 0.f};
    s16x8 At[4][2], B0[2][2], B1[2][2];
    const char* cA = tl.A; const char* cB = tl.B;
    int ready = tl.fA == nullptr ? NCH : 0;
    if (wid == 0) ensure(tl, 0, ready, call, tmo, lane);
    BAR;
    STAMP(wsl_dbg, 1);
    STAGE(SB(0, 0), cB, voffB); STAGE(SA(0, 0), cA, voffA); STAGE(SB(0, 1), cB + hstepB, voffB); STAGE(SA(0, 1), cA + hstepA, voffA);
    if (wr == 1) BAR;
    WAIT_V(4); BAR;
    STAGE(SB(1, 0), cB + kstepB, voffB); STAGE(SA(1, 0), cA + 128, voffA); STAGE(SB(1, 1), cB + hstepB + kstepB, voffB);
    WAIT_V(6); BAR;
    for (int t = 0; t < NT; t += 2) {
        const int t2 = (t == NT - 2) ? 0 : t + 2;
        if (wid == 0) ensure(tl, t2 >> 1, ready, call, tmo, lane);
        const char* a1 = cA + KOFFA(t + 1);
        const char* a2 = cA + KOFFA(t2); const char* b2 = cB + (size_t)t2 * kstepB;
        const char* a3 = a2 + 128; const char* b3 = b2 + kstepB;
        LDB(B0, 0, 0); SCHED; LDA(At, 0, 0); STAGE(SA(1, 1), a1 + hstepA, voffA);
        WAIT_L(8); BAR; WAIT_L(0); SCHED; MMA(0, 0, At, B0); BAR; SCHED;
        LDB(B1, 0, 1); STAGE(SB(0, 0), b2, voffB);
        BAR; WAIT_L(0); SCHED; MMA(0, 1, At, B1); BAR; SCHED;
        LDA(At, 0, 1); STAGE(SA(0, 0), a2, voffA);
        BAR; WAIT_L(0); SCHED; MMA(1, 0, At, B0); BAR; SCHED;
        STAGE(SB(0, 1), b2 + hstepB, voffB);
        WAIT_V(6); BAR; MMA(1, 1, At, B1); BAR; SCHED;
        LDB(B0, 1, 0); SCHED; LDA(At, 1, 0); STAGE(SA(0, 1), a2 + hstepA, voffA);
        WAIT_L(8); BAR; WAIT_L(0); SCHED; MMA(0, 0, At, B0); BAR; SCHED;
        LDB(B1, 1, 1); STAGE(SB(1, 0), b3, voffB);
        BAR; WAIT_L(0); SCHED; MMA(0, 1, At, B1); BAR; SCHED;
        LDA(At, 1, 1); STAGE(SA(1, 0), a3, voffA);
        BAR; WAIT_L(0); SCHED; MMA(1, 0, At, B0); BAR; SCHED;
        STAGE(SB(1, 1), b3 + hstepB, voffB);
        WAIT_V(6); BAR; MMA(1, 1, At, B1); BAR; SCHED;
    }
    WAIT_V(0);
    if (wr == 0) BAR;
    BAR;
    STAMP(wsl_dbg, 2);
    unsigned bad = 0;
    if (tl.fA != nullptr) bad = timed_out(tmo, call);
    const int tz = opaque_tid();
    const int lane2 = tz & 63, fr2 = lane2 & 15, fq2 = lane2 >> 4;
#pragma unroll
    for (int ai = 0; ai < 2; ++ai)
#pragma unroll
        for (int m = 0; m < 4; ++m) { const int row = ai * HALF + wr * 64 + m * 16 + fr2;
#pragma unroll
            for (int bj = 0; bj < 2; ++bj)
#pragma unroll
                for (int n = 0; n < 2; ++n) {
                    const int col = bj * HALF + wc * 32 + n * 16 + fq2 * 4; const int ch = col >> 3, hf = (col >> 2) & 1;
                    *(LAS u32x2*)(lds + row * 512 + ((ch ^ fr2) << 4) + hf * 8) = gelu_pack(acc[ai][bj][m][n], bad); } }
    WAIT_L(0); BAR;
#pragma unroll 4
    for (int i = 0; i < 16; ++i) { const int row = wid * 32 + i * 2 + (lane2 >> 5), chp = lane2 & 31;
        const u32x4 v = *(const LAS u32x4*)(lds + row * 512 + chp * 16);
        OUT_ST(tl.O + (size_t)row * 2048 + ((chp ^ (row & 15)) << 4), v); }
    WAIT_L(0);
    BAR;
    STAMP(wsl_dbg, 3);
#undef SA
#undef SB
#undef STAGE
#undef LDA
#undef LDB
#undef MMA
#undef KOFFA
}

__device__ __forceinline__ int block_row(int r, int rb) {
    if (rb < 4) return r * 1024 + rb * 256;
    if (rb < 8) return (r ^ 1) * 1024 + (rb - 4) * 256;
    const int j = (rb - 8) >> 1, xv = (j & 1) + 2 + (j >> 1) * 4;
    return (r ^ xv) * 1024 + (r >> 2) * 512 + (rb & 1) * 256;
}
__device__ __forceinline__ int g1_block(int g1) { return g1 < 2 ? g1 : (g1 == 2 ? 4 : (g1 < 7 ? g1 + 5 : (g1 < 9 ? g1 - 5 : (g1 == 9 ? 6 : g1 + 2)))); }
__device__ __forceinline__ int link_xor(int L) { return L == 0 ? 1 : (L < 5 ? ((L - 1) & 1) + 2 + ((L - 1) >> 1) * 4 : L - 1); }
__device__ __forceinline__ int xlink_index(int xv) { return (xv & 1) + ((xv >> 2) & 1) * 2; }
__device__ __forceinline__ Tile make_tile(const Params& p, char* wsl, size_t paroff, int rb, int sub, int cb, int ou) {
    Tile t; const int ro = sub < 0 ? 0 : sub * 64; char* par = wsl + paroff; const int r = p.rank;
    if (rb < 4) { t.A = p.x + (size_t)(rb * 256 + ro) * XROW; t.lda = XROW; t.csA = 256; t.fA = nullptr; }
    else { t.A = par + D_XR + (size_t)(rb - 4) * NCH * UNIT + ro * 256; t.lda = 256; t.csA = (unsigned)UNIT; t.fA = (const unsigned*)(wsl + WS_FA) + rb * NCH * 8; }
    if (cb < 8) {
        t.B = p.w + cb * 256; t.ldb = 2048; t.fB = nullptr;
        t.O = p.out + (size_t)(block_row(r, rb) + ro) * 2048 + cb * 256; t.fO = nullptr; t.nfO = 0;
    } else {
        const int h = (cb - 8) >> 2, cl = cb & 3;
        const int cp = h * 2 + (cl >> 1);
        t.B = par + D_WR + (size_t)cp * NCH * UNIT + (cl & 1) * 256; t.ldb = 512;
        t.fB = (const unsigned*)(wsl + WS_FB) + cp * NCH * 8;
        const int g = ou * 4 + cl, uo = h * 64 + g;
        t.O = par + D_STG + (size_t)uo * UNIT + ro * 256;
        t.nfO = sub < 0 ? 8 : 2;
        if (g < NDIR1 || g >= 56)
            t.fO = (unsigned*)(p.ws[r ^ (4 + h)] + WS_AU) + ((5 + h) * 64 + (g < NDIR1 ? g : g - 4 * RPF)) * 8 + (sub < 0 ? 0 : sub * 2);
        else { const int jx = (g - NDIR1) & 3, k = (g - NDIR1) >> 2;
            t.fO = (unsigned*)(p.ws[r ^ link_xor(jx + 1)] + WS_AU) + ((jx + 1) * 64 + (26 - (r & 1)) + k * 2 + h) * 8; }
    }
    return t;
}

struct Slice { const char* src; char* dst; unsigned sstride, dstride; int sh; int dep; bool push; unsigned* f0; unsigned* f1; unsigned* f2; };

constexpr int NPUBU = (NCH - NPUSH) * 8 + 2 * ((NPUSH + 4) / 5);
constexpr int NS1 = 2 * NCH;
__device__ __forceinline__ int link_late(int r, int L) { return L == 0 ? 26 : 26 - ((r ^ link_xor(L)) & 1); }
__device__ __forceinline__ int link_total(int r, int L) { return 8 * (L == 7 ? NPUBU : (L >= 5 ? NS1 + NDIR1 + 8 : NS1 + link_late(r, L) + (L == 0 ? 0 : 4 * RPF))); }
__device__ __forceinline__ void make_slice(const Params& p, char* wsl, size_t paroff, int L, int tk, Slice& u) {
    const int r = p.rank;
    char* par = wsl + paroff;
    const int un = tk >> 3, sl = tk & 7;
    unsigned* const fA = (unsigned*)(wsl + WS_FA);
    u.dep = -1; u.push = false; u.sh = 4; u.sstride = 256; u.dstride = 256; u.f0 = nullptr; u.f1 = nullptr; u.f2 = nullptr;
    if (L == 7) {
        int c = (un >> 3) + NPUSH, b = un & 7;
        if (un >= (NCH - NPUSH) * 8) { const int e = un - (NCH - NPUSH) * 8; c = (e >> 1) * 5; b = 1 + 2 * (e & 1); }
        if (b < 4) {
            u.src = p.x + (size_t)(b * 256 + sl * 32) * XROW + c * 256; u.sstride = XROW;
            const int hi = ((r >> 2) ^ (b >> 1)) & 1;
            const int a1 = r ^ (2 + 4 * hi), a2 = r ^ (3 + 4 * hi), rb1 = 8 + 4 * hi + (b & 1), rb2 = rb1 + 2;
            unsigned* mate_late = (unsigned*)(p.ws[r ^ 1] + WS_AU) + (2 * (c / 5) + (b >> 1)) * 8 + sl;
            u.dst = par + D_PX + ((size_t)b * NCH + c) * UNIT + sl * 8192;
            if ((b & 1) && c % 5 == 0) u.f0 = mate_late;
            if (c >= NPUSH) {
                if (!(b & 1)) u.f0 = (unsigned*)(p.ws[r ^ 1] + WS_AA) + ((4 + b) * NCH + c) * 8 + sl;
                u.f1 = (unsigned*)(p.ws[a1] + WS_AA) + (rb1 * NCH + c) * 8 + sl;
                u.f2 = (unsigned*)(p.ws[a2] + WS_AA) + (rb2 * NCH + c) * 8 + sl;
            }
        } else {
            const int cp = b - 4, J = r ^ (4 + (cp >> 1));
            u.src = p.w + (size_t)(c * 128 + sl * 16) * 2048 + cp * 512; u.sstride = 2048; u.dstride = 512; u.sh = 5;
            u.dst = par + D_PW + ((size_t)cp * NCH + c) * UNIT + sl * 8192;
            if (c >= NPUSH) u.f0 = (unsigned*)(p.ws[J] + WS_AB) + (cp * NCH + c) * 8 + sl;
        }
        return;
    }
    const int xv = link_xor(L);
    const int peer = r ^ xv;
    const char* ppar = p.ws[peer] + paroff;
    if (un < 2 * NPUSH) {
        const int c = un >> 1, b = un & 1;
        char* pp = p.ws[peer] + paroff;
        u.push = true;
        if (L < 5) {
            const int blk = L == 0 ? 2 * b : 2 * (peer >> 2) + b;
            const int rb = L == 0 ? 4 + 2 * b : 8 + 2 * (L - 1) + b;
            u.src = p.x + (size_t)(blk * 256 + sl * 32) * XROW + c * 256; u.sstride = XROW;
            u.dst = pp + D_XR + ((size_t)(rb - 4) * NCH + c) * UNIT + sl * 8192;
            u.f0 = (unsigned*)(p.ws[peer] + WS_FA) + (rb * NCH + c) * 8 + sl;
            if (L != 0 && b == 1) { const int c5 = c % 5 - 1 - (peer & 1);
                if (c5 == 0 || c5 == 2) u.f1 = (unsigned*)(p.ws[r ^ 1] + WS_AU) + ((1 + xlink_index(r ^ 1 ^ peer)) * 64 + 2 * (c / 5) + (c5 >> 1)) * 8 + sl; }
        } else {
            const int cp = (L - 5) * 2 + b;
            u.src = p.w + (size_t)(c * 128 + sl * 16) * 2048 + cp * 512; u.sstride = 2048; u.dstride = 512; u.sh = 5;
            u.dst = pp + D_WR + ((size_t)cp * NCH + c) * UNIT + sl * 8192;
            u.f0 = (unsigned*)(p.ws[peer] + WS_FB) + (cp * NCH + c) * 8 + sl;
        }
        return;
    }
    if (un < NS1) {
        const int c = un >> 1, b = un & 1;
        if (L < 5) {
            const int blk = L == 0 ? 2 * b : 2 * (r >> 2) + b;
            const int rb = L == 0 ? 4 + 2 * b : 8 + 2 * (L - 1) + b;
            u.src = ppar + D_PX + ((size_t)blk * NCH + c) * UNIT + sl * 8192;
            u.dst = par + D_XR + ((size_t)(rb - 4) * NCH + c) * UNIT + sl * 8192;
            u.f0 = fA + (rb * NCH + c) * 8 + sl;
            if (L != 0 && b == 1) { const int c5 = c % 5 - 1 - (r & 1);
                if (c5 == 0 || c5 == 2) u.f1 = (unsigned*)(p.ws[peer ^ 1] + WS_AU) + ((1 + xlink_index(peer ^ 1 ^ r)) * 64 + 2 * (c / 5) + (c5 >> 1)) * 8 + sl; }
        } else {
            const int cp = (L - 5) * 2 + b;
            u.src = ppar + D_PW + ((size_t)cp * NCH + c) * UNIT + sl * 8192;
            u.dst = par + D_WR + ((size_t)cp * NCH + c) * UNIT + sl * 8192;
            u.f0 = (unsigned*)(wsl + WS_FB) + (cp * NCH + c) * 8 + sl;
        }
        return;
    }
    int m = un - NS1;
    u.dep = m;
    if (L >= 5) {
        const int h = L - 5, g = m < NDIR1 ? m : m + 4 * RPF, ou = g >> 2, cl = g & 3, uo = h * 64 + g;
        const int rb = ou < 14 ? g1_block(ou) : 5 + 2 * (ou - 14);
        u.src = ppar + D_STG + (size_t)uo * UNIT + sl * 8192;
        u.dst = p.out + (size_t)(block_row(peer, rb) + sl * 32) * 2048 + h * 1024 + cl * 256; u.dstride = 2048;
        return;
    }
    const int nlate = link_late(r, L);
    if (m < nlate) {
        int c, blk;
        if (L == 0) { c = (m >> 1) * 5; blk = 1 + 2 * (m & 1); u.src = ppar + D_PX + ((size_t)blk * NCH + c) * UNIT + sl * 8192; }
        else { c = (m >> 1) * 5 + 1 + (peer & 1) + 2 * (m & 1); blk = 2 * (peer >> 2) + 1;
            const int rbl = 8 + 2 * xlink_index(peer ^ r ^ 1) + 1;
            u.src = ppar + D_XR + ((size_t)(rbl - 4) * NCH + c) * UNIT + sl * 8192; }
        u.dst = par + D_XR + ((size_t)blk * NCH + c) * UNIT + sl * 8192;
        u.f0 = fA + ((4 + blk) * NCH + c) * 8 + sl;
        return;
    }
    m -= nlate;
    const int h = m & 1;
    if (m < 2 * RPF) {
        const int k = m >> 1, g = NDIR1 + k * 4 + (L - 1), uo = h * 64 + g, slot = ((L - 1) * 2 + h) * RPF + k, J = peer ^ (4 + h);
        u.src = ppar + D_STG + (size_t)uo * UNIT + sl * 8192;
        u.dst = par + D_RB + (size_t)slot * UNIT + sl * 8192;
        u.f0 = (unsigned*)(p.ws[J] + WS_AU) + ((1 + xlink_index(J ^ r)) * 64 + (26 - (r & 1)) + 2 * RPF + m) * 8 + sl;
    } else {
        const int k = (m - 2 * RPF) >> 1, rs = r ^ (4 + h), jx = xlink_index(rs ^ peer);
        const int g = NDIR1 + k * 4 + jx, ou = g >> 2, cl = g & 3, slot = (jx * 2 + h) * RPF + k;
        u.src = ppar + D_RB + (size_t)slot * UNIT + sl * 8192;
        u.dst = p.out + (size_t)(block_row(rs, g1_block(ou)) + sl * 32) * 2048 + h * 1024 + cl * 256; u.dstride = 2048;
    }
}
template <int I0, int I1>
__device__ __forceinline__ void slice_load(const Slice& u, u32x4 (&v)[8], int lane) {
    const char* s = u.src + (size_t)(lane >> u.sh) * u.sstride + (lane & ((1 << u.sh) - 1)) * 16;
    const size_t st = (size_t)(64 >> u.sh) * u.sstride;
#pragma unroll
    for (int i = I0; i < I1; ++i) asm volatile("global_load_dwordx4 %0, %1, off sc0 sc1" : "=v"(v[i]) : "v"(s + i * st) : "memory");
}
template <int I0, int I1>
__device__ __forceinline__ void slice_store(const Slice& u, u32x4 (&v)[8], int lane) {
    char* d = u.dst + (size_t)(lane >> u.sh) * u.dstride + (lane & ((1 << u.sh) - 1)) * 16;
    const size_t st = (size_t)(64 >> u.sh) * u.dstride;
#pragma unroll
    for (int i = I0; i < I1; ++i) st16_sys(d + i * st, v[i]);
}
#define PIN8(v) asm volatile("" : "+v"(v[0]), "+v"(v[1]), "+v"(v[2]), "+v"(v[3]), "+v"(v[4]), "+v"(v[5]), "+v"(v[6]), "+v"(v[7]))

__device__ __forceinline__ void copy_wave(const Params& p, char* wsl, size_t paroff, int L, int j, int nw, unsigned call, unsigned* tmo) {
    const int lane = threadIdx.x & 63;
    const int total = link_total(p.rank, L);
    unsigned* ctr = (unsigned*)(wsl + WS_Q + 128 * L);
    const unsigned tbase = (call - 1u) * (unsigned)total - (unsigned)nw;
    u32x4 va[8], vb[8];
    Slice ua, ub;
    unsigned tkv = 0;
#define DRAW() do { if (lane0()) asm volatile("global_atomic_add %0, %1, %2, off sc0" : "+v"(tkv) : "v"(ctr), "v"(1u) : "memory"); } while (0)
#define PUBLISH(a, b, c) do { { const int _l = opaque_tid() & 63; unsigned* _f = _l == 0 ? (a) : (_l == 1 ? (b) : (c)); if (_l < 3 && _f != nullptr) st_sys(_f, call); } } while (0)
    unsigned* p0 = nullptr; unsigned* p1 = nullptr; unsigned* p2 = nullptr;
    const unsigned* av0 = L < 5 ? (const unsigned*)(wsl + WS_AA) + (L == 0 ? 4 : 8 + 2 * (L - 1)) * NCH * 8 : (const unsigned*)(wsl + WS_AB) + (L - 5) * 2 * NCH * 8;
    const unsigned* av1 = av0 + (L == 0 ? 2 : 1) * NCH * 8;
    const unsigned* avU = (const unsigned*)(wsl + WS_AU) + L * 64 * 8;
    int aready = L == 7 ? NCH : 0, uready = 0;
    make_slice(p, wsl, paroff, L, j, ua);
    DRAW();
    if (L < 7 && j >= 16 * NPUSH) ensure2(av0, av1, j >> 4, aready, call, tmo, lane);
    slice_load<0, 8>(ua, va, lane);
#define STEP(cur, vcur, nxt, vnxt) { \
        WAIT_V(0); \
        PIN8(vcur); asm volatile("" : "+v"(tkv)); \
        PUBLISH(p0, p1, p2); \
        const int tk = (int)((unsigned)__builtin_amdgcn_readfirstlane(tkv) - tbase); \
        slice_store<0, 4>(cur, vcur, lane); \
        if (!cur.push) slice_store<4, 8>(cur, vcur, lane); \
        p0 = cur.f0; p1 = cur.f1; p2 = cur.f2; \
        if (tk >= total) { if (cur.push) { WAIT_V(0); slice_store<4, 8>(cur, vcur, lane); } break; } \
        make_slice(p, wsl, paroff, L, tk, nxt); \
        const bool w1 = L < 7 && tk >= 16 * NPUSH && tk < NS1 * 8 && (tk >> 4) >= aready, w2 = nxt.dep >= uready; \
        if (cur.push && (w1 || w2 || !nxt.push)) { WAIT_V(0); slice_store<4, 8>(cur, vcur, lane); cur.push = false; } \
        if (w1 || w2) { \
            WAIT_V(0); PUBLISH(p0, p1, p2); p0 = nullptr; p1 = nullptr; p2 = nullptr; \
            if (w1) ensure2(av0, av1, tk >> 4, aready, call, tmo, lane); \
            else ensure2(avU, nullptr, nxt.dep, uready, call, tmo, lane); } \
        DRAW(); \
        { const bool rem = HS < 8 && L < 7 && !nxt.push;       \
          slice_load<0, 2>(nxt, vnxt, lane); if (rem && HS == 2) WAIT_V(0); \
          slice_load<2, 4>(nxt, vnxt, lane); if (rem) WAIT_V(0); \
          slice_load<4, 6>(nxt, vnxt, lane); if (rem && HS == 2) WAIT_V(0); \
          slice_load<6, 8>(nxt, vnxt, lane); } \
        if (cur.push) { WAIT_V(0); slice_store<4, 8>(cur, vcur, lane); } \
        if (L < 7 && tk >= NS1 * 8 - 24 && tk < NS1 * 8) STAMP(wsl, 2); \
        if (L < 5 && tk >= (NS1 + 25) * 8 - 24 && tk < (NS1 + 25) * 8) STAMP(wsl, 4); \
        if (L >= 1 && L < 5 && tk >= (NS1 + 26 + 2 * RPF) * 8 - 24 && tk < (NS1 + 26 + 2 * RPF) * 8) STAMP(wsl, 5); \
        if (L >= 5 && L < 7 && tk >= (NS1 + NDIR1) * 8 - 24 && tk < (NS1 + NDIR1) * 8) STAMP(wsl, 5); }
    for (;;) {
        STEP(ua, va, ub, vb)
        STEP(ub, vb, ua, va)
    }
#undef STEP
#undef DRAW
    WAIT_V(0);
    PUBLISH(p0, p1, p2);
#undef PUBLISH
    if (L < 7) STAMP(wsl, 3);
}

__global__ __launch_bounds__(NTHREADS, 2) void ag_gemm_kernel(const Params p) {
    extern __shared__ __attribute__((aligned(16))) unsigned char shm[];
    LAS unsigned char* lds = (LAS unsigned char*)shm;
    char* wsl = p.ws[p.rank];
    unsigned* tmo = (unsigned*)(wsl + WS_TMO);
    const int bid = blockIdx.x, tid = threadIdx.x;
    unsigned call;
    {
        unsigned* ctr = (unsigned*)(wsl + WS_CTR + (size_t)bid * 64);
        unsigned v;
        asm volatile("global_load_dword %0, %1, off sc1\n\ts_waitcnt vmcnt(0)" : "=v"(v) : "v"(ctr) : "memory");
        call = __builtin_amdgcn_readfirstlane(v) + 1u;
        if (call == 0u) call = 1u;
        __syncthreads();
        if (thread0()) st_agent(ctr, call);
    }
    const size_t paroff = WS_DATA + (size_t)(call & 1u) * PAR_BYTES;
    STAMP(wsl, 0);
    if (bid < FOR0) {
        const int wid = __builtin_amdgcn_readfirstlane(tid >> 6);
        if (wid < AWP) { const int gw = bid * AWP + wid, L = gw % 7; if (gw >= NPW) return; copy_wave(p, wsl, paroff, L, gw / 7, (NPW - L + 6) / 7, call, tmo);
            if (L >= 1 && timed_out(tmo, call) && lane0()) *(u16*)(p.out + (size_t)p.rank * 1024 * 2048) = 0x7fc0; }
        else if (wid < AWP + NPUB && bid * NPUB + wid - AWP < NPUBW) copy_wave(p, wsl, paroff, 7, bid * NPUB + wid - AWP, NPUBW, call, tmo);
        return;
    }
    if (bid < OWN0) {
        const int g = bid - FOR0, xcd = g & 7, slot = g >> 3;
        const int g1 = (xcd >> 2) * 7 + slot % 7, cb = 8 + (xcd & 3) * 2 + slot / 7;
        const Tile t = make_tile(p, wsl, paroff, g1_block(g1), -1, cb, g1);
        gemm_tile<256, 128, 1, 3>(lds, t, call, tmo, 1);
    } else if (bid < EXT0) {
        const int g = bid - OWN0, xcd = g & 7, slot = g >> 3;
        const int g1 = (xcd >> 2) * 7 + slot, cb = (xcd & 3) * 2;
        const Tile t = make_tile(p, wsl, paroff, g1_block(g1), -1, cb, g1);
        gemm_tile_256(lds, t, call, tmo);
    }
    const int xq = bid & 7;
    unsigned* q = (unsigned*)(wsl + WS_Q + 128 * (8 + xq));
    const unsigned qbase = (call - 1u) * (unsigned)(24 + NGEMM / 8);
    for (;;) {
        LAS unsigned* sw = (LAS unsigned*)(lds + LDS_BYTES - 16);
        if (thread0()) *sw = __hip_atomic_fetch_add(q, 1u, __ATOMIC_RELAXED, __HIP_MEMORY_SCOPE_AGENT) - qbase;
        __syncthreads();
        const unsigned tk = __builtin_amdgcn_readfirstlane(*sw);
        __syncthreads();
        if (tk >= 24u) break;
        if (tk >= 8u) {
            const int g2 = (tk - 8) >> 1, hf = tk & 1;
            Tile t = make_tile(p, wsl, paroff, 5 + 2 * (g2 >> 2), g2 & 3, 8 + xq, 14 + (g2 >> 2));
            t.B += hf * 128; t.O += hf * 128; t.fO += hf; t.nfO = 1;
            gemm_tile<64, 64, 2, 4>(lds, t, call, tmo, 4);
        } else {
            const Tile t = make_tile(p, wsl, paroff, 5 + 2 * (tk >> 2), tk & 3, xq, 14 + (tk >> 2));
            gemm_tile<64, 128, 2, 3>(lds, t, call, tmo, 4);
        }
    }
}

}

extern "C" void kernel_launch(int rank, int nranks, ncclComm_t comm,
                              hipStream_t stream,
                              void* const* d_in, const int* in_sizes, int n_in,
                              void* d_out, int out_size,
                              const int* in_dtypes, int out_dtype,
                              void* const* d_out_all,
                              void* const* d_ws_all, size_t ws_bytes) {
    using namespace agk;
    if (nranks != NR || (unsigned)rank >= (unsigned)NR || ws_bytes < WS_NEED) { fprintf(stderr, "unsupported configuration\n"); return; }
    static bool attr_set[64] = {};
    int dev = 0; HIP_CHECK(hipGetDevice(&dev));
    if (dev >= 0 && dev < 64 && !attr_set[dev]) {
        HIP_CHECK(hipFuncSetAttribute((const void*)ag_gemm_kernel, hipFuncAttributeMaxDynamicSharedMemorySize, LDS_BYTES));
        attr_set[dev] = true;
    }
    Params p;
    p.x = (const char*)d_in[0]; p.w = (const char*)d_in[1]; p.out = (char*)d_out; p.rank = rank;
    for (int i = 0; i < NR; ++i) { p.ws[i] = (char*)d_ws_all[i]; p.outs[i] = (char*)d_out_all[i]; }
    hipLaunchKernelGGL(ag_gemm_kernel, dim3(NWG), dim3(NTHREADS), LDS_BYTES, stream, p);
    HIP_CHECK(hipGetLastError());
}
```
